```python
import numpy as np

kernel_name = "gpu_gemm_rs_m4096_k4096_n16384_f32_none"


ABSMAX_RTOL = 0.001

COMPUTE_DTYPE = "float32"
EPILOGUE = "none"
IN_QUANT = None
OUT_QUANT = None
IN_DTYPES = {"x": "float32", "w_mat": "float32"}
WORLD_SIZE = 8

def setup_inputs():

    x = np.random.randn(4096, 4096).astype(np.float32)
    w_mat = (np.random.randn(4096, 16384) / np.sqrt(4096)).astype(np.float32)
    return {"x": x, "w_mat": w_mat}

def _epilogue(y):
    return y

def reference(x, w_mat):
    y = (x @ w_mat).astype(np.float32)
    return _epilogue(y)
```

```cpp
#include <hip/hip_runtime.h>
#include <rccl/rccl.h>
#include <cstdio>

#define HIP_CHECK(expr) do { hipError_t _e = (expr); if (_e != hipSuccess) { \
    fprintf(stderr, "HIP error %s at %s:%d\n", hipGetErrorString(_e), __FILE__, __LINE__); \
    return; } } while (0)

namespace {

typedef unsigned u4 __attribute__((ext_vector_type(4)));
typedef float f4 __attribute__((ext_vector_type(4)));
typedef _Float16 h8 __attribute__((ext_vector_type(8)));
#define LAS __attribute__((address_space(3)))

constexpr int NR = 8;
constexpr int MG = 4096, NG = 16384, KS = 512, NS = 2048, MS = 512;
constexpr int NWG = 256, NSEND = 32;
constexpr unsigned FLG_OFF = 0;
constexpr unsigned DONE_OFF = 1024;
constexpr unsigned TMO_OFF = 2048;
constexpr unsigned SEQ_OFF = 4096;
constexpr unsigned STAMP_OFF = 65536;
constexpr unsigned STG_OFF = 1u << 20;
constexpr unsigned A_BYTES = 32u << 20, B_BYTES = 16u << 20, SLOT_BYTES = A_BYTES + B_BYTES;
constexpr unsigned WS_NEED = STG_OFF + 2 * SLOT_BYTES;
constexpr unsigned OUT_BYTES = (unsigned)MS * NG * 4;
constexpr unsigned long long DEADLINE = 100000000ull;
constexpr int AUX_SYS = 17;
constexpr int AUX_LD = 16;
constexpr int STAGE_A = 32768, STAGE_B = 16384, STAGE = STAGE_A + STAGE_B, NSTAGE = 3;
constexpr int CTL = NSTAGE * STAGE;
constexpr int LDS_BYTES = CTL + 128;

struct Params {
    const float* x; const float* w;
    char* ws[NR]; float* out[NR];
    int rank;
};

#define WAIT_V(n) asm volatile("s_waitcnt vmcnt(" #n ")" ::: "memory")
#define BAR() do { asm volatile("s_waitcnt lgkmcnt(0)" ::: "memory"); __builtin_amdgcn_s_barrier(); asm volatile("" ::: "memory"); } while (0)

#define STAMP(i) do {} while (0)

__device__ __forceinline__ __amdgpu_buffer_rsrc_t mk_rsrc(const void* p, unsigned bytes) {
    return __builtin_amdgcn_make_buffer_rsrc((void*)p, 0, (int)bytes, 0x00020000);
}
__device__ __forceinline__ void st16(u4 v, __amdgpu_buffer_rsrc_t r, unsigned off) {
    __builtin_amdgcn_raw_buffer_store_b128(v, r, (int)off, 0, AUX_SYS);
}
__device__ __forceinline__ u4 cvt8(f4 a, f4 b) {
    h8 h = {(_Float16)a[0], (_Float16)a[1], (_Float16)a[2], (_Float16)a[3], (_Float16)b[0], (_Float16)b[1], (_Float16)b[2], (_Float16)b[3]};
    return __builtin_bit_cast(u4, h);
}

__global__ __launch_bounds__(512, 2) void gemm_rs_kernel(const Params p) {
    __shared__ __attribute__((aligned(16))) char smem[LDS_BYTES];
    const int tid = threadIdx.x, lane = tid & 63;
    const int wid = __builtin_amdgcn_readfirstlane(tid >> 6);
    const int b = blockIdx.x, rank = p.rank;
    char* const myws = p.ws[rank];
    LAS unsigned* const ctl = (LAS unsigned*)((LAS char*)smem + CTL);
    STAMP(0);

    if (tid == 0) {
        const unsigned c = __hip_atomic_fetch_add((unsigned*)(myws + SEQ_OFF + b * 128), 1u, __ATOMIC_RELAXED, __HIP_MEMORY_SCOPE_AGENT) + 1u;
        ctl[18] = c; ctl[19] = 0;
    }
    if (tid >= 64 && tid < 80) ctl[tid - 64] = 0;

    const int xcd = b & 7, slot = b >> 3;
    const int orel = slot >> 2;
    const int owner = (rank + orel) & 7;
    const int mt = owner * 2 + (xcd & 1), nt = (xcd >> 1) * 4 + (slot & 3);
    unsigned call;

    if (b < NSEND) {
        const int j = wid;
        const float* xs = p.x + (size_t)((j * 32 + b) * 16 + (lane & 15)) * KS + (lane >> 4) * 8;
        const float* wsrc = p.w + (size_t)((lane >> 4) * 8) * NG + j * NS + b * 64 + (lane & 15) * 4;
        f4 xr[4], wr[16];
        auto load_chunk = [&](int c) {
#pragma unroll
            for (int ks = 0; ks < 2; ++ks) {
                xr[ks * 2] = *(const f4*)(xs + c * 64 + ks * 32);
                xr[ks * 2 + 1] = *(const f4*)(xs + c * 64 + ks * 32 + 4);
            }
#pragma unroll
            for (int ks = 0; ks < 2; ++ks)
#pragma unroll
                for (int i = 0; i < 8; ++i) wr[ks * 8 + i] = __builtin_nontemporal_load((const f4*)(wsrc + (size_t)(c * 64 + ks * 32 + i) * NG));
        };
        load_chunk(0);
        BAR();
        call = __builtin_amdgcn_readfirstlane(ctl[18]);
        const unsigned stg = STG_OFF + (call & 1u) * SLOT_BYTES;
        const unsigned xoff = stg + (unsigned)(rank * 8) * (16u * STAGE_A) + (unsigned)(j * 32 + b) * 2048u;
        const unsigned woff = stg + A_BYTES + (unsigned)(rank * 8) * (16u * STAGE_B) + (unsigned)b * 8192u;
        const int voff = lane * 16;
        const __amdgpu_buffer_rsrc_t rj = mk_rsrc(p.ws[j], WS_NEED);
        auto signal = [&](int c) {
            unsigned old = 0;
            if (lane == 0) old = __hip_atomic_fetch_add(ctl + c, 1u, __ATOMIC_RELAXED, __HIP_MEMORY_SCOPE_WORKGROUP);
            old = __builtin_amdgcn_readfirstlane(old);
            if (old == 7u && lane < NR)
                __hip_atomic_fetch_add((unsigned*)(p.ws[lane] + FLG_OFF + rank * 128 + c * 4), 1u, __ATOMIC_RELAXED, __HIP_MEMORY_SCOPE_SYSTEM);
        };
        for (int c = 0; c < 8; ++c) {
            u4 xh[2], wh[8];
#pragma unroll
            for (int ks = 0; ks < 2; ++ks) xh[ks] = cvt8(xr[ks * 2], xr[ks * 2 + 1]);
#pragma unroll
            for (int ks = 0; ks < 2; ++ks)
#pragma unroll
                for (int jj = 0; jj < 4; ++jj) {
                    const f4 lo = {wr[ks * 8][jj], wr[ks * 8 + 1][jj], wr[ks * 8 + 2][jj], wr[ks * 8 + 3][jj]};
                    const f4 hi = {wr[ks * 8 + 4][jj], wr[ks * 8 + 5][jj], wr[ks * 8 + 6][jj], wr[ks * 8 + 7][jj]};
                    wh[ks * 4 + jj] = cvt8(lo, hi);
                }
            asm volatile("" ::: "memory");
            if (c < 7) load_chunk(c + 1);
            asm volatile("" ::: "memory");
#pragma unroll
            for (int ks = 0; ks < 2; ++ks) *(LAS u4*)((LAS char*)smem + j * 16384 + (c * 2 + ks) * 1024 + voff) = xh[ks];
            const unsigned xo = xoff + (unsigned)c * (16u * STAGE_A), wo = woff + (unsigned)c * (16u * STAGE_B);
#pragma unroll
            for (int q = 0; q < 8; ++q) {
                st16(wh[q], rj, wo + voff + q * 1024);
                if (q < 7) {
                    const int d = (j + 1 + q) & 7;
                    const __amdgpu_buffer_rsrc_t rd = mk_rsrc(p.ws[d], WS_NEED);
                    st16(xh[0], rd, xo + voff);
                    st16(xh[1], rd, xo + voff + 1024);
                }
            }
            WAIT_V(0);
            signal(c);
            STAMP(1 + c);
            BAR();
        }
        {
            u4 xh[16];
#pragma unroll
            for (int q = 0; q < 16; ++q) xh[q] = *(LAS u4*)((LAS char*)smem + j * 16384 + q * 1024 + voff);
#pragma unroll
            for (int q = 0; q < 16; ++q)
                st16(xh[q], rj, xoff + (unsigned)(q >> 1) * (16u * STAGE_A) + voff + (q & 1) * 1024);
            WAIT_V(0);
            signal(8);
            STAMP(9);
        }
        BAR();
    } else {
        BAR();
        call = __builtin_amdgcn_readfirstlane(ctl[18]);
    }

    const unsigned target = call * (unsigned)NSEND;
    const unsigned stg = STG_OFF + (call & 1u) * SLOT_BYTES;
    const __amdgpu_buffer_rsrc_t rws = mk_rsrc(myws, WS_NEED);
    const unsigned abase = stg + (unsigned)mt * STAGE_A, bbase = stg + A_BYTES + (unsigned)nt * STAGE_B;
    const int wm = wid >> 1, wn = wid & 1;
    const int lvoff = wid * 1024 + lane * 16;
    const unsigned* const myflag = (const unsigned*)(myws + FLG_OFF + (lane >> 3) * 128 + (orel == 0 ? 8 : (lane & 7)) * 4);
    unsigned* const tmo = (unsigned*)(myws + TMO_OFF);
    f4 acc[4][4];
#pragma unroll
    for (int m = 0; m < 4; ++m)
#pragma unroll
        for (int n = 0; n < 4; ++n) acc[m][n] = (f4){0.f, 0.f, 0.f, 0.f};

    auto stage = [&](int kt, int buf) {
        const unsigned ao = abase + (unsigned)kt * (16u * STAGE_A), bo = bbase + (unsigned)kt * (16u * STAGE_B);
        LAS char* const l = (LAS char*)smem + buf * STAGE + wid * 1024;
#pragma unroll
        for (int i = 0; i < 4; ++i) __builtin_amdgcn_raw_ptr_buffer_load_lds(rws, l + i * 8192, 16, lvoff, ao + i * 8192, 0, AUX_LD);
#pragma unroll
        for (int i = 0; i < 2; ++i) __builtin_amdgcn_raw_ptr_buffer_load_lds(rws, l + STAGE_A + i * 8192, 16, lvoff, bo + i * 8192, 0, AUX_LD);
    };
    auto compute = [&](int buf) {
        const LAS char* const la = (const LAS char*)smem + buf * STAGE + wm * 8192 + lane * 16;
        const LAS char* const lb = (const LAS char*)smem + buf * STAGE + STAGE_A + wn * 8192 + lane * 16;
#pragma unroll
        for (int ks = 0; ks < 2; ++ks) {
            h8 af[4], bf[4];
#pragma unroll
            for (int n = 0; n < 4; ++n) bf[n] = *(const LAS h8*)(lb + (ks * 4 + n) * 1024);
#pragma unroll
            for (int m = 0; m < 4; ++m) af[m] = *(const LAS h8*)(la + (m * 2 + ks) * 1024);
#pragma unroll
            for (int m = 0; m < 4; ++m)
#pragma unroll
                for (int n = 0; n < 4; ++n) acc[m][n] = __builtin_amdgcn_mfma_f32_16x16x32_f16(af[m], bf[n], acc[m][n], 0, 0, 0);
        }
    };

    unsigned long long done = 0;
    int buf = 0;
    bool bad = false;
    int nbatch = 0;
    while (done != ~0ull) {
        if (wid == 0) {
            const unsigned long long t0 = __builtin_amdgcn_s_memrealtime();
            unsigned long long m;
            for (;;) {
                const unsigned v = __hip_atomic_load(myflag, __ATOMIC_RELAXED, __HIP_MEMORY_SCOPE_SYSTEM);
                m = __ballot((int)(v - target) >= 0) & ~done;
                if (m) break;
                if (__builtin_amdgcn_readfirstlane(__hip_atomic_load(tmo, __ATOMIC_RELAXED, __HIP_MEMORY_SCOPE_AGENT)) == call
                    || __builtin_amdgcn_s_memrealtime() - t0 > DEADLINE) {
                    if (lane == 0) __hip_atomic_store(tmo, call, __ATOMIC_RELAXED, __HIP_MEMORY_SCOPE_AGENT);
                    break;
                }
                __builtin_amdgcn_s_sleep(8);
            }
            if (lane == 0) { ctl[16] = (unsigned)m; ctl[17] = (unsigned)(m >> 32); }
        }
        BAR();
        unsigned long long avail = ((unsigned long long)__builtin_amdgcn_readfirstlane(ctl[17]) << 32) | (unsigned)__builtin_amdgcn_readfirstlane(ctl[16]);
        if (avail == 0) { bad = true; break; }
        done |= avail;
        int left = __builtin_popcountll(avail);
        int b1 = buf + 1 == NSTAGE ? 0 : buf + 1;
        stage(__builtin_ctzll(avail), buf); avail &= avail - 1;
        if (avail) { stage(__builtin_ctzll(avail), b1); avail &= avail - 1; }
        for (; left > 0; --left) {
            if (left > 1) WAIT_V(6); else WAIT_V(0);
            BAR();
            b1 = buf + 1 == NSTAGE ? 0 : buf + 1;
            const int b2 = b1 + 1 == NSTAGE ? 0 : b1 + 1;
            if (avail) { stage(__builtin_ctzll(avail), b2); avail &= avail - 1; }
            compute(buf);
            buf = b1;
        }
    }
    STAMP(10);

    {
        if (!bad) bad = __builtin_amdgcn_readfirstlane(__hip_atomic_load(tmo, __ATOMIC_RELAXED, __HIP_MEMORY_SCOPE_AGENT)) == call;
        const __amdgpu_buffer_rsrc_t ro = mk_rsrc(p.out[owner], OUT_BYTES);
        const unsigned row0 = (unsigned)((xcd & 1) * 256 + wm * 64), col0 = (unsigned)(rank * NS + nt * 128 + wn * 64);
        const unsigned so = (row0 * NG + col0) * 4u;
        const int vo = ((lane >> 4) * 4 * NG + (lane & 15) * 4) * 4;
        const float qn = __builtin_nanf("");
#pragma unroll
        for (int m = 0; m < 4; ++m)
#pragma unroll
            for (int r = 0; r < 4; ++r) {
                f4 v = {acc[m][0][r], acc[m][1][r], acc[m][2][r], acc[m][3][r]};
                if (bad) v = (f4){qn, qn, qn, qn};
                st16(__builtin_bit_cast(u4, v), ro, so + vo + (unsigned)(m * 16 + r) * (NG * 4u));
            }
        WAIT_V(0);
        STAMP(11);
        if (orel != 0) {
            BAR();
            if (tid == 0) __hip_atomic_fetch_add((unsigned*)(p.ws[owner] + DONE_OFF + rank * 128), 1u, __ATOMIC_RELAXED, __HIP_MEMORY_SCOPE_SYSTEM);
        } else if (b == 0 && wid == 0) {
            const unsigned* f = (const unsigned*)(myws + DONE_OFF + (lane & 7) * 128);
            const unsigned long long t0 = __builtin_amdgcn_s_memrealtime();
            for (;;) {
                const unsigned v = __hip_atomic_load(f, __ATOMIC_RELAXED, __HIP_MEMORY_SCOPE_SYSTEM);
                if (__all((lane & 7) == rank || (int)(v - target) >= 0)) break;
                if (__builtin_amdgcn_readfirstlane(__hip_atomic_load(tmo, __ATOMIC_RELAXED, __HIP_MEMORY_SCOPE_AGENT)) == call
                    || __builtin_amdgcn_s_memrealtime() - t0 > DEADLINE) {
                    if (lane == 0) {
                        __hip_atomic_store(tmo, call, __ATOMIC_RELAXED, __HIP_MEMORY_SCOPE_AGENT);
                        p.out[rank][(size_t)rank * NS] = qn;
                    }
                    break;
                }
                __builtin_amdgcn_s_sleep(2);
            }
            __builtin_amdgcn_fence(__ATOMIC_ACQUIRE, "");
            STAMP(12);
        }
    }
}

}

extern "C" void kernel_launch(int rank, int nranks, ncclComm_t comm,
                              hipStream_t stream,
                              void* const* d_in, const int* in_sizes, int n_in,
                              void* d_out, int out_size,
                              const int* in_dtypes, int out_dtype,
                              void* const* d_out_all,
                              void* const* d_ws_all, size_t ws_bytes) {
    (void)comm; (void)in_dtypes; (void)out_dtype; (void)d_out;
    if (nranks != NR || (unsigned)rank >= (unsigned)NR || n_in < 2 || in_sizes[0] != MG * KS || in_sizes[1] != KS * NG
        || out_size != MS * NG || ws_bytes < WS_NEED) {
        fprintf(stderr, "kernel_launch: unexpected problem shape\n");
        return;
    }
    Params p;
    p.x = (const float*)d_in[0]; p.w = (const float*)d_in[1]; p.rank = rank;
    for (int i = 0; i < NR; ++i) { p.ws[i] = (char*)d_ws_all[i]; p.out[i] = (float*)d_out_all[i]; }
    hipLaunchKernelGGL(gemm_rs_kernel, dim3(NWG), dim3(512), 0, stream, p);
    HIP_CHECK(hipGetLastError());
}
```

```cpp
#include <hip/hip_runtime.h>
#include <rccl/rccl.h>
#include <cstdio>

#define HIP_CHECK(expr) do { hipError_t _e = (expr); if (_e != hipSuccess) { \
    fprintf(stderr, "HIP error %s at %s:%d\n", hipGetErrorString(_e), __FILE__, __LINE__); \
    return; } } while (0)

namespace {

typedef unsigned u4 __attribute__((ext_vector_type(4)));
typedef float f4 __attribute__((ext_vector_type(4)));
typedef _Float16 h8 __attribute__((ext_vector_type(8)));
#define LAS __attribute__((address_space(3)))
#define GAS __attribute__((address_space(1)))

constexpr int NR = 8;
constexpr int MG = 4096, NG = 16384, KS = 512, NS = 2048, MS = 512;
constexpr int NWG = 256, NSEND = 32;
constexpr int STAGE_A = 32768, STAGE_B = 16384, STAGE = STAGE_A + STAGE_B, NSTAGE = 3;
constexpr int CTL = NSTAGE * STAGE;
constexpr int LDS_BYTES = CTL + 128;
constexpr unsigned FLG_OFF = 0;
constexpr unsigned TMO_OFF = 2048;
constexpr unsigned ORG_OFF = 2304;
constexpr unsigned SEQ_OFF = 4096;
constexpr unsigned PFL_OFF = 65536;
constexpr unsigned STAMP_OFF = 131072;
constexpr unsigned W2_OFF = 1u << 20, W2_BYTES = 16u << 20;
constexpr unsigned STG_OFF = W2_OFF + W2_BYTES;
constexpr unsigned A_BYTES = 32u << 20;
constexpr unsigned B_BYTES = 16u << 20;
constexpr unsigned P_BYTES = 32u << 20;
constexpr unsigned SLOT_BYTES = A_BYTES + B_BYTES + P_BYTES;
constexpr unsigned WS_NEED = STG_OFF + 2 * SLOT_BYTES;
constexpr unsigned KT_A = 16u * STAGE_A, KT_B = 16u * STAGE_B;
constexpr unsigned OUT_BYTES = (unsigned)MS * NG * 4;
constexpr unsigned long long DEADLINE = 100000000ull;
constexpr int AUX_SYS = 17;
constexpr int AUX_LD = 16;

struct Params {
    const float* x; const float* w;
    char* ws[NR]; float* out;
    int rank;
};

#define WAIT_V(n) asm volatile("s_waitcnt vmcnt(" #n ")" ::: "memory")
#define WAIT_VN(n) asm volatile("s_waitcnt vmcnt(%0)" :: "i"(n) : "memory")
#define GBS 59.0
#define T_START 250
#define CONV_TICKS 256
#define LD_SLEEP 16
#define SWIN 3
#define EWIN 2
constexpr unsigned TAU_Q = (unsigned)(32768.0 / GBS / 10.0 * 256.0);
#define BAR() do { asm volatile("s_waitcnt lgkmcnt(0)" ::: "memory"); __builtin_amdgcn_s_barrier(); asm volatile("" ::: "memory"); } while (0)

#define STAMPW(w, i) do {} while (0)
#define STAMP(i) STAMPW(0, i)

__device__ __forceinline__ __amdgpu_buffer_rsrc_t mk_rsrc(const void* p, unsigned bytes) {
    return __builtin_amdgcn_make_buffer_rsrc((void*)p, 0, (int)bytes, 0x00020000);
}
__device__ __forceinline__ void st16(u4 v, __amdgpu_buffer_rsrc_t r, unsigned off) {
    __builtin_amdgcn_raw_buffer_store_b128(v, r, (int)off, 0, AUX_SYS);
}
__device__ __forceinline__ unsigned now32() { return (unsigned)__builtin_amdgcn_s_memrealtime(); }
__device__ __forceinline__ void pace(unsigned t0, unsigned n) {
    const unsigned due = t0 + ((n * TAU_Q) >> 8);
    while ((int)(now32() - due) < 0) __builtin_amdgcn_s_sleep(1);
    asm volatile("" ::: "memory");
}
__device__ __forceinline__ u4 cvt8(f4 a, f4 b) {
    h8 h = {(_Float16)a[0], (_Float16)a[1], (_Float16)a[2], (_Float16)a[3], (_Float16)b[0], (_Float16)b[1], (_Float16)b[2], (_Float16)b[3]};
    return __builtin_bit_cast(u4, h);
}
__device__ __forceinline__ bool give_up(unsigned* tmo, unsigned call, unsigned long long t0, int lane) {
    if (__builtin_amdgcn_readfirstlane(__hip_atomic_load(tmo, __ATOMIC_RELAXED, __HIP_MEMORY_SCOPE_AGENT)) == call
        || __builtin_amdgcn_s_memrealtime() - t0 > DEADLINE) {
        if (lane == 0) __hip_atomic_store(tmo, call, __ATOMIC_RELAXED, __HIP_MEMORY_SCOPE_AGENT);
        return true;
    }
    return false;
}

__global__ __launch_bounds__(512, 2) void gemm_rs_kernel(const Params p) {
    __shared__ __attribute__((aligned(16))) char smem[LDS_BYTES];
    const int tid = threadIdx.x, lane = tid & 63;
    const int wid = __builtin_amdgcn_readfirstlane(tid >> 6);
    const int b = blockIdx.x, rank = p.rank;
    char* const myws = p.ws[rank];
    LAS unsigned* const ctl = (LAS unsigned*)((LAS char*)smem + CTL);
    STAMP(0);
    const unsigned tk0 = now32() + T_START;

    if (tid == 511) ctl[18] = __hip_atomic_fetch_add((unsigned*)(myws + SEQ_OFF + b * 128), 1u, __ATOMIC_RELAXED, __HIP_MEMORY_SCOPE_AGENT) + 1u;
    if (tid < 16) ctl[tid] = 0;

    const int xcd = b & 7, slot = b >> 3;
    const int orel = slot >> 2;
    const int owner = (rank + orel) & 7;
    const int hh = xcd & 1, nt = (xcd >> 1) * 4 + (slot & 3);
    const __amdgpu_buffer_rsrc_t rws = mk_rsrc(myws, WS_NEED);
    const int voff = lane * 16;

    const bool sender = b < NSEND;
    f4 xr[4], wr[16];
    u4 xh[2], wh[8];
    auto xsrc = [&](int j, int sb) { return p.x + (size_t)((j * 32 + sb) * 16 + (lane & 15)) * KS + (lane >> 4) * 8; };
    auto wsrc = [&](int j, int sb) { return p.w + (size_t)((lane >> 4) * 8) * NG + j * NS + sb * 64 + (lane & 15) * 4; };
    auto load_x = [&](const float* xs, int c) {
#pragma unroll
        for (int i = 0; i < 4; ++i) xr[i] = *(const f4*)(xs + c * 64 + (i >> 1) * 32 + (i & 1) * 4);
    };
    auto load_w1 = [&](const float* ws, int c, int i) { wr[i] = *(const f4*)(ws + (size_t)(c * 64 + (i >> 3) * 32 + (i & 7)) * NG); };
    auto cvt_x = [&]() {
#pragma unroll
        for (int ks = 0; ks < 2; ++ks) xh[ks] = cvt8(xr[ks * 2], xr[ks * 2 + 1]);
    };
    auto cvt_w = [&]() {
#pragma unroll
        for (int ks = 0; ks < 2; ++ks)
#pragma unroll
            for (int jj = 0; jj < 4; ++jj) {
                const f4 lo = {wr[ks * 8][jj], wr[ks * 8 + 1][jj], wr[ks * 8 + 2][jj], wr[ks * 8 + 3][jj]};
                const f4 hi = {wr[ks * 8 + 4][jj], wr[ks * 8 + 5][jj], wr[ks * 8 + 6][jj], wr[ks * 8 + 7][jj]};
                wh[ks * 4 + jj] = cvt8(lo, hi);
            }
    };
    unsigned call;
    unsigned long long* const torg = (unsigned long long*)(myws + ORG_OFF);

    if (!sender) {
        BAR();
        call = __builtin_amdgcn_readfirstlane(ctl[18]);
        const unsigned stg = STG_OFF + (call & 1u) * SLOT_BYTES;
        for (int u = b - NSEND; u < 256; u += NWG - NSEND) {
            const int c = u >> 5, j = (u >> 2) & 7, sb = (u & 3) * 8 + wid;
            while ((int)(now32() - tk0 - (unsigned)(b - NSEND) * CONV_TICKS / 16u) < 0) __builtin_amdgcn_s_sleep(2);
            const float* ws = wsrc(j, sb);
#pragma unroll
            for (int i = 0; i < 16; ++i) load_w1(ws, c, i);
            load_x(xsrc(j, sb), c);
            cvt_w(); cvt_x();
            const unsigned wo = (j == rank ? stg + A_BYTES : W2_OFF) + (unsigned)(j * 8 + c) * KT_B + (unsigned)sb * 8192u + voff;
            const unsigned xo = stg + (unsigned)(rank * 8 + c) * KT_A + (unsigned)(j * 32 + sb) * 2048u + voff;
#pragma unroll
            for (int q = 0; q < 8; ++q) st16(wh[q], rws, wo + q * 1024);
            st16(xh[0], rws, xo); st16(xh[1], rws, xo + 1024);
            WAIT_V(0);
            BAR();
            if (tid == 0) __hip_atomic_fetch_add((unsigned*)(myws + FLG_OFF + rank * 128) + c, 1u, __ATOMIC_RELAXED, __HIP_MEMORY_SCOPE_SYSTEM);
        }
        STAMP(1);
    } else {
        LAS char* const ring = (LAS char*)smem + voff;
        if (wid < 7) {
            const int j = (rank + 1 + wid) & 7;
            const float* xs = xsrc(j, b);
            const float* ws = wsrc(j, b);
            load_x(xs, 0);
            asm volatile("" ::: "memory");
#pragma unroll
            for (int i = 0; i < 16; ++i) load_w1(ws, 0, i);
            BAR();
            auto publish = [&](unsigned v) {
                asm volatile("s_waitcnt lgkmcnt(0)" ::: "memory");
                if (lane == 0) __hip_atomic_store(ctl + wid, v, __ATOMIC_RELAXED, __HIP_MEMORY_SCOPE_WORKGROUP);
            };
            auto wait_free = [&](int k) {
                while ((int)__hip_atomic_load(ctl + 8, __ATOMIC_RELAXED, __HIP_MEMORY_SCOPE_WORKGROUP) < k - 1) __builtin_amdgcn_s_sleep(4);
                asm volatile("" ::: "memory");
            };
            for (int k = 0; k < 8; ++k) {
                LAS char* const r = ring + ((k & 1) * 7 + wid) * 10240;
                wait_free(k);
                cvt_x();
                *(LAS u4*)(r) = xh[0]; *(LAS u4*)(r + 1024) = xh[1];
                publish(2 * k + 1);
                cvt_w();
#pragma unroll
                for (int q = 0; q < 8; ++q) *(LAS u4*)(r + 2048 + q * 1024) = wh[q];
                publish(2 * k + 2);
                if (k < 7) {
                    load_x(xs, k + 1);
#pragma unroll
                    for (int i = 0; i < 16; ++i) { if ((i & 1) == 0) __builtin_amdgcn_s_sleep(LD_SLEEP); asm volatile("" ::: "memory"); load_w1(ws, k + 1, i); }
                }
            }
            for (int k = 8; k < 10; ++k) {
                LAS char* const r = ring + ((k & 1) * 7 + wid) * 10240;
#pragma unroll
                for (int i = 0; i < 16; ++i) { if ((i & 1) == 0) __builtin_amdgcn_s_sleep(LD_SLEEP); asm volatile("" ::: "memory"); wr[i] = *(const f4*)(xs + (k - 8) * 256 + (i >> 1) * 32 + (i & 1) * 4); }
                wait_free(k);
#pragma unroll
                for (int q = 0; q < 8; ++q) *(LAS u4*)(r + q * 1024) = cvt8(wr[2 * q], wr[2 * q + 1]);
                publish(k + 9);
            }
            call = __builtin_amdgcn_readfirstlane(ctl[18]);
        } else {
            unsigned* flg = (unsigned*)(p.ws[lane & 7] + FLG_OFF + rank * 128);
            asm volatile("" : "+v"(flg));
            __amdgpu_buffer_rsrc_t rd[7];
            unsigned xg[7];
#pragma unroll
            for (int a = 0; a < 7; ++a) { const int j = (rank + 1 + a) & 7; rd[a] = mk_rsrc(p.ws[j], WS_NEED); xg[a] = (unsigned)j * 65536u; }
            BAR();
            call = __builtin_amdgcn_readfirstlane(ctl[18]);
            const unsigned stg = STG_OFF + (call & 1u) * SLOT_BYTES;
            const unsigned xoff = stg + (unsigned)(rank * 8) * KT_A + (unsigned)b * 2048u + voff;
            const unsigned woff = stg + A_BYTES + (unsigned)(rank * 8) * KT_B + (unsigned)b * 8192u + voff;
            auto wait_full = [&](unsigned v) {
                while (!__all(lane >= 7 || __hip_atomic_load(ctl + (lane & 7), __ATOMIC_RELAXED, __HIP_MEMORY_SCOPE_WORKGROUP) >= v)) __builtin_amdgcn_s_sleep(1);
                asm volatile("" ::: "memory");
            };
            auto signal = [&](int c) {
                if (lane < NR && lane != rank) __hip_atomic_fetch_add((GAS unsigned*)flg + c, 1u, __ATOMIC_RELAXED, __HIP_MEMORY_SCOPE_SYSTEM);
            };
            auto release = [&](int k) {
                asm volatile("s_waitcnt lgkmcnt(0)" ::: "memory");
                if (lane == 0) __hip_atomic_store(ctl + 8, (unsigned)(k + 1), __ATOMIC_RELAXED, __HIP_MEMORY_SCOPE_WORKGROUP);
            };
            wait_full(1);
            const unsigned long long t64 = __builtin_amdgcn_s_memrealtime();
            const unsigned ts = (unsigned)t64;
            if (lane == 0) __hip_atomic_fetch_max(torg, t64, __ATOMIC_RELAXED, __HIP_MEMORY_SCOPE_AGENT);
            for (int k = 0; k < 8; ++k) {
                const LAS char* const r = ring + (k & 1) * 71680;
                const unsigned xo = xoff + k * KT_A, wo = woff + k * KT_B;
#pragma unroll
                for (int i = 0; i < 20; ++i) {
                    if (i == 0) wait_full(2 * k + 1);
                    if (i == 12) wait_full(2 * k + 2);
                    u4 d[7];
#pragma unroll
                    for (int a = 0; a < 7; ++a) {
                        const int u = i < 12 ? (a + 6 - i / 2) % 7 : a;
                        d[a] = *(const LAS u4*)(r + u * 10240 + (i < 12 ? (i & 1) : i - 10) * 1024);
                    }
                    if (i == 19) release(k);
                    pace(ts, k * 20 + i);
#pragma unroll
                    for (int a = 0; a < 7; ++a) {
                        const int u = i < 12 ? (a + 6 - i / 2) % 7 : a;
                        st16(d[a], rd[a], i < 12 ? xo + xg[u] + (i & 1) * 1024 : wo + (i - 12) * 1024);
                    }
                    WAIT_VN(7 * (SWIN - 1));
                    if (i == SWIN - 2 && k > 0) { signal(k - 1); STAMPW(7, k); }
                }
            }
            for (int k = 8; k < 10; ++k) {
                const LAS char* const r = ring + (k & 1) * 71680;
                wait_full(k + 9);
#pragma unroll
                for (int i = 0; i < 8; ++i) {
                    u4 d[7];
#pragma unroll
                    for (int a = 0; a < 7; ++a) d[a] = *(const LAS u4*)(r + a * 10240 + i * 1024);
                    pace(ts, 160 + (k - 8) * 8 + i);
#pragma unroll
                    for (int a = 0; a < 7; ++a) st16(d[a], rd[a], xoff + xg[a] + (unsigned)((k - 8) * 4 + (i >> 1)) * KT_A + (i & 1) * 1024);
                    WAIT_VN(7);
                    if (i == 0 && k == 8) { signal(7); STAMPW(7, 8); }
                }
            }
            WAIT_V(0);
            signal(8);
            STAMPW(7, 9);
        }
        BAR();
    }
    const unsigned stg = STG_OFF + (call & 1u) * SLOT_BYTES;

    const unsigned abase = stg + (unsigned)(owner * 2 + hh) * STAGE_A, bbase = stg + A_BYTES + (unsigned)nt * STAGE_B;
    const unsigned abase2 = stg + (unsigned)(rank * 2 + hh) * STAGE_A + (unsigned)((rank - owner) * 8) * KT_A;
    const unsigned bbase2 = W2_OFF + (unsigned)nt * STAGE_B;
    const int osrc = orel == 0 ? -1 : owner;
    const int wm = wid >> 1, wn = wid & 1;
    const int lvoff = wid * 1024 + voff;
    const unsigned* const myflag = (const unsigned*)(myws + FLG_OFF + ((lane >> 3) == osrc ? rank : (lane >> 3)) * 128 + (orel == 0 && (lane >> 3) != rank ? 8 : (lane & 7)) * 4);
    const unsigned target = call * 32u;
    unsigned* const tmo = (unsigned*)(myws + TMO_OFF);
    f4 acc[4][4], acc2[4][4];
#pragma unroll
    for (int m = 0; m < 4; ++m)
#pragma unroll
        for (int n = 0; n < 4; ++n) { acc[m][n] = (f4){0.f, 0.f, 0.f, 0.f}; acc2[m][n] = (f4){0.f, 0.f, 0.f, 0.f}; }

    auto stage = [&](int kt, int buf, unsigned ab, unsigned bb) {
        const unsigned ao = ab + (unsigned)kt * KT_A, bo = bb + (unsigned)kt * KT_B;
        LAS char* const l = (LAS char*)smem + buf * STAGE + wid * 1024;
#pragma unroll
        for (int i = 0; i < 4; ++i) __builtin_amdgcn_raw_ptr_buffer_load_lds(rws, l + i * 8192, 16, lvoff, ao + i * 8192, 0, AUX_LD);
#pragma unroll
        for (int i = 0; i < 2; ++i) __builtin_amdgcn_raw_ptr_buffer_load_lds(rws, l + STAGE_A + i * 8192, 16, lvoff, bo + i * 8192, 0, AUX_LD);
    };
    auto compute = [&](f4 (&c)[4][4], int buf) {
        const LAS char* const la = (const LAS char*)smem + buf * STAGE + wm * 8192 + voff;
        const LAS char* const lb = (const LAS char*)smem + buf * STAGE + STAGE_A + wn * 8192 + voff;
        h8 af[2][4], bf[2][4];
#pragma unroll
        for (int ks = 0; ks < 2; ++ks) {
#pragma unroll
            for (int n = 0; n < 4; ++n) bf[ks][n] = *(const LAS h8*)(lb + (ks * 4 + n) * 1024);
#pragma unroll
            for (int m = 0; m < 4; ++m) af[ks][m] = *(const LAS h8*)(la + (m * 2 + ks) * 1024);
        }
        __builtin_amdgcn_sched_barrier(0);
#pragma unroll
        for (int ks = 0; ks < 2; ++ks)
#pragma unroll
            for (int m = 0; m < 4; ++m)
#pragma unroll
                for (int n = 0; n < 4; ++n) c[m][n] = __builtin_amdgcn_mfma_f32_16x16x32_f16(af[ks][m], bf[ks][n], c[m][n], 0, 0, 0);
    };

    int buf = 0;
    auto run = [&](f4 (&c)[4][4], unsigned long long avail, unsigned ab, unsigned bb) {
        int left = __builtin_popcountll(avail);
        int b1 = buf + 1 == NSTAGE ? 0 : buf + 1;
        stage(__builtin_ctzll(avail), buf, ab, bb); avail &= avail - 1;
        if (avail) { stage(__builtin_ctzll(avail), b1, ab, bb); avail &= avail - 1; }
        do {
            if (left > 1) WAIT_V(6); else WAIT_V(0);
            BAR();
            b1 = buf + 1 == NSTAGE ? 0 : buf + 1;
            const int b2 = b1 + 1 == NSTAGE ? 0 : b1 + 1;
            if (avail) { stage(__builtin_ctzll(avail), b2, ab, bb); avail &= avail - 1; }
            compute(c, buf);
            buf = b1;
        } while (--left > 0);
    };

    const unsigned long long locm = orel == 0 ? 0ull : 0xffull << (owner * 8);
    unsigned long long done = 0;
    bool bad = false;
    int nbatch = 0;
    while (done != ~0ull) {
        if (wid == 0) {
            const unsigned long long t0 = __builtin_amdgcn_s_memrealtime();
            unsigned long long m;
            for (;;) {
                const unsigned v = __hip_atomic_load(myflag, __ATOMIC_RELAXED, __HIP_MEMORY_SCOPE_SYSTEM);
                m = __ballot((int)(v - target) >= 0) & ~done;
                if (m || give_up(tmo, call, t0, lane)) break;
                __builtin_amdgcn_s_sleep(8);
            }
            if (lane == 0) { ctl[16] = (unsigned)m; ctl[17] = (unsigned)(m >> 32); }
        }
        BAR();
        const unsigned long long avail = ((unsigned long long)__builtin_amdgcn_readfirstlane(ctl[17]) << 32) | (unsigned)__builtin_amdgcn_readfirstlane(ctl[16]);
        if (avail == 0) { bad = true; break; }
        done |= avail;
        if (avail & locm) run(acc2, avail & locm, abase2, bbase2);
        if (avail & ~locm) run(acc, avail & ~locm, abase, bbase);
    }
    STAMP(10);

    const float qn = __builtin_nanf("");
    if (orel != 0) {
        const unsigned tile = (unsigned)(hh * 16 + nt);
        {
            const __amdgpu_buffer_rsrc_t ro = mk_rsrc(p.ws[owner], WS_NEED);
            const unsigned po = stg + A_BYTES + B_BYTES + (((unsigned)rank * 32u + tile) * 8u + (unsigned)wid) * 16384u + voff;
            unsigned te = (unsigned)__builtin_amdgcn_readfirstlane((unsigned)__hip_atomic_load(torg, __ATOMIC_RELAXED, __HIP_MEMORY_SCOPE_AGENT)) + ((176u * TAU_Q) >> 8);
            { const unsigned now = now32(); if ((int)(now - te) > 0) te = now; }
            if (bad) {
#pragma unroll
                for (int q = 0; q < 16; ++q) acc[q >> 2][q & 3] = (f4){qn, qn, qn, qn};
            }
            pace(te, wid * 16);
            while (__hip_atomic_load(ctl + 12, __ATOMIC_RELAXED, __HIP_MEMORY_SCOPE_WORKGROUP) < (unsigned)wid) __builtin_amdgcn_s_sleep(1);
#pragma unroll
            for (int q = 0; q < 16; ++q) {
                pace(te, wid * 16 + q);
                st16(__builtin_bit_cast(u4, acc[q >> 2][q & 3]), ro, po + q * 1024);
                WAIT_VN(EWIN - 1);
            }
            if (lane == 0) __hip_atomic_fetch_add(ctl + 12, 1u, __ATOMIC_RELAXED, __HIP_MEMORY_SCOPE_WORKGROUP);
            WAIT_V(0);
            if (lane == 0) __hip_atomic_store((unsigned*)(p.ws[owner] + PFL_OFF + ((unsigned)rank * 32u + tile) * 128u) + wid, call, __ATOMIC_RELAXED, __HIP_MEMORY_SCOPE_SYSTEM);
        }
        STAMP(11); STAMPW(7, 12);
        {
            const unsigned* f = (const unsigned*)(myws + PFL_OFF + ((unsigned)owner * 32u + tile) * 128u) + wid;
            const unsigned long long t0 = __builtin_amdgcn_s_memrealtime();
            for (;;) {
                const unsigned v = __builtin_amdgcn_readfirstlane(__hip_atomic_load(f, __ATOMIC_RELAXED, __HIP_MEMORY_SCOPE_SYSTEM));
                if ((int)(v - call) >= 0) break;
                if (give_up(tmo, call, t0, lane)) { bad = true; break; }
                __builtin_amdgcn_s_sleep(2);
            }
            asm volatile("" ::: "memory");
            STAMPW(7, 13);
            const unsigned pi = stg + A_BYTES + B_BYTES + (((unsigned)owner * 32u + tile) * 8u + (unsigned)wid) * 16384u + voff;
#pragma unroll
            for (int q = 0; q < 16; ++q) acc[q >> 2][q & 3] = __builtin_bit_cast(f4, __builtin_amdgcn_raw_buffer_load_b128(rws, pi + q * 1024, 0, AUX_LD));
#pragma unroll
            for (int q = 0; q < 16; ++q) acc[q >> 2][q & 3] += acc2[q >> 2][q & 3];
        }
    }
    {
        if (!bad) bad = __builtin_amdgcn_readfirstlane(__hip_atomic_load(tmo, __ATOMIC_RELAXED, __HIP_MEMORY_SCOPE_AGENT)) == call;
        const __amdgpu_buffer_rsrc_t ro = mk_rsrc(p.out, OUT_BYTES);
        const unsigned row0 = (unsigned)(hh * 256 + wm * 64 + (lane >> 4) * 4), col0 = (unsigned)(owner * NS + nt * 128 + wn * 64 + (lane & 15) * 4);
        const unsigned oo = (row0 * NG + col0) * 4u;
#pragma unroll
        for (int m = 0; m < 4; ++m)
#pragma unroll
            for (int r = 0; r < 4; ++r) {
                f4 v = {acc[m][0][r], acc[m][1][r], acc[m][2][r], acc[m][3][r]};
                if (bad) v = (f4){qn, qn, qn, qn};
                st16(__builtin_bit_cast(u4, v), ro, oo + (unsigned)(m * 16 + r) * (NG * 4u));
            }
    }
}

}

extern "C" void kernel_launch(int rank, int nranks, ncclComm_t comm,
                              hipStream_t stream,
                              void* const* d_in, const int* in_sizes, int n_in,
                              void* d_out, int out_size,
                              const int* in_dtypes, int out_dtype,
                              void* const* d_out_all,
                              void* const* d_ws_all, size_t ws_bytes) {
    (void)comm; (void)in_dtypes; (void)out_dtype; (void)d_out_all;
    if (nranks != NR || (unsigned)rank >= (unsigned)NR || n_in < 2 || in_sizes[0] != MG * KS || in_sizes[1] != KS * NG
        || out_size != MS * NG || ws_bytes < WS_NEED) {
        fprintf(stderr, "kernel_launch: unexpected problem shape\n");
        return;
    }
    Params p;
    p.x = (const float*)d_in[0]; p.w = (const float*)d_in[1]; p.out = (float*)d_out; p.rank = rank;
    for (int i = 0; i < NR; ++i) p.ws[i] = (char*)d_ws_all[i];
    hipLaunchKernelGGL(gemm_rs_kernel, dim3(NWG), dim3(512), 0, stream, p);
    HIP_CHECK(hipGetLastError());
}
```

```cpp
#include <hip/hip_runtime.h>
#include <rccl/rccl.h>
#include <cstdio>

#define HIP_CHECK(expr) do { hipError_t _e = (expr); if (_e != hipSuccess) { \
    fprintf(stderr, "HIP error %s at %s:%d\n", hipGetErrorString(_e), __FILE__, __LINE__); \
    return; } } while (0)

namespace {

typedef unsigned u4 __attribute__((ext_vector_type(4)));
typedef float f4 __attribute__((ext_vector_type(4)));
typedef _Float16 h8 __attribute__((ext_vector_type(8)));
#define LAS __attribute__((address_space(3)))
#define GAS __attribute__((address_space(1)))

constexpr int NR = 8;
constexpr int MG = 4096, NG = 16384, KS = 512, NS = 2048, MS = 512;
constexpr int NWG = 256, NOWN = 32;
constexpr int STAGE_A = 32768, STAGE_B = 16384, STAGE = STAGE_A + STAGE_B, NSTAGE = 3;
constexpr int CTL = NSTAGE * STAGE;
constexpr int LDS_BYTES = CTL + 128;
constexpr unsigned TMO_OFF = 0;
constexpr unsigned ORG_OFF = 256;
constexpr unsigned AGG_OFF = 512;
constexpr unsigned SEQ_OFF = 4096;
constexpr unsigned PFL_OFF = 65536;
constexpr unsigned FLG_OFF = 98304, FLG_REP = 4352;
constexpr unsigned STAMP_OFF = 262144;
constexpr unsigned W2_OFF = 1u << 20, W2_BYTES = 16u << 20;
constexpr unsigned STG_OFF = W2_OFF + W2_BYTES;
constexpr unsigned A_BYTES = 32u << 20;
constexpr unsigned B_BYTES = 16u << 20;
constexpr unsigned P_BYTES = 32u << 20;
constexpr unsigned B_REL = A_BYTES, P_REL = B_REL + B_BYTES, SLOT_BYTES = P_REL + P_BYTES;
constexpr unsigned WS_NEED = STG_OFF + 2 * SLOT_BYTES;
constexpr unsigned KT_A = 16u * STAGE_A, KT_B = 16u * STAGE_B;
constexpr unsigned OOB = 0x80000000u;
constexpr unsigned OUT_BYTES = (unsigned)MS * NG * 4;
constexpr unsigned long long DEADLINE = 100000000ull;
constexpr int AUX_SYS = 17;
constexpr int AUX_LD = 16;

struct Params {
    const float* x; const float* w;
    char* ws[NR]; float* out;
    int rank;
};

#define WAIT_V(n) asm volatile("s_waitcnt vmcnt(" #n ")" ::: "memory")
#define WAIT_VN(n) asm volatile("s_waitcnt vmcnt(%0)" :: "i"(n) : "memory")
#define GBS 58.0
#define GBS_OUT 61.0
#define T_START 150
#define CONV_TICKS 192
#define SWIN 2
#define POLL_SCOPE __HIP_MEMORY_SCOPE_SYSTEM
#define POLL_SLEEP 4
#define FLAG_LAT 90
#define LEAD 12
#define TILE_TICKS 100
constexpr unsigned TAU_ST = (unsigned)(32768.0 / GBS / 10.0 * 256.0), TAU_LD = (unsigned)(32768.0 / GBS_OUT / 10.0 * 256.0);
#define BAR() do { asm volatile("" ::: "memory"); __builtin_amdgcn_s_waitcnt(0xC07F);   __builtin_amdgcn_s_barrier(); asm volatile("" ::: "memory"); } while (0)

#define STAMPW(w, i) do {} while (0)
#define STAMP(i) STAMPW(0, i)

__device__ __forceinline__ __amdgpu_buffer_rsrc_t mk_rsrc(const void* p, unsigned bytes) {
    return __builtin_amdgcn_make_buffer_rsrc((void*)p, 0, (int)bytes, 0x00020000);
}
__device__ __forceinline__ void st16(u4 v, __amdgpu_buffer_rsrc_t r, unsigned off) {
    __builtin_amdgcn_raw_buffer_store_b128(v, r, (int)off, 0, AUX_SYS);
}
__device__ __forceinline__ u4 ld16(__amdgpu_buffer_rsrc_t r, unsigned off) {
    return __builtin_amdgcn_raw_buffer_load_b128(r, (int)off, 0, AUX_SYS);
}
__device__ __forceinline__ unsigned now32() { return (unsigned)__builtin_amdgcn_s_memrealtime(); }
template <unsigned TAU> __device__ __forceinline__ unsigned slots(unsigned n) { return (n * TAU) >> 8; }
__device__ __forceinline__ void wait_until(unsigned due) {
    while ((int)(now32() - due) < 0) __builtin_amdgcn_s_sleep(1);
    asm volatile("" ::: "memory");
}
template <unsigned TAU> __device__ __forceinline__ void pace(unsigned t0, unsigned n) { wait_until(t0 + slots<TAU>(n)); }
__device__ __forceinline__ unsigned lds_ld(LAS unsigned* p) {
    return __builtin_amdgcn_readfirstlane(__hip_atomic_load(p, __ATOMIC_RELAXED, __HIP_MEMORY_SCOPE_WORKGROUP));
}
__device__ __forceinline__ unsigned lds_inc(LAS unsigned* p, int lane) {
    unsigned old = 0;
    if (lane == 0) old = __hip_atomic_fetch_add(p, 1u, __ATOMIC_RELAXED, __HIP_MEMORY_SCOPE_WORKGROUP);
    return __builtin_amdgcn_readfirstlane(old);
}
__device__ __forceinline__ u4 cvt8(f4 a, f4 b) {
    h8 h = {(_Float16)a[0], (_Float16)a[1], (_Float16)a[2], (_Float16)a[3], (_Float16)b[0], (_Float16)b[1], (_Float16)b[2], (_Float16)b[3]};
    return __builtin_bit_cast(u4, h);
}
__device__ __forceinline__ bool give_up(unsigned* tmo, unsigned call, unsigned long long t0, int lane) {
    if (__builtin_amdgcn_readfirstlane(__hip_atomic_load(tmo, __ATOMIC_RELAXED, __HIP_MEMORY_SCOPE_AGENT)) == call
        || __builtin_amdgcn_s_memrealtime() - t0 > DEADLINE) {
        if (lane == 0) __hip_atomic_store(tmo, call, __ATOMIC_RELAXED, __HIP_MEMORY_SCOPE_AGENT);
        return true;
    }
    return false;
}

__global__ __launch_bounds__(512, 2) void gemm_rs_kernel(const Params p) {
    __shared__ __attribute__((aligned(16))) char smem[LDS_BYTES];
    const int tid = threadIdx.x, lane = tid & 63;
    const int wid = __builtin_amdgcn_readfirstlane(tid >> 6);
    const int b = blockIdx.x, rank = p.rank;
    char* const myws = p.ws[rank];
    LAS unsigned* const ctl = (LAS unsigned*)((LAS char*)smem + CTL);
    STAMP(0);
    const unsigned long long tstart = __builtin_amdgcn_s_memrealtime();

    const int xcd = b & 7, slot = b >> 3;
    const int orel = slot >> 2;
    const int owner = (rank + orel) & 7;
    const int hh = xcd & 1, nt = (xcd >> 1) * 4 + (slot & 3);
    const __amdgpu_buffer_rsrc_t rws = mk_rsrc(myws, WS_NEED);
    const int voff = lane * 16;
    unsigned* const tmo = (unsigned*)(myws + TMO_OFF);
    unsigned long long* const torg = (unsigned long long*)(myws + ORG_OFF);

    const int sb = b & 31;
    const float* const xs = p.x + (size_t)((wid * 32 + sb) * 16 + (lane & 15)) * KS + (lane >> 4) * 8;
    const float* const wsrc = p.w + (size_t)((lane >> 4) * 8) * NG + wid * NS + sb * 64 + (lane & 15) * 4;
    u4 xh[2] = {}, wh[8] = {};
    f4 xr[4], wr[16];
    auto load_x = [&](int c) {
#pragma unroll
        for (int i = 0; i < 4; ++i) xr[i] = *(const f4*)(xs + c * 64 + (i >> 1) * 32 + (i & 1) * 4);
    };
    auto load_w = [&](int c) {
#pragma unroll
        for (int i = 0; i < 16; ++i) wr[i] = *(const f4*)(wsrc + (size_t)(c * 64 + (i >> 3) * 32 + (i & 7)) * NG);
    };
    auto cvt_x = [&]() { xh[0] = cvt8(xr[0], xr[1]); xh[1] = cvt8(xr[2], xr[3]); };
    auto cvt_w = [&]() {
#pragma unroll
        for (int ks = 0; ks < 2; ++ks)
#pragma unroll
            for (int jj = 0; jj < 4; ++jj) {
                const f4 lo = {wr[ks * 8][jj], wr[ks * 8 + 1][jj], wr[ks * 8 + 2][jj], wr[ks * 8 + 3][jj]};
                const f4 hi = {wr[ks * 8 + 4][jj], wr[ks * 8 + 5][jj], wr[ks * 8 + 6][jj], wr[ks * 8 + 7][jj]};
                wh[ks * 4 + jj] = cvt8(lo, hi);
            }
    };
    const int uc = (b >> 5) == 7 ? 0 : b >> 5;
    const bool ux = b < 224, uw = b >= NOWN;
    if (ux && uw) wait_until((unsigned)tstart + T_START + (unsigned)(b - NOWN) * CONV_TICKS / 16u);
    if (ux) load_x(uc);
    if (uw) load_w(uc);
    unsigned* flg = (unsigned*)(p.ws[lane >> 3] + FLG_OFF + (lane & 7) * FLG_REP);
    asm volatile("" : "+v"(flg));

    if (wid == rank) {
        if (lane == 0) ctl[18] = __hip_atomic_fetch_add((unsigned*)(myws + SEQ_OFF + b * 128), 1u, __ATOMIC_RELAXED, __HIP_MEMORY_SCOPE_AGENT) + 1u;
        if (lane < 16) ctl[lane] = 0;
    }
    BAR();
    const unsigned call = lds_ld(ctl + 18);
    const unsigned stg = STG_OFF + (call & 1u) * SLOT_BYTES;
    const unsigned xoff = stg + (unsigned)(rank * 8) * KT_A + (unsigned)(wid * 32 + sb) * 2048u + voff;
    const unsigned woff = stg + B_REL + (unsigned)(rank * 8) * KT_B + (unsigned)sb * 8192u + voff;

    auto announce = [&](int c, unsigned weight, bool local) {
        unsigned old = 0;
        if (lane == 0) old = __hip_atomic_fetch_add((unsigned*)(myws + AGG_OFF) + ((local ? 9 : 0) + c) * 32, weight, __ATOMIC_RELAXED, __HIP_MEMORY_SCOPE_AGENT);
        if (__builtin_amdgcn_readfirstlane(old) + weight == call * 64u && ((lane >> 3) == rank) == local)
            __hip_atomic_store((GAS unsigned*)flg + (c < 8 ? rank * 8 + c : 64 + rank), call, __ATOMIC_RELAXED, __HIP_MEMORY_SCOPE_SYSTEM);
    };
    auto local_copy = [&](int c, bool dx, bool dw, int cnt) {
        if (dx) { st16(xh[0], rws, xoff + c * KT_A); st16(xh[1], rws, xoff + c * KT_A + 1024); }
        if (dw) {
            const unsigned wo = (wid == rank ? woff : W2_OFF + (unsigned)(wid * 8) * KT_B + (unsigned)sb * 8192u + voff) + c * KT_B;
#pragma unroll
            for (int q = 0; q < 8; ++q) st16(wh[q], rws, wo + q * 1024);
        }
        WAIT_V(0);
        if (lds_inc(ctl + cnt, lane) == 7u) announce(c, (unsigned)dx + (unsigned)dw, true);
    };
    auto send = [&](int c, bool dx, bool dw, unsigned ts) {
        if (dx) {
            const int u = (wid - rank - 1) & 7;
            __amdgpu_buffer_rsrc_t rd[6];
#pragma unroll
            for (int s = 0; s < 6; ++s) { int a = u + 1 + s; if (a >= 7) a -= 7; rd[s] = mk_rsrc(p.ws[(rank + 1 + a) & 7], WS_NEED); }
#pragma unroll
            for (int i = 0; i < 12; ++i) {
                pace<TAU_ST>(ts, c * 20 + i);
                st16(xh[i & 1], rd[i >> 1], xoff + c * KT_A + (i & 1) * 1024);
                WAIT_VN(SWIN - 1);
            }
        }
        STAMPW((rank + 1) & 7, 53);
        if (dw) {
            const __amdgpu_buffer_rsrc_t rj = mk_rsrc(p.ws[wid], WS_NEED);
#pragma unroll
            for (int q = 0; q < 8; ++q) {
                pace<TAU_ST>(ts, c * 20 + 12 + q);
                st16(wh[q], rj, woff + c * KT_B + q * 1024);
                WAIT_VN(SWIN - 1);
            }
        }
    };

    if (ux) cvt_x();
    if (uw) cvt_w();
    bool duty = b >= NOWN;
    if (duty) {
        local_copy(uc, ux, uw, 0);
        STAMP(15);
    } else {
        const unsigned long long t64 = __builtin_amdgcn_s_memrealtime();
        const unsigned ts = (unsigned)t64;
        if (wid == rank) { if (lane == 0) __hip_atomic_fetch_max(torg, t64, __ATOMIC_RELAXED, __HIP_MEMORY_SCOPE_AGENT); }
        else { send(0, true, false, ts); WAIT_V(0); if (lds_inc(ctl + 1, lane) == 6u) announce(0, 1u, false); STAMPW(7, 1); }
        local_copy(0, true, false, 0);
        pace<TAU_ST>(ts, 80 + sb);
        load_x(7); load_w(7);
        cvt_x(); cvt_w();
        local_copy(7, true, true, 2);
        if (wid != rank) {
            const unsigned long long t0 = __builtin_amdgcn_s_memrealtime();
            const unsigned* f = (const unsigned*)(myws + FLG_OFF + xcd * FLG_REP) + rank * 8 + (lane & 7);
            while (!__all((int)(__hip_atomic_load(f, __ATOMIC_RELAXED, __HIP_MEMORY_SCOPE_SYSTEM) - call) >= 0)) {
                if (give_up(tmo, call, t0, lane)) break;
                __builtin_amdgcn_s_sleep(32);
            }
            asm volatile("" ::: "memory");
            u4 xd[16];
#pragma unroll
            for (int q = 0; q < 16; ++q) xd[q] = ld16(rws, xoff + (unsigned)(q >> 1) * KT_A + (q & 1) * 1024);
            const unsigned to = __builtin_amdgcn_readfirstlane((unsigned)__hip_atomic_load(torg, __ATOMIC_RELAXED, __HIP_MEMORY_SCOPE_AGENT));
            WAIT_V(0);
            send(7, true, true, to);
            const __amdgpu_buffer_rsrc_t rj = mk_rsrc(p.ws[wid], WS_NEED);
#pragma unroll
            for (int q = 0; q < 16; ++q) {
                pace<TAU_ST>(to, 160 + q);
                st16(xd[q], rj, xoff + (unsigned)(q >> 1) * KT_A + (q & 1) * 1024);
                WAIT_V(1);
                if (q == 0) { lds_inc(ctl + 3, lane); STAMPW(7, 8); STAMPW(wid, 56 + wid); }
            }
            WAIT_V(0);
            if (lds_inc(ctl + 4, lane) == 6u) announce(8, 2u, false);
            STAMPW(7, 9);
        }
        else {
            pace<TAU_ST>(ts, 158);
            while (lds_ld(ctl + 3) < 7u) __builtin_amdgcn_s_sleep(1);
            announce(7, 2u, false);
        }
        BAR();
    }

    const unsigned abase = stg + (unsigned)(owner * 2 + hh) * STAGE_A, bbase = stg + B_REL + (unsigned)nt * STAGE_B;
    const unsigned abase2 = stg + (unsigned)(rank * 2 + hh) * STAGE_A + (unsigned)((rank - owner) * 8) * KT_A;
    const unsigned bbase2 = W2_OFF + (unsigned)nt * STAGE_B;
    const int osrc = orel == 0 ? -1 : owner;
    const int wm = wid >> 1, wn = wid & 1;
    const int lvoff = wid * 1024 + voff;
    const unsigned* const myflag = (const unsigned*)(myws + FLG_OFF + xcd * FLG_REP)
        + ((lane >> 3) == osrc ? rank * 8 + (lane & 7) : orel == 0 && (lane >> 3) != rank ? 64 + (lane >> 3) : lane);
    f4 acc[4][4], acc2[4][4];
#pragma unroll
    for (int m = 0; m < 4; ++m)
#pragma unroll
        for (int n = 0; n < 4; ++n) { acc[m][n] = (f4){0.f, 0.f, 0.f, 0.f}; acc2[m][n] = (f4){0.f, 0.f, 0.f, 0.f}; }

    auto stage1 = [&](unsigned ao, unsigned bo, int buf, int i) {
        LAS char* const l = (LAS char*)smem + buf * STAGE + wid * 1024 + i * 8192;
        __builtin_amdgcn_raw_ptr_buffer_load_lds(rws, l, 16, lvoff, i < 4 ? ao + i * 8192 : bo + (i - 4) * 8192, 0, AUX_LD);
    };
    auto stage = [&](int kt, int buf, unsigned ab, unsigned bb) {
#pragma unroll
        for (int i = 0; i < 6; ++i) stage1(ab + (unsigned)kt * KT_A, bb + (unsigned)kt * KT_B, buf, i);
    };
    auto frag = [&](h8 (&af)[4], h8 (&bf)[4], int buf, int ks, int g) {
        const LAS char* const l = (const LAS char*)smem + buf * STAGE + voff;
        if (g < 4) bf[g] = *(const LAS h8*)(l + STAGE_A + wn * 8192 + ks * 4096 + g * 1024);
        else af[g - 4] = *(const LAS h8*)(l + wm * 8192 + ks * 1024 + (g - 4) * 2048);
    };
    auto kstep = [&](f4 (&c)[4][4], const h8 (&af)[4], const h8 (&bf)[4], h8 (&an)[4], h8 (&bn)[4], int rbuf, int rks,
                     bool dma, unsigned ao, unsigned bo, int sbuf) {
#pragma unroll
        for (int g = 0; g < 8; ++g) {
            const int m = g >> 1, n0 = (g & 1) * 2;
            c[m][n0] = __builtin_amdgcn_mfma_f32_16x16x32_f16(af[m], bf[n0], c[m][n0], 0, 0, 0);
            if (g < 2) { frag(an, bn, rbuf, rks, 2 * g); frag(an, bn, rbuf, rks, 2 * g + 1); }
            else if (g < 6) frag(an, bn, rbuf, rks, g + 2);
            c[m][n0 + 1] = __builtin_amdgcn_mfma_f32_16x16x32_f16(af[m], bf[n0 + 1], c[m][n0 + 1], 0, 0, 0);
            if (dma && g >= 2) stage1(ao, bo, sbuf, g - 2);
            __builtin_amdgcn_sched_barrier(0);
        }
    };
    int buf = 0;
    auto run = [&](f4 (&c)[4][4], unsigned long long avail, unsigned ab, unsigned bb) {
        int left = __builtin_popcountll(avail);
        int b1 = buf + 1 == NSTAGE ? 0 : buf + 1;
        stage(__builtin_ctzll(avail), buf, ab, bb); avail &= avail - 1;
        if (avail) { stage(__builtin_ctzll(avail), b1, ab, bb); avail &= avail - 1; WAIT_V(6); } else WAIT_V(0);
        BAR();
        h8 a0[4], b0[4], a1[4], b1f[4];
#pragma unroll
        for (int g = 0; g < 8; ++g) frag(a0, b0, buf, 0, g);
        do {
            b1 = buf + 1 == NSTAGE ? 0 : buf + 1;
            const int b2 = b1 + 1 == NSTAGE ? 0 : b1 + 1;
            const unsigned kt = (unsigned)__builtin_ctzll(avail | (1ull << 63));
            const unsigned ao = avail ? ab + kt * KT_A : OOB, bo = avail ? bb + kt * KT_B : OOB;
            avail &= avail - 1;
            kstep(c, a0, b0, a1, b1f, buf, 1, true, ao, bo, b2);
            WAIT_V(6);
            BAR();
            kstep(c, a1, b1f, a0, b0, b1, 0, false, 0, 0, 0);
            buf = b1;
        } while (--left > 0);
    };

    const unsigned long long locm = orel == 0 ? 0ull : 0xffull << (owner * 8);
    unsigned long long done = 0;
    bool bad = false;
    unsigned org = 0, tduty = 0;
    if (duty && wid == 0) {
        unsigned long long o;
        while ((o = __hip_atomic_load(torg, __ATOMIC_RELAXED, __HIP_MEMORY_SCOPE_AGENT)) + 1000ull < tstart) {
            if (give_up(tmo, call, tstart, lane)) break;
            __builtin_amdgcn_s_sleep(4);
        }
        org = __builtin_amdgcn_readfirstlane((unsigned)o);
        tduty = org + slots<TAU_ST>(uc * 20 + (ux ? 0 : 12)) - 200u;
    }
    int nbatch = 0;
    while (done != ~0ull || duty) {
        if (wid == 0) {
            const unsigned long long t0 = __builtin_amdgcn_s_memrealtime();
            unsigned long long m = 0;
            unsigned go = 0;
            for (unsigned it = 1;; ++it) {
                int room = 64;
                if (duty) {
                    const int dt = (int)(tduty - now32());
                    room = (dt - 200) / TILE_TICKS;
                    if (room <= 0) { go = 1; break; }
                }
                const unsigned v = __hip_atomic_load(myflag, __ATOMIC_RELAXED, POLL_SCOPE);
                m = __ballot((int)(v - call) >= 0) & ~done;
                if (m) {
                    if (__builtin_popcountll(m) > room) {
                        unsigned long long rest = m;
                        for (int i = 0; i < room; ++i) rest &= rest - 1;
                        m &= ~rest;
                    }
                    break;
                }
                if ((it & 63u) == 0 && give_up(tmo, call, t0, lane)) break;
                __builtin_amdgcn_s_sleep(POLL_SLEEP);
            }
            if (lane == 0) { ctl[16] = (unsigned)m; ctl[17] = (unsigned)(m >> 32); ctl[19] = go; ctl[20] = org; }
        }
        BAR();
        const unsigned long long avail = ((unsigned long long)lds_ld(ctl + 17) << 32) | lds_ld(ctl + 16);
        if (lds_ld(ctl + 19)) {
            const unsigned ts = lds_ld(ctl + 20);
            STAMPW((rank + 1) & 7, 54);
            if (wid != rank) {
                xh[0] = ld16(rws, xoff + uc * KT_A); xh[1] = ld16(rws, xoff + uc * KT_A + 1024);
#pragma unroll
                for (int q = 0; q < 8; ++q) wh[q] = ld16(rws, W2_OFF + (unsigned)(wid * 8 + uc) * KT_B + (unsigned)sb * 8192u + voff + q * 1024);
                WAIT_V(0);
                STAMPW((rank + 1) & 7, 55);
                send(uc, ux, uw, ts); WAIT_V(0); if (lds_inc(ctl + 1, lane) == 6u) announce(uc, (unsigned)ux + (unsigned)uw, false); STAMPW(7, 1 + uc); STAMPW(wid, 56 + wid);
            }
            duty = false;
        } else if (avail == 0) { bad = true; done = ~0ull; tduty = now32(); }
        if (avail == 0) BAR();
        done |= avail;
        if (avail & locm) run(acc2, avail & locm, abase2, bbase2);
        if (avail & ~locm) run(acc, avail & ~locm, abase, bbase);
    }
    STAMP(10);

    const float qn = __builtin_nanf("");
    if (orel != 0) {
        const unsigned tile = (unsigned)(hh * 16 + nt);
        {
            unsigned tx = lds_ld(ctl + 20) + slots<TAU_ST>(176u) - FLAG_LAT;
            { const unsigned now = now32(); if ((int)(now - tx) > 0) tx = now; }
            if (wid > 0) pace<TAU_LD>(tx, wid * 16 - LEAD);
            if (bad) {
#pragma unroll
                for (int q = 0; q < 16; ++q) acc[q >> 2][q & 3] = (f4){qn, qn, qn, qn};
            }
            const unsigned po = stg + P_REL + (((unsigned)owner * 32u + tile) * 8u + (unsigned)wid) * 16384u + voff;
#pragma unroll
            for (int q = 0; q < 16; ++q) st16(__builtin_bit_cast(u4, acc[q >> 2][q & 3]), rws, po + q * 1024);
            WAIT_V(0);
            wait_until(tx);
            if (lane == 0) __hip_atomic_store((unsigned*)(p.ws[owner] + PFL_OFF + ((unsigned)rank * 32u + tile) * 128u) + wid, call, __ATOMIC_RELAXED, __HIP_MEMORY_SCOPE_SYSTEM);
        }
        STAMP(11);
        {
            const unsigned* f = (const unsigned*)(myws + PFL_OFF + ((unsigned)owner * 32u + tile) * 128u) + wid;
            const unsigned long long t0 = __builtin_amdgcn_s_memrealtime();
            for (unsigned it = 1;; ++it) {
                const unsigned v = __builtin_amdgcn_readfirstlane(__hip_atomic_load(f, __ATOMIC_RELAXED, __HIP_MEMORY_SCOPE_SYSTEM));
                if ((int)(v - call) >= 0) break;
                if ((it & 15u) == 0 && give_up(tmo, call, t0, lane)) { bad = true; break; }
                __builtin_amdgcn_s_sleep(2);
            }
            const unsigned tf = now32();
            if (!bad) bad = __builtin_amdgcn_readfirstlane(__hip_atomic_load(tmo, __ATOMIC_RELAXED, __HIP_MEMORY_SCOPE_AGENT)) == call;
            while (lds_ld(ctl + 12) < (unsigned)wid) __builtin_amdgcn_s_sleep(1);
            unsigned te = wid > 0 ? lds_ld(ctl + 13) : tf;
            if ((int)(tf - te - slots<TAU_LD>(wid * 16)) > 0) te = tf - slots<TAU_LD>(wid * 16);
            STAMP(12);
            const __amdgpu_buffer_rsrc_t ro = mk_rsrc(p.ws[owner], WS_NEED);
            const unsigned pi = stg + P_REL + (((unsigned)rank * 32u + tile) * 8u + (unsigned)wid) * 16384u + voff;
#pragma unroll
            for (int q = 0; q < 16; ++q) {
                pace<TAU_LD>(te, wid * 16 + q);
                acc[q >> 2][q & 3] = __builtin_bit_cast(f4, ld16(ro, pi + q * 1024));
                WAIT_V(1);
            }
            if (lane == 0) {
                __hip_atomic_store(ctl + 13, te, __ATOMIC_RELAXED, __HIP_MEMORY_SCOPE_WORKGROUP);
                __hip_atomic_store(ctl + 12, (unsigned)wid + 1u, __ATOMIC_RELAXED, __HIP_MEMORY_SCOPE_WORKGROUP);
            }
            STAMPW(7, 13);
#pragma unroll
            for (int q = 0; q < 16; ++q) acc[q >> 2][q & 3] += acc2[q >> 2][q & 3];
        }
    } else if (!bad) bad = __builtin_amdgcn_readfirstlane(__hip_atomic_load(tmo, __ATOMIC_RELAXED, __HIP_MEMORY_SCOPE_AGENT)) == call;
    {
        const __amdgpu_buffer_rsrc_t ro = mk_rsrc(p.out, OUT_BYTES);
        const unsigned row0 = (unsigned)(hh * 256 + wm * 64 + (lane >> 4) * 4), col0 = (unsigned)(owner * NS + nt * 128 + wn * 64 + (lane & 15) * 4);
        const unsigned oo = (row0 * NG + col0) * 4u;
#pragma unroll
        for (int m = 0; m < 4; ++m)
#pragma unroll
            for (int r = 0; r < 4; ++r) {
                f4 v = {acc[m][0][r], acc[m][1][r], acc[m][2][r], acc[m][3][r]};
                if (bad) v = (f4){qn, qn, qn, qn};
                st16(__builtin_bit_cast(u4, v), ro, oo + (unsigned)(m * 16 + r) * (NG * 4u));
            }
    }
}

}

extern "C" void kernel_launch(int rank, int nranks, ncclComm_t comm,
                              hipStream_t stream,
                              void* const* d_in, const int* in_sizes, int n_in,
                              void* d_out, int out_size,
                              const int* in_dtypes, int out_dtype,
                              void* const* d_out_all,
                              void* const* d_ws_all, size_t ws_bytes) {
    (void)comm; (void)in_dtypes; (void)out_dtype; (void)d_out_all;
    if (nranks != NR || (unsigned)rank >= (unsigned)NR || n_in < 2 || in_sizes[0] != MG * KS || in_sizes[1] != KS * NG
        || out_size != MS * NG || ws_bytes < WS_NEED) {
        fprintf(stderr, "kernel_launch: unexpected problem shape\n");
        return;
    }
    Params p;
    p.x = (const float*)d_in[0]; p.w = (const float*)d_in[1]; p.out = (float*)d_out; p.rank = rank;
    for (int i = 0; i < NR; ++i) p.ws[i] = (char*)d_ws_all[i];
    hipLaunchKernelGGL(gemm_rs_kernel, dim3(NWG), dim3(512), 0, stream, p);
    HIP_CHECK(hipGetLastError());
}
```

```cpp
#include <hip/hip_runtime.h>
#include <rccl/rccl.h>
#include <cstdio>

#define HIP_CHECK(expr) do { hipError_t _e = (expr); if (_e != hipSuccess) { \
    fprintf(stderr, "HIP error %s at %s:%d\n", hipGetErrorString(_e), __FILE__, __LINE__); \
    return; } } while (0)

namespace {

typedef unsigned u4 __attribute__((ext_vector_type(4)));
typedef float f4 __attribute__((ext_vector_type(4)));
typedef _Float16 h8 __attribute__((ext_vector_type(8)));
#define LAS __attribute__((address_space(3)))
#define GAS __attribute__((address_space(1)))

constexpr int NR = 8;
constexpr int MG = 4096, NG = 16384, KS = 512, NS = 2048, MS = 512;
constexpr int NWG = 256, NOWN = 32;
constexpr int STAGE_A = 32768, STAGE_B = 16384, STAGE = STAGE_A + STAGE_B, NSTAGE = 3;
constexpr int CTL = NSTAGE * STAGE;
constexpr int LDS_BYTES = CTL + 128;
constexpr unsigned TMO_OFF = 0;
constexpr unsigned ORG_OFF = 256;
constexpr unsigned LNK_OFF = 512;
constexpr unsigned SEQ_OFF = 4096;
constexpr unsigned AGG_OFF = 36864;
constexpr unsigned RDY_OFF = 49152;
constexpr unsigned PFL_OFF = 65536;
constexpr unsigned FLG_OFF = 98304, FLG_REP = 4352;
constexpr unsigned STAMP_OFF = 262144;
constexpr unsigned STG_OFF = 1u << 20;
constexpr unsigned A_BYTES = 32u << 20;
constexpr unsigned B_BYTES = 16u << 20;
constexpr unsigned W_BYTES = 16u << 20;
constexpr unsigned P_BYTES = 32u << 20;
constexpr unsigned B_REL = A_BYTES, W_REL = B_REL + B_BYTES, P_REL = W_REL + W_BYTES, SLOT_BYTES = P_REL + P_BYTES;
constexpr unsigned WS_NEED = STG_OFF + 2 * SLOT_BYTES;
constexpr unsigned KT_A = 16u * STAGE_A, KT_B = 16u * STAGE_B;
constexpr unsigned OOB = 0x80000000u;
constexpr unsigned OUT_BYTES = (unsigned)MS * NG * 4;
constexpr unsigned long long DEADLINE = 100000000ull;
constexpr int AUX_SYS = 17;
constexpr int AUX_LD = 16;

struct Params {
    const float* x; const float* w;
    char* ws[NR]; float* out;
    int rank;
};

#define WAIT_V(n) asm volatile("s_waitcnt vmcnt(" #n ")" ::: "memory")
#define WAIT_VN(n) asm volatile("s_waitcnt vmcnt(%0)" :: "i"(n) : "memory")
#define GBS 58.0
#define GBS_OUT 61.0
#define T_START 400
#define CONV_TICKS 192
#define POLL_SLEEP 4
#define RDY_LAT 150
#define SLACK 60
#define PRO_SLEEP 12
#define ENDB 1
#define LEAD 12
#define PD 1
#define GBS_IN 60.6
constexpr unsigned TAU_ST = (unsigned)(32768.0 / GBS / 10.0 * 256.0), TAU_IN = (unsigned)(32768.0 / GBS_IN / 10.0 * 256.0), TAU_LD = (unsigned)(32768.0 / GBS_OUT / 10.0 * 256.0);
#define BAR() do { asm volatile("" ::: "memory"); __builtin_amdgcn_s_waitcnt(0xC07F);   __builtin_amdgcn_s_barrier(); asm volatile("" ::: "memory"); } while (0)

#define STAMPW(w, i) do {} while (0)
#define STAMP(i) STAMPW(0, i)

__device__ __forceinline__ __amdgpu_buffer_rsrc_t mk_rsrc(const void* p, unsigned bytes) {
    return __builtin_amdgcn_make_buffer_rsrc((void*)p, 0, (int)bytes, 0x00020000);
}
__device__ __forceinline__ void st16(u4 v, __amdgpu_buffer_rsrc_t r, unsigned off) {
    __builtin_amdgcn_raw_buffer_store_b128(v, r, (int)off, 0, AUX_SYS);
}
__device__ __forceinline__ u4 ld16(__amdgpu_buffer_rsrc_t r, unsigned off) {
    return __builtin_amdgcn_raw_buffer_load_b128(r, (int)off, 0, AUX_SYS);
}
__device__ __forceinline__ unsigned now32() { return (unsigned)__builtin_amdgcn_s_memrealtime(); }
template <unsigned TAU> __device__ __forceinline__ unsigned slots(unsigned n) { return (n * TAU) >> 8; }
__device__ __forceinline__ void wait_until(unsigned due) {
    while ((int)(now32() - due) < 0) __builtin_amdgcn_s_sleep(1);
    asm volatile("" ::: "memory");
}
template <unsigned TAU> __device__ __forceinline__ void pace(unsigned t0, unsigned n) { wait_until(t0 + slots<TAU>(n)); }
__device__ __forceinline__ unsigned lds_ld(LAS unsigned* p) {
    return __builtin_amdgcn_readfirstlane(__hip_atomic_load(p, __ATOMIC_RELAXED, __HIP_MEMORY_SCOPE_WORKGROUP));
}
__device__ __forceinline__ unsigned lds_inc(LAS unsigned* p, int lane) {
    unsigned old = 0;
    if (lane == 0) old = __hip_atomic_fetch_add(p, 1u, __ATOMIC_RELAXED, __HIP_MEMORY_SCOPE_WORKGROUP);
    return __builtin_amdgcn_readfirstlane(old);
}
__device__ __forceinline__ u4 cvt8(f4 a, f4 b) {
    h8 h = {(_Float16)a[0], (_Float16)a[1], (_Float16)a[2], (_Float16)a[3], (_Float16)b[0], (_Float16)b[1], (_Float16)b[2], (_Float16)b[3]};
    return __builtin_bit_cast(u4, h);
}
__device__ __forceinline__ bool give_up(unsigned* tmo, unsigned call, unsigned long long t0, int lane) {
    if (__builtin_amdgcn_readfirstlane(__hip_atomic_load(tmo, __ATOMIC_RELAXED, __HIP_MEMORY_SCOPE_AGENT)) == call
        || __builtin_amdgcn_s_memrealtime() - t0 > DEADLINE) {
        if (lane == 0) __hip_atomic_store(tmo, call, __ATOMIC_RELAXED, __HIP_MEMORY_SCOPE_AGENT);
        return true;
    }
    return false;
}

__global__ __launch_bounds__(512, 2) void gemm_rs_kernel(const Params p) {
    __shared__ __attribute__((aligned(16))) char smem[LDS_BYTES];
    const int tid = threadIdx.x, lane = tid & 63;
    const int wid = __builtin_amdgcn_readfirstlane(tid >> 6);
    const int b = blockIdx.x, rank = p.rank;
    char* const myws = p.ws[rank];
    LAS unsigned* const ctl = (LAS unsigned*)((LAS char*)smem + CTL);
    STAMP(0);
    const unsigned long long tstart = __builtin_amdgcn_s_memrealtime();

    const int xcd = b & 7, slot = b >> 3;
    const int orel = slot >> 2;
    const int owner = (rank + orel) & 7;
    const int hh = xcd & 1, nt = (xcd >> 1) * 4 + (slot & 3);
    const __amdgpu_buffer_rsrc_t rws = mk_rsrc(myws, WS_NEED);
    const int voff = lane * 16;
    unsigned* const tmo = (unsigned*)(myws + TMO_OFF);
    unsigned long long* const torg = (unsigned long long*)(myws + ORG_OFF);

    const int sb = b & 31;
    const float* const xs = p.x + (size_t)((wid * 32 + sb) * 16 + (lane & 15)) * KS + (lane >> 4) * 8;
    const float* const wsrc = p.w + (size_t)((lane >> 4) * 8) * NG + wid * NS + sb * 64 + (lane & 15) * 4;
    u4 xh[2] = {}, wh[8] = {};
    f4 xr[4], wr[16];
    auto load_x = [&](int c) {
#pragma unroll
        for (int i = 0; i < 4; ++i) xr[i] = *(const f4*)(xs + c * 64 + (i >> 1) * 32 + (i & 1) * 4);
    };
    auto load_w = [&](int c) {
#pragma unroll
        for (int i = 0; i < 16; ++i) wr[i] = *(const f4*)(wsrc + (size_t)(c * 64 + (i >> 3) * 32 + (i & 7)) * NG);
    };
    auto cvt_x = [&]() { xh[0] = cvt8(xr[0], xr[1]); xh[1] = cvt8(xr[2], xr[3]); };
    auto cvt_w = [&]() {
#pragma unroll
        for (int ks = 0; ks < 2; ++ks)
#pragma unroll
            for (int jj = 0; jj < 4; ++jj) {
                const f4 lo = {wr[ks * 8][jj], wr[ks * 8 + 1][jj], wr[ks * 8 + 2][jj], wr[ks * 8 + 3][jj]};
                const f4 hi = {wr[ks * 8 + 4][jj], wr[ks * 8 + 5][jj], wr[ks * 8 + 6][jj], wr[ks * 8 + 7][jj]};
                wh[ks * 4 + jj] = cvt8(lo, hi);
            }
    };
    const int uc = (b >> 5) == 7 ? 0 : b >> 5;
    const bool ux = b < 224, uw = b >= NOWN;
    if (ux && b >= 64) wait_until((unsigned)tstart + T_START + (unsigned)(b - 64) * CONV_TICKS / 16u);
    if (ux) load_x(uc);
    if (uw) load_w(uc);
    unsigned* rdy = (unsigned*)(p.ws[lane & 7] + RDY_OFF + sb * 512);
    asm volatile("" : "+v"(rdy));

    if (wid == rank) {
        if (lane == 0) ctl[18] = __hip_atomic_fetch_add((unsigned*)(myws + SEQ_OFF + b * 128), 1u, __ATOMIC_RELAXED, __HIP_MEMORY_SCOPE_AGENT) + 1u;
        if (lane < 32 && lane != 18) ctl[lane] = 0;
    }
    BAR();
    const unsigned call = lds_ld(ctl + 18);
    const unsigned stg = STG_OFF + (call & 1u) * SLOT_BYTES;
    const unsigned xoff = stg + (unsigned)(rank * 8) * KT_A + (unsigned)(wid * 32 + sb) * 2048u + voff;

    auto arrive = [&](int fw, unsigned weight) {
        unsigned old = 0;
        if (lane == 0) old = __hip_atomic_fetch_add((unsigned*)(myws + AGG_OFF) + fw * 32, weight, __ATOMIC_RELAXED, __HIP_MEMORY_SCOPE_AGENT);
        return old + weight;
    };
    auto raise = [&](int fw, unsigned sum) {
        if (__builtin_amdgcn_readfirstlane(sum) == call * 64u && lane < 8)
            __hip_atomic_store((unsigned*)(myws + FLG_OFF + lane * FLG_REP) + fw, call, __ATOMIC_RELAXED, __HIP_MEMORY_SCOPE_AGENT);
    };
    auto tell_peers = [&](int word) {
        if (lane < NR && lane != rank) __hip_atomic_store((GAS unsigned*)rdy + word, call, __ATOMIC_RELAXED, __HIP_MEMORY_SCOPE_SYSTEM);
    };
    auto store_x = [&](int c) { st16(xh[0], rws, xoff + c * KT_A); st16(xh[1], rws, xoff + c * KT_A + 1024); };
    auto store_w = [&](int c) {
        const unsigned wo = stg + (wid == rank ? B_REL : W_REL) + (unsigned)(wid * 8 + c) * KT_B + (unsigned)sb * 8192u + voff;
#pragma unroll
        for (int q = 0; q < 8; ++q) st16(wh[q], rws, wo + q * 1024);
    };

    if (b >= NOWN) {
        if (ux) { cvt_x(); store_x(uc); }
        cvt_w(); store_w(uc);
        WAIT_V(0);
        if (lds_inc(ctl + 0, lane) == 7u) {
            raise(rank * 8 + uc, arrive(rank * 8 + uc, ux ? 2u : 1u));
            if (!ux) {
                unsigned long long o;
                while ((o = __hip_atomic_load(torg, __ATOMIC_RELAXED, __HIP_MEMORY_SCOPE_AGENT)) + 1000ull < tstart) {
                    if (give_up(tmo, call, tstart, lane)) break;
                    __builtin_amdgcn_s_sleep(4);
                }
                wait_until(__builtin_amdgcn_readfirstlane((unsigned)o) + slots<TAU_ST>(12u) - RDY_LAT);
            }
            tell_peers(rank * 8 + uc);
        }
        if (!ux) {
            wait_until((unsigned)tstart + T_START + (unsigned)(b - 64) * CONV_TICKS / 16u);
            load_x(7); load_w(7);
            cvt_x(); store_x(7); cvt_w(); store_w(7);
            WAIT_V(0);
            if (lds_inc(ctl + 1, lane) == 7u) { raise(rank * 8 + 7, arrive(rank * 8 + 7, 2u)); tell_peers(rank * 8 + 7); }
        }
        STAMP(15);
    } else {
        cvt_x();
        const unsigned long long t64 = __builtin_amdgcn_s_memrealtime();
        if (wid == rank) {
            if (lane == 0) __hip_atomic_fetch_max(torg, t64, __ATOMIC_RELAXED, __HIP_MEMORY_SCOPE_AGENT);
            store_x(0);
            WAIT_V(0);
            const unsigned* f = (const unsigned*)(myws + RDY_OFF + sb * 512);
            bool told = false, all = false;
            unsigned fx = 0, ann = lane < NR && lane != rank ? 0u : 10u;
            pace<TAU_ST>((unsigned)t64, 7);
            for (unsigned it = 1;; ++it) {
                if (!told && lds_ld(ctl + 0) == 7u) { told = true; tell_peers(64 + rank); raise(rank * 8, arrive(rank * 8, 1u)); }
                if (!all && (fx != 0xffu || (it & 7u) == 0)) {
                    const unsigned v = __hip_atomic_load(f + lane, __ATOMIC_RELAXED, __HIP_MEMORY_SCOPE_SYSTEM);
                    const unsigned vx = __hip_atomic_load(f + 64 + (lane & 7), __ATOMIC_RELAXED, __HIP_MEMORY_SCOPE_SYSTEM);
                    unsigned long long full = __ballot((lane >> 3) == rank || (int)(v - call) >= 0);
                    fx = (unsigned)__ballot((lane & 7) == rank || (int)(vx - call) >= 0) & 0xffu;
                    if ((it & 31u) == 0 && give_up(tmo, call, tstart, lane)) { full = ~0ull; fx = 0xffu; }
                    if (lane < 2) __hip_atomic_store(ctl + 4 + lane, (unsigned)(full >> (lane * 32)), __ATOMIC_RELAXED, __HIP_MEMORY_SCOPE_WORKGROUP);
                    all = full == ~0ull && fx == 0xffu;
                }
                const unsigned prog = __hip_atomic_load(ctl + 24 + (lane & 7), __ATOMIC_RELAXED, __HIP_MEMORY_SCOPE_WORKGROUP);
                if (prog > ann && ((fx >> (lane & 7)) & 1u)) {
                    const int fw = ann < 7u ? lane * 8 + (int)ann : ann == 7u ? 72 + lane : ann == 8u ? lane * 8 + 7 : 64 + lane;
                    const unsigned old = __hip_atomic_fetch_add((unsigned*)(myws + AGG_OFF) + fw * 32, 2u, __ATOMIC_RELAXED, __HIP_MEMORY_SCOPE_AGENT);
                    if (old + 2u == call * 64u) {
#pragma unroll
                        for (int r = 0; r < 8; ++r) __hip_atomic_store((unsigned*)(myws + FLG_OFF + r * FLG_REP) + fw, call, __ATOMIC_RELAXED, __HIP_MEMORY_SCOPE_AGENT);
                    }
                    ++ann;
                }
                if (told && __all(ann == 10u)) break;
                __builtin_amdgcn_s_sleep(1);
            }
        } else {
            const int u = (wid - rank - 1) & 7;
            unsigned xg[6];
#pragma unroll
            for (int s = 0; s < 6; ++s) { int a = u + 1 + s; if (a >= 7) a -= 7; xg[s] = (unsigned)(rank + 1 + a) & 7u; }
            {
                __amdgpu_buffer_rsrc_t rd[6];
#pragma unroll
                for (int s = 0; s < 6; ++s) rd[s] = mk_rsrc(p.ws[xg[s]], WS_NEED);
#pragma unroll
                for (int i = 0; i < 12; ++i) {
                    pace<TAU_ST>((unsigned)t64, i);
                    st16(xh[i & 1], rd[i >> 1], xoff + (i & 1) * 1024);
                    if (i == 0) store_x(0); else WAIT_V(1);
                }
            }
            const __amdgpu_buffer_rsrc_t rs = mk_rsrc(p.ws[wid], WS_NEED);
            const unsigned xo = stg + (unsigned)(wid * 8) * KT_A + (unsigned)sb * 2048u + voff;
            const unsigned wro = stg + W_REL + (unsigned)(rank * 8) * KT_B + (unsigned)sb * 8192u + voff;
            const unsigned wlo = stg + B_REL + (unsigned)(wid * 8) * KT_B + (unsigned)sb * 8192u + voff;
            unsigned long long* const lnk = (unsigned long long*)(myws + LNK_OFF + wid * 64);
            unsigned ts = 0;
            auto need = [&](int c) {
                while (!((lds_ld(ctl + 4 + (wid >> 2)) >> ((wid & 3) * 8 + c)) & 1u)) __builtin_amdgcn_s_sleep(1);
                asm volatile("" ::: "memory");
            };
            need(0);
            {
                unsigned long long t = __builtin_amdgcn_s_memrealtime();
                const unsigned long long tp = t64 + slots<TAU_ST>(12u);
                if (t < tp + SLACK) t = tp;
                ts = (unsigned)t - slots<TAU_IN>(12u);
                if (lane == 0) __hip_atomic_fetch_max(lnk, t + slots<TAU_IN>(164u), __ATOMIC_RELAXED, __HIP_MEMORY_SCOPE_AGENT);
            }
            u4 d[4] = {};
            unsigned lo[4] = {OOB, OOB, OOB, OOB};
            auto progress = [&](unsigned v) { if (lane == 0) __hip_atomic_store(ctl + 24 + wid, v, __ATOMIC_RELAXED, __HIP_MEMORY_SCOPE_WORKGROUP); };
            auto slot = [&](int i, unsigned n, unsigned ro, unsigned l) {
                pace<TAU_IN>(ts, n);
                lo[i & 3] = l;
                d[i & 3] = ld16(rs, ro);
                asm volatile("" ::: "memory");
                st16(d[(i - PD) & 3], rws, lo[(i - PD) & 3]);
                asm volatile("" ::: "memory");
            };
#pragma unroll
            for (int i = 12; i < 20; ++i) {
                slot(i, i, wro + (i - 12) * 1024, wlo + (i - 12) * 1024);
                if (i == 12) { WAIT_V(2); lds_inc(ctl + 0, lane); }
            }
            for (int c = 1; c < 8; ++c) {
                need(c);
#pragma unroll
                for (int i = 0; i < 20; ++i) {
                    const int ks = i / 10, r = i % 10;
                    if (r < 6) { const unsigned o = xo + c * KT_A + xg[r] * 65536u + ks * 1024; slot(i, c * 20 + i, o, o); }
                    else slot(i, c * 20 + i, wro + c * KT_B + (ks * 4 + r - 6) * 1024, wlo + c * KT_B + (ks * 4 + r - 6) * 1024);
                    if (i == PD) { WAIT_V(2); progress(c); STAMPW(7, c); }
                    if (i == 10 + PD && c == 7) { WAIT_V(2); progress(8); }
                }
            }
#pragma unroll
            for (int i = 0; i < 16; ++i) {
                const unsigned o = xo + (unsigned)(i >> 1) * KT_A + (unsigned)rank * 65536u + (i & 1) * 1024;
                slot(i, 160 + i, o, o);
                if (i == PD - 1) { WAIT_V(0); progress(9); STAMPW(wid, 56 + wid); STAMPW(7, 8); }
            }
#pragma unroll
            for (int i = 16; i < 16 + PD; ++i) st16(d[(i - PD) & 3], rws, lo[(i - PD) & 3]);
            WAIT_V(0);
            progress(10);
            STAMPW(7, 9);
        }
        BAR();
    }

    const int wm = wid >> 1, wn = wid & 1;
    const int lvoff = wid * 1024 + voff;
    int fw0, fw1;
    unsigned taba, tabb;
    {
        const int src = lane >> 3, c = lane & 7;
        fw0 = fw1 = orel == 0 && src != rank ? 64 + src : lane;
        taba = stg + (unsigned)lane * KT_A + (unsigned)(owner * 2 + hh) * STAGE_A;
        tabb = stg + B_REL + (unsigned)lane * KT_B + (unsigned)nt * STAGE_B;
        if (orel != 0 && src == owner) {
            fw0 = fw1 = rank * 8 + c;
            taba = stg + (unsigned)(rank * 8 + c) * KT_A + (unsigned)(rank * 2 + hh) * STAGE_A;
            tabb = stg + W_REL + (unsigned)lane * KT_B + (unsigned)nt * STAGE_B;
        } else if (orel != 0 && c == 7 && src != rank) {
            const int t = (src - rank) & 7, k = t < orel ? t - 1 : t - 2, ka = k & ~1, kb = k | 1, h = k & 1;
            const int sa = (rank + (ka + 1 < orel ? ka + 1 : ka + 2)) & 7, sb2 = (rank + (kb + 1 < orel ? kb + 1 : kb + 2)) & 7;
            fw0 = h ? sa * 8 + 7 : 72 + sa; fw1 = h ? sb2 * 8 + 7 : 72 + sb2;
            taba += (unsigned)((((wid & 1) ? sb2 : sa) - src) * 8) * KT_A + (unsigned)((h - (wid & 1)) * 1024);
            tabb += (unsigned)((((wid >> 2) ? sb2 : sa) - src) * 8) * KT_B + (unsigned)((h - (wid >> 2)) * 4096);
        }
    }
    const unsigned* const myflags = (const unsigned*)(myws + FLG_OFF + xcd * FLG_REP);
    f4 acc[4][4], acc2[4][4];
#pragma unroll
    for (int m = 0; m < 4; ++m)
#pragma unroll
        for (int n = 0; n < 4; ++n) { acc[m][n] = (f4){0.f, 0.f, 0.f, 0.f}; acc2[m][n] = (f4){0.f, 0.f, 0.f, 0.f}; }

    auto stage1 = [&](unsigned ao, unsigned bo, int buf, int i) {
        LAS char* const l = (LAS char*)smem + buf * STAGE + wid * 1024 + i * 8192;
        __builtin_amdgcn_raw_ptr_buffer_load_lds(rws, l, 16, lvoff, i < 4 ? ao + i * 8192 : bo + (i - 4) * 8192, 0, AUX_LD);
    };
    auto stage = [&](int kt, int buf) {
        const unsigned ao = __builtin_amdgcn_readlane(taba, kt), bo = __builtin_amdgcn_readlane(tabb, kt);
#pragma unroll
        for (int i = 0; i < 6; ++i) stage1(ao, bo, buf, i);
    };
    auto frag = [&](h8 (&af)[4], h8 (&bf)[4], int buf, int ks, int g) {
        const LAS char* const l = (const LAS char*)smem + buf * STAGE + voff;
        if (g < 4) bf[g] = *(const LAS h8*)(l + STAGE_A + wn * 8192 + ks * 4096 + g * 1024);
        else af[g - 4] = *(const LAS h8*)(l + wm * 8192 + ks * 1024 + (g - 4) * 2048);
    };
    auto kstep = [&](f4 (&c)[4][4], const h8 (&af)[4], const h8 (&bf)[4], h8 (&an)[4], h8 (&bn)[4], int rbuf, int rks,
                     bool dma, unsigned ao, unsigned bo, int sbuf) {
#pragma unroll
        for (int g = 0; g < 8; ++g) {
            const int m = g >> 1, n0 = (g & 1) * 2;
            c[m][n0] = __builtin_amdgcn_mfma_f32_16x16x32_f16(af[m], bf[n0], c[m][n0], 0, 0, 0);
            if (g < 2) { frag(an, bn, rbuf, rks, 2 * g); frag(an, bn, rbuf, rks, 2 * g + 1); }
            else if (g < 6) frag(an, bn, rbuf, rks, g + 2);
            c[m][n0 + 1] = __builtin_amdgcn_mfma_f32_16x16x32_f16(af[m], bf[n0 + 1], c[m][n0 + 1], 0, 0, 0);
            if (dma && g >= 2) stage1(ao, bo, sbuf, g - 2);
            __builtin_amdgcn_sched_barrier(0);
        }
    };
    int buf = 0;
    unsigned long long done = 0, seen = 0;
    const unsigned* const pf0 = myflags + fw0;
    const unsigned* const pf1 = myflags + fw1;
    auto run = [&](f4 (&c)[4][4], unsigned long long kind) {
        unsigned long long avail = seen & kind & ~done;
        done |= avail;
        stage(__builtin_ctzll(avail), buf); avail &= avail - 1;
        int sbuf = buf + 1 == NSTAGE ? 0 : buf + 1, ahead = 0, par = 0;
        if (avail) { stage(__builtin_ctzll(avail), sbuf); avail &= avail - 1; sbuf = sbuf + 1 == NSTAGE ? 0 : sbuf + 1; ahead = 1; WAIT_V(6); }
        else {
            if (wid == 0) {
                __builtin_amdgcn_s_sleep(PRO_SLEEP);
                const unsigned v0 = __hip_atomic_load(pf0, __ATOMIC_RELAXED, __HIP_MEMORY_SCOPE_AGENT);
                const unsigned v1 = __hip_atomic_load(pf1, __ATOMIC_RELAXED, __HIP_MEMORY_SCOPE_AGENT);
                const unsigned long long m = __ballot((int)(v0 - call) >= 0 && (int)(v1 - call) >= 0);
                if (lane < 2) ctl[14 + lane] = (unsigned)(m >> (lane * 32));
            }
            WAIT_V(0);
        }
        BAR();
        h8 a0[4], b0[4], a1[4], b1f[4];
#pragma unroll
        for (int g = 0; g < 8; ++g) frag(a0, b0, buf, 0, g);
        if (ahead == 0) {
            avail = (((unsigned long long)lds_ld(ctl + 15) << 32) | lds_ld(ctl + 14)) & kind & ~done;
            done |= avail;
        }
        bool more;
        do {
            unsigned v0 = 0, v1 = 0;
            if (wid == 0) asm volatile("global_load_dword %0, %2, off sc1\n\tglobal_load_dword %1, %3, off sc1" : "=&v"(v0), "=&v"(v1) : "v"(pf0), "v"(pf1) : "memory");
            if (ahead == 0 && avail) { stage(__builtin_ctzll(avail), sbuf); avail &= avail - 1; sbuf = sbuf + 1 == NSTAGE ? 0 : sbuf + 1; ahead = 1; }
            more = ahead != 0;
            const int kt = __builtin_ctzll(avail | (1ull << 63));
            const unsigned ao = avail ? __builtin_amdgcn_readlane(taba, kt) : OOB, bo = avail ? __builtin_amdgcn_readlane(tabb, kt) : OOB;
            const int b1 = buf + 1 == NSTAGE ? 0 : buf + 1;
            kstep(c, a0, b0, a1, b1f, buf, 1, true, ao, bo, sbuf);
            if (avail) { avail &= avail - 1; sbuf = sbuf + 1 == NSTAGE ? 0 : sbuf + 1; } else ahead = 0;
            WAIT_V(6);
            if (wid == 0) {
                asm volatile("" : "+v"(v0), "+v"(v1));
                const unsigned long long m = __ballot((int)(v0 - call) >= 0 && (int)(v1 - call) >= 0);
                if (lane < 2) ctl[8 + par + lane] = (unsigned)(m >> (lane * 32));
            }
            BAR();
            const unsigned mlo = ctl[8 + par], mhi = ctl[9 + par];
            kstep(c, a1, b1f, a0, b0, b1, 0, false, 0, 0, 0);
            seen = ((unsigned long long)__builtin_amdgcn_readfirstlane(mhi) << 32) | (unsigned)__builtin_amdgcn_readfirstlane(mlo);
            if (more) { avail |= seen & kind & ~done; done |= avail; }
            par ^= 2;
            buf = b1;
        } while (more);
    };

    const unsigned long long locm = orel == 0 ? 0ull : 0xffull << (owner * 8);
    bool bad = false;
    int nbatch = 0;
    while (done != ~0ull) {
        if (!(seen & ~done)) {
            if (wid == 0) {
                const unsigned long long t0 = __builtin_amdgcn_s_memrealtime();
                unsigned long long m;
                for (unsigned it = 1;; ++it) {
                    const unsigned v0 = __hip_atomic_load(pf0, __ATOMIC_RELAXED, __HIP_MEMORY_SCOPE_AGENT);
                    const unsigned v1 = __hip_atomic_load(pf1, __ATOMIC_RELAXED, __HIP_MEMORY_SCOPE_AGENT);
                    m = __ballot((int)(v0 - call) >= 0 && (int)(v1 - call) >= 0);
                    if ((m & ~done) || ((it & 63u) == 0 && give_up(tmo, call, t0, lane))) break;
                    __builtin_amdgcn_s_sleep(POLL_SLEEP);
                }
                if (lane == 0) { ctl[16] = (unsigned)m; ctl[17] = (unsigned)(m >> 32); }
            }
            BAR();
            seen = ((unsigned long long)lds_ld(ctl + 17) << 32) | lds_ld(ctl + 16);
            if (!(seen & ~done)) { bad = true; done = ~0ull; }
        }
        if (seen & ~done & locm) run(acc2, locm);
        if (seen & ~done & ~locm) run(acc, ~locm);
    }
    STAMP(10);

    const float qn = __builtin_nanf("");
    const __amdgpu_buffer_rsrc_t rout = mk_rsrc(p.out, OUT_BYTES);
    const unsigned oo = ((unsigned)(hh * 256 + wm * 64 + (lane >> 4) * 4) * NG + (unsigned)(owner * NS + nt * 128 + wn * 64 + (lane & 15) * 4)) * 4u;
    auto put = [&](int m) {
#pragma unroll
        for (int r = 0; r < 4; ++r) {
            f4 v = {acc[m][0][r] + acc2[m][0][r], acc[m][1][r] + acc2[m][1][r], acc[m][2][r] + acc2[m][2][r], acc[m][3][r] + acc2[m][3][r]};
            if (bad) v = (f4){qn, qn, qn, qn};
            st16(__builtin_bit_cast(u4, v), rout, oo + (unsigned)(m * 16 + r) * (NG * 4u));
        }
    };
    if (orel != 0) {
        const unsigned tile = (unsigned)(hh * 16 + nt);
        {
            if (wid > 0) pace<TAU_LD>(now32(), wid * 16 - LEAD);
            if (bad) {
#pragma unroll
                for (int q = 0; q < 16; ++q) acc[q >> 2][q & 3] = (f4){qn, qn, qn, qn};
            }
            const unsigned po = stg + P_REL + (((unsigned)owner * 32u + tile) * 8u + (unsigned)wid) * 16384u + voff;
#pragma unroll
            for (int q = 0; q < 16; ++q) st16(__builtin_bit_cast(u4, acc[q >> 2][q & 3]), rws, po + q * 1024);
            WAIT_V(0);
            if (lane == 0) __hip_atomic_store((unsigned*)(p.ws[owner] + PFL_OFF + ((unsigned)rank * 32u + tile) * 128u) + wid, call, __ATOMIC_RELAXED, __HIP_MEMORY_SCOPE_SYSTEM);
        }
        STAMP(11);
        {
            const unsigned tl = __builtin_amdgcn_readfirstlane((unsigned)__hip_atomic_load((unsigned long long*)(myws + LNK_OFF + owner * 64), __ATOMIC_RELAXED, __HIP_MEMORY_SCOPE_AGENT));
            const unsigned* f = (const unsigned*)(myws + PFL_OFF + ((unsigned)owner * 32u + tile) * 128u) + wid;
            const unsigned long long t0 = __builtin_amdgcn_s_memrealtime();
            for (unsigned it = 1;; ++it) {
                const unsigned v = __hip_atomic_load(f, __ATOMIC_RELAXED, __HIP_MEMORY_SCOPE_SYSTEM);
                const unsigned tm = __hip_atomic_load(tmo, __ATOMIC_RELAXED, __HIP_MEMORY_SCOPE_AGENT);
                if (__builtin_amdgcn_readfirstlane(tm) == call) { bad = true; break; }
                if ((int)(__builtin_amdgcn_readfirstlane(v) - call) >= 0) break;
                if ((it & 15u) == 0 && give_up(tmo, call, t0, lane)) { bad = true; break; }
                __builtin_amdgcn_s_sleep(1);
            }
            const unsigned tf = now32();
            while (lds_ld(ctl + 12) < (unsigned)wid) __builtin_amdgcn_s_sleep(1);
            unsigned te = wid > 0 ? lds_ld(ctl + 13) : tl;
            if ((int)(tf - te - slots<TAU_LD>(wid * 16)) > 0) te = tf - slots<TAU_LD>(wid * 16);
            const unsigned tlast = tl + slots<TAU_LD>(128u - ENDB);
            STAMP(12);
            const __amdgpu_buffer_rsrc_t ro = mk_rsrc(p.ws[owner], WS_NEED);
            const unsigned pi = stg + P_REL + (((unsigned)rank * 32u + tile) * 8u + (unsigned)wid) * 16384u + voff;
#pragma unroll
            for (int q = 0; q < 16; ++q) {
                const unsigned due = te + slots<TAU_LD>(wid * 16 + q);
                wait_until((int)(due - tlast) > 0 ? tlast : due);
                acc[q >> 2][q & 3] = __builtin_bit_cast(f4, ld16(ro, pi + q * 1024));
                if ((q & 3) == 1 && q > 4) WAIT_V(5); else WAIT_V(1);
                if ((q & 3) == 0 && q > 0) put((q >> 2) - 1);
            }
            if (lane == 0) {
                __hip_atomic_store(ctl + 13, te, __ATOMIC_RELAXED, __HIP_MEMORY_SCOPE_WORKGROUP);
                __hip_atomic_store(ctl + 12, (unsigned)wid + 1u, __ATOMIC_RELAXED, __HIP_MEMORY_SCOPE_WORKGROUP);
            }
            STAMPW(7, 13);
            put(3);
        }
    } else {
        if (!bad) bad = __builtin_amdgcn_readfirstlane(__hip_atomic_load(tmo, __ATOMIC_RELAXED, __HIP_MEMORY_SCOPE_AGENT)) == call;
#pragma unroll
        for (int m = 0; m < 4; ++m) put(m);
    }
}

}

extern "C" void kernel_launch(int rank, int nranks, ncclComm_t comm,
                              hipStream_t stream,
                              void* const* d_in, const int* in_sizes, int n_in,
                              void* d_out, int out_size,
                              const int* in_dtypes, int out_dtype,
                              void* const* d_out_all,
                              void* const* d_ws_all, size_t ws_bytes) {
    (void)comm; (void)in_dtypes; (void)out_dtype; (void)d_out_all;
    if (nranks != NR || (unsigned)rank >= (unsigned)NR || n_in < 2 || in_sizes[0] != MG * KS || in_sizes[1] != KS * NG
        || out_size != MS * NG || ws_bytes < WS_NEED) {
        fprintf(stderr, "kernel_launch: unexpected problem shape\n");
        return;
    }
    Params p;
    p.x = (const float*)d_in[0]; p.w = (const float*)d_in[1]; p.out = (float*)d_out; p.rank = rank;
    for (int i = 0; i < NR; ++i) p.ws[i] = (char*)d_ws_all[i];
    hipLaunchKernelGGL(gemm_rs_kernel, dim3(NWG), dim3(512), 0, stream, p);
    HIP_CHECK(hipGetLastError());
}
```

```cpp
#include <hip/hip_runtime.h>
#include <rccl/rccl.h>
#include <cstdio>

#define HIP_CHECK(expr) do { hipError_t _e = (expr); if (_e != hipSuccess) { \
    fprintf(stderr, "HIP error %s at %s:%d\n", hipGetErrorString(_e), __FILE__, __LINE__); \
    return; } } while (0)

namespace {

typedef unsigned u4 __attribute__((ext_vector_type(4)));
typedef float f4 __attribute__((ext_vector_type(4)));
typedef _Float16 h8 __attribute__((ext_vector_type(8)));
#define LAS __attribute__((address_space(3)))
#define GAS __attribute__((address_space(1)))

constexpr int NR = 8;
constexpr int MG = 4096, NG = 16384, KS = 512, NS = 2048, MS = 512;
constexpr int NWG = 256, NOWN = 32;
constexpr int STAGE_A = 32768, STAGE_B = 16384, STAGE = STAGE_A + STAGE_B, NSTAGE = 3;
constexpr int CTL = NSTAGE * STAGE;
constexpr int LDS_BYTES = CTL + 128;
constexpr unsigned TMO_OFF = 0;
constexpr unsigned LNK_OFF = 3072;
constexpr unsigned ORG_OFF = 1024;
constexpr unsigned SEQ_OFF = 4096;
constexpr unsigned AGG_OFF = 36864;
constexpr unsigned PFL_OFF = 65536;
constexpr unsigned FLG_OFF = 98304, FLG_REP = 4352;
constexpr unsigned RDY_OFF = 147456;
constexpr unsigned EXP_OFF = 196608;
constexpr unsigned STAMP_OFF = 262144;
constexpr unsigned STG_OFF = 1u << 20;
constexpr unsigned A_BYTES = 32u << 20;
constexpr unsigned B_BYTES = 16u << 20;
constexpr unsigned W_BYTES = 16u << 20;
constexpr unsigned P_BYTES = 32u << 20;
constexpr unsigned B_REL = A_BYTES, W_REL = B_REL + B_BYTES, P_REL = W_REL + W_BYTES, SLOT_BYTES = P_REL + P_BYTES;
constexpr unsigned WS_NEED = STG_OFF + 2 * SLOT_BYTES;
constexpr unsigned KT_A = 16u * STAGE_A, KT_B = 16u * STAGE_B;
constexpr unsigned OOB = 0x80000000u;
constexpr unsigned OUT_BYTES = (unsigned)MS * NG * 4;
constexpr unsigned long long DEADLINE = 100000000ull;
constexpr int AUX_SYS = 17;
constexpr int AUX_LD = 16;

struct Params {
    const float* x; const float* w;
    char* ws[NR]; float* out;
    int rank;
};

#define WAIT_V(n) asm volatile("s_waitcnt vmcnt(" #n ")" ::: "memory")
#define WAIT_VN(n) asm volatile("s_waitcnt vmcnt(%0)" :: "i"(n) : "memory")
#define GBS 58.0
#define GBS_OUT 61.0
#define T_START 400
#define CONV_TICKS 192
#define POLL_SLEEP 4
#define RDY_LAT 150
#define SLACK 60
#define PRO_SLEEP 12
#define ENDB 1
#define LOW_EXP 9
constexpr unsigned LOW_BITS = (127u - LOW_EXP) << 23;
#define GAP_IN 51
#define GAP_OUT 51
#define LEAD 12
#define GBS_IN 60.6
constexpr unsigned TAU_ST = (unsigned)(32768.0 / GBS / 10.0 * 256.0), TAU_IN = (unsigned)(32768.0 / GBS_IN / 10.0 * 256.0), TAU_LD = (unsigned)(32768.0 / GBS_OUT / 10.0 * 256.0);
#define BAR() do { asm volatile("" ::: "memory"); __builtin_amdgcn_s_waitcnt(0xC07F);   __builtin_amdgcn_s_barrier(); asm volatile("" ::: "memory"); } while (0)

#define STAMPW(w, i) do {} while (0)
#define STAMP(i) STAMPW(0, i)

__device__ __forceinline__ __amdgpu_buffer_rsrc_t mk_rsrc(const void* p, unsigned bytes) {
    return __builtin_amdgcn_make_buffer_rsrc((void*)p, 0, (int)bytes, 0x00020000);
}
__device__ __forceinline__ void st16(u4 v, __amdgpu_buffer_rsrc_t r, unsigned off) {
    __builtin_amdgcn_raw_buffer_store_b128(v, r, (int)off, 0, AUX_SYS);
}
__device__ __forceinline__ u4 ld16(__amdgpu_buffer_rsrc_t r, unsigned off) {
    return __builtin_amdgcn_raw_buffer_load_b128(r, (int)off, 0, AUX_SYS);
}
__device__ __forceinline__ unsigned now32() { return (unsigned)__builtin_amdgcn_s_memrealtime(); }
template <unsigned TAU> __device__ __forceinline__ unsigned slots(unsigned n) { return (n * TAU) >> 8; }
__device__ __forceinline__ void wait_until(unsigned due) {
    while ((int)(now32() - due) < 0) __builtin_amdgcn_s_sleep(1);
    asm volatile("" ::: "memory");
}
template <unsigned TAU> __device__ __forceinline__ void pace(unsigned t0, unsigned n) { wait_until(t0 + slots<TAU>(n)); }
__device__ __forceinline__ void pace_soft(unsigned due, unsigned& next, unsigned gap) {
    if ((int)(next - due) > 0) due = next;
    const unsigned now = now32();
    if ((int)(now - due) > 0) due = now; else wait_until(due);
    asm volatile("" ::: "memory");
    next = due + gap;
}
__device__ __forceinline__ unsigned lds_ld(LAS unsigned* p) {
    return __builtin_amdgcn_readfirstlane(__hip_atomic_load(p, __ATOMIC_RELAXED, __HIP_MEMORY_SCOPE_WORKGROUP));
}
__device__ __forceinline__ unsigned lds_add(LAS unsigned* p, unsigned v, int lane) {
    unsigned old = 0;
    if (lane == 0) old = __hip_atomic_fetch_add(p, v, __ATOMIC_RELAXED, __HIP_MEMORY_SCOPE_WORKGROUP);
    return __builtin_amdgcn_readfirstlane(old) + v;
}
__device__ __forceinline__ u4 cvt8(f4 a, f4 b) {
    h8 h = {(_Float16)a[0], (_Float16)a[1], (_Float16)a[2], (_Float16)a[3], (_Float16)b[0], (_Float16)b[1], (_Float16)b[2], (_Float16)b[3]};
    return __builtin_bit_cast(u4, h);
}
template <int N> __device__ __forceinline__ int block_scale(f4 (&v)[N]) {
    float m = 0.f;
#pragma unroll
    for (int i = 0; i < N; ++i) m = fmaxf(fmaxf(m, fmaxf(fabsf(v[i][0]), fabsf(v[i][1]))), fmaxf(fabsf(v[i][2]), fabsf(v[i][3])));
    if (__builtin_expect(!__any(m >= 32768.f) && __any(m >= __uint_as_float(LOW_BITS)), 1)) return 0;
    unsigned mb = 0;
#pragma unroll
    for (int i = 0; i < N; ++i)
#pragma unroll
        for (int j = 0; j < 4; ++j) { const unsigned t = __float_as_uint(v[i][j]) & 0x7fffffffu; if (t < 0x7f800000u && t > mb) mb = t; }
#pragma unroll
    for (int o = 32; o > 0; o >>= 1) { const unsigned t = (unsigned)__shfl_xor((int)mb, o); mb = t > mb ? t : mb; }
    mb = __builtin_amdgcn_readfirstlane(mb);
    if (mb == 0u || (mb >= LOW_BITS && mb < 0x47000000u)) return 0;
    const int ex = mb >= 0x00800000u ? (int)(mb >> 23) - 127 : -118 - __builtin_clz(mb);
    const int e = (ex - 6) & ~7;
#pragma unroll
    for (int i = 0; i < N; ++i)
#pragma unroll
        for (int j = 0; j < 4; ++j) v[i][j] = ldexpf(v[i][j], -e);
    return e;
}
__device__ __forceinline__ bool give_up(unsigned* tmo, unsigned call, unsigned long long t0, int lane) {
    if (__builtin_amdgcn_readfirstlane(__hip_atomic_load(tmo, __ATOMIC_RELAXED, __HIP_MEMORY_SCOPE_AGENT)) == call
        || __builtin_amdgcn_s_memrealtime() - t0 > DEADLINE) {
        if (lane == 0) __hip_atomic_store(tmo, call, __ATOMIC_RELAXED, __HIP_MEMORY_SCOPE_AGENT);
        return true;
    }
    return false;
}

__global__ __launch_bounds__(512, 2) void gemm_rs_kernel(const Params p) {
    __shared__ __attribute__((aligned(16))) char smem[LDS_BYTES];
    const int tid = threadIdx.x, lane = tid & 63;
    const int wid = __builtin_amdgcn_readfirstlane(tid >> 6);
    const int b = blockIdx.x, rank = p.rank;
    char* const myws = p.ws[rank];
    LAS unsigned* const ctl = (LAS unsigned*)((LAS char*)smem + CTL);
    STAMP(0);
    const unsigned long long tstart = __builtin_amdgcn_s_memrealtime();
    if (wid == rank) {
        if (lane == 0) ctl[18] = __hip_atomic_fetch_add((unsigned*)(myws + SEQ_OFF + b * 128), 1u, __ATOMIC_RELAXED, __HIP_MEMORY_SCOPE_AGENT) + 1u;
        if (lane < 32 && lane != 18) ctl[lane] = 0;
    }

    const int xcd = b & 7, slot = b >> 3;
    const int orel = slot >> 2;
    const int owner = (rank + orel) & 7;
    const int hh = xcd & 1, nt = (xcd >> 1) * 4 + (slot & 3);
    const __amdgpu_buffer_rsrc_t rws = mk_rsrc(myws, WS_NEED);
    const int voff = lane * 16;
    unsigned* const tmo = (unsigned*)(myws + TMO_OFF);

    const int sb = b & 31;
    unsigned long long* const torg = (unsigned long long*)(myws + ORG_OFF + sb * 64);
    const float* const xs = p.x + (size_t)((wid * 32 + sb) * 16 + (lane & 15)) * KS + (lane >> 4) * 8;
    const float* const wsrc = p.w + (size_t)((lane >> 4) * 8) * NG + wid * NS + sb * 64 + (lane & 15) * 4;
    u4 xh[2] = {}, wh[8] = {};
    f4 xr[4], wr[16];
    int xe = 0, we = 0;
    auto load_x = [&](int c) {
#pragma unroll
        for (int i = 0; i < 4; ++i) xr[i] = *(const f4*)(xs + c * 64 + (i >> 1) * 32 + (i & 1) * 4);
    };
    auto load_w = [&](int c) {
#pragma unroll
        for (int i = 0; i < 16; ++i) wr[i] = *(const f4*)(wsrc + (size_t)(c * 64 + (i >> 3) * 32 + (i & 7)) * NG);
    };
    auto cvt_x = [&]() { xe = block_scale(xr); xh[0] = cvt8(xr[0], xr[1]); xh[1] = cvt8(xr[2], xr[3]); };
    auto cvt_w = [&]() {
        we = block_scale(wr);
#pragma unroll
        for (int ks = 0; ks < 2; ++ks)
#pragma unroll
            for (int jj = 0; jj < 4; ++jj) {
                const f4 lo = {wr[ks * 8][jj], wr[ks * 8 + 1][jj], wr[ks * 8 + 2][jj], wr[ks * 8 + 3][jj]};
                const f4 hi = {wr[ks * 8 + 4][jj], wr[ks * 8 + 5][jj], wr[ks * 8 + 6][jj], wr[ks * 8 + 7][jj]};
                wh[ks * 4 + jj] = cvt8(lo, hi);
            }
    };
    const int uc = (b >> 5) == 7 ? 0 : b >> 5;
    const bool ux = b < 224, uw = b >= NOWN;
    if (ux && b >= 64) wait_until((unsigned)tstart + T_START + (unsigned)(b - 64) * CONV_TICKS / 16u);
    if (ux) load_x(uc);
    if (uw) load_w(uc);
    BAR();
    const unsigned call = lds_ld(ctl + 18), call2 = 2u * call;
    const unsigned stg = STG_OFF + (call & 1u) * SLOT_BYTES;
    const unsigned rdyo = RDY_OFF + (call & 1u) * 16384u + (unsigned)sb * 512u, expo = EXP_OFF + (call & 1u) * 16384u;
    const unsigned xoff = stg + (unsigned)(rank * 8) * KT_A + (unsigned)(wid * 32 + sb) * 2048u + voff;

    auto arrive = [&](bool on, int fw, unsigned weight, bool scaled) {
        if (on) {
            unsigned long long* cnt = (unsigned long long*)(myws + AGG_OFF + fw * 128);
            const unsigned long long add = weight | ((unsigned long long)scaled << 32);
            const unsigned long long sum = __hip_atomic_fetch_add(cnt, add, __ATOMIC_RELAXED, __HIP_MEMORY_SCOPE_AGENT) + add;
            if (((unsigned)sum & 0x7fffffffu) == ((call * 64u) & 0x7fffffffu)) {
#pragma unroll
                for (int r = 0; r < 8; ++r)
                    __hip_atomic_store((unsigned*)(myws + FLG_OFF + r * FLG_REP) + fw, call2 + (unsigned)((sum >> 32) != 0), __ATOMIC_RELAXED, __HIP_MEMORY_SCOPE_AGENT);
                if (sum >> 31) __hip_atomic_fetch_and(cnt, 0x7fffffffull, __ATOMIC_RELAXED, __HIP_MEMORY_SCOPE_AGENT);
            }
        }
    };
    auto tell_peers = [&](int word, bool scaled) {
        char* w = p.ws[0];
#pragma unroll
        for (int i = 1; i < NR; ++i) if (lane == i) w = p.ws[i];
        if (lane < NR && lane != rank) __hip_atomic_store((GAS unsigned*)(w + rdyo) + word, call2 + (unsigned)scaled, __ATOMIC_RELAXED, __HIP_MEMORY_SCOPE_SYSTEM);
    };
    auto store_e = [&](int which, int c, int e) {
        if (lane == 0) __hip_atomic_store((int*)(myws + expo) + ((which * 8 + c) * 32 + sb) * 8 + wid, e, __ATOMIC_RELAXED, __HIP_MEMORY_SCOPE_SYSTEM);
    };
    auto store_x = [&](int c) { st16(xh[0], rws, xoff + c * KT_A); st16(xh[1], rws, xoff + c * KT_A + 1024); store_e(0, c, xe); };
    auto store_w = [&](int c) {
        const unsigned wo = stg + (wid == rank ? B_REL : W_REL) + (unsigned)(wid * 8 + c) * KT_B + (unsigned)sb * 8192u + voff;
#pragma unroll
        for (int q = 0; q < 8; ++q) st16(wh[q], rws, wo + q * 1024);
        store_e(1, c, we);
    };

    if (b >= NOWN) {
        if (ux) { cvt_x(); store_x(uc); }
        cvt_w(); store_w(uc);
        WAIT_V(0);
        unsigned sum = lds_add(ctl + 0, (xe | we) ? 257u : 1u, lane);
        if ((sum & 255u) == 8u) {
            arrive(lane == 0, rank * 8 + uc, ux ? 2u : 1u, sum > 255u);
            if (!ux) {
                unsigned long long o;
                bool late = false;
                while ((o = __hip_atomic_load(torg, __ATOMIC_RELAXED, __HIP_MEMORY_SCOPE_AGENT)) + 1000ull < tstart) {
                    if (give_up(tmo, call, tstart, lane)) { late = true; break; }
                    __builtin_amdgcn_s_sleep(4);
                }
                if (!late) wait_until(__builtin_amdgcn_readfirstlane((unsigned)o) + slots<TAU_ST>(12u) - RDY_LAT);
            }
            tell_peers(rank * 8 + uc, sum > 255u);
        }
        if (!ux) {
            wait_until((unsigned)tstart + T_START + (unsigned)(b - 64) * CONV_TICKS / 16u);
            load_x(7); load_w(7);
            cvt_x(); store_x(7); cvt_w(); store_w(7);
            WAIT_V(0);
            sum = lds_add(ctl + 1, (xe | we) ? 257u : 1u, lane);
            if ((sum & 255u) == 8u) { arrive(lane == 0, rank * 8 + 7, 2u, sum > 255u); tell_peers(rank * 8 + 7, sum > 255u); }
        }
        STAMP(15);
    } else {
        cvt_x();
        asm volatile("" : "+v"(xh[0]), "+v"(xh[1]));
        const unsigned long long t64 = __builtin_amdgcn_s_memrealtime();
        if (wid == rank) {
            store_x(0);
            WAIT_V(0);
            lds_add(ctl + 0, xe ? 257u : 1u, lane);
            const unsigned* f = (const unsigned*)(myws + rdyo);
            bool told = false, all = false;
            unsigned fx = 0, ann = lane < NR && lane != rank ? 0u : 10u;
            unsigned long long scl = 0;
            pace<TAU_ST>((unsigned)t64, 6);
            for (unsigned it = 1;; ++it) {
                if (!told) {
                    const unsigned sum = lds_ld(ctl + 0);
                    if ((sum & 255u) == 8u) { told = true; tell_peers(64 + rank, sum > 255u); arrive(lane == 0, rank * 8, 1u, sum > 255u); }
                }
                if (!all && (fx != 0xffu || (it & 7u) == 0)) {
                    const unsigned v = __hip_atomic_load(f + lane, __ATOMIC_RELAXED, __HIP_MEMORY_SCOPE_SYSTEM);
                    const unsigned vx = __hip_atomic_load(f + 64 + (lane >> 3), __ATOMIC_RELAXED, __HIP_MEMORY_SCOPE_SYSTEM);
                    unsigned long long full = __ballot((lane >> 3) == rank || (int)(v - call2) >= 0);
                    const unsigned long long fx8 = __ballot((lane >> 3) == rank || (int)(vx - call2) >= 0);
                    scl = __ballot(v == call2 + 1u || ((lane & 7) == 0 && vx == call2 + 1u));
                    fx = (unsigned)__ballot((fx8 >> ((lane & 7) * 8)) & 1u) & 0xffu;
                    if ((it & 31u) == 0 && give_up(tmo, call, tstart, lane)) { full = ~0ull; fx = 0xffu; }
                    if (lane < 3) __hip_atomic_store(ctl + 4 + lane, lane == 2 ? fx : (unsigned)(full >> (lane * 32)), __ATOMIC_RELAXED, __HIP_MEMORY_SCOPE_WORKGROUP);
                    all = full == ~0ull && fx == 0xffu;
                }
                const unsigned prog = __hip_atomic_load(ctl + 24 + (lane & 7), __ATOMIC_RELAXED, __HIP_MEMORY_SCOPE_WORKGROUP);
                const bool on = prog > ann && ((fx >> (lane & 7)) & 1u);
                const unsigned sc = (unsigned)(scl >> ((lane & 7) * 8)) & 0xffu;
                arrive(on, ann < 7u ? lane * 8 + (int)ann : ann == 7u ? 72 + lane : ann == 8u ? lane * 8 + 7 : 64 + lane, 2u,
                       ann < 9u ? (sc >> (ann < 7u ? ann : 7u)) & 1u : sc != 0u);
                if (on) ++ann;
                if (told && __all(ann == 10u)) break;
                __builtin_amdgcn_s_sleep(1);
            }
        } else {
            const int u = (wid - rank - 1) & 7;
            unsigned xg[6];
#pragma unroll
            for (int s = 0; s < 6; ++s) { int a = u + 1 + s; if (a >= 7) a -= 7; xg[s] = (unsigned)(rank + 1 + a) & 7u; }
            {
                __amdgpu_buffer_rsrc_t rd[6];
#pragma unroll
                for (int s = 0; s < 6; ++s) rd[s] = mk_rsrc(p.ws[xg[s]], WS_NEED);
#pragma unroll
                for (int i = 0; i < 12; ++i) {
                    pace<TAU_ST>((unsigned)t64, i);
                    st16(xh[i & 1], rd[i >> 1], xoff + (i & 1) * 1024);
                    if (i == 0) {
                        store_x(0);
                        if (u == 0 && lane == 0) __hip_atomic_store(torg, t64, __ATOMIC_RELAXED, __HIP_MEMORY_SCOPE_AGENT);
                        STAMPW(7, 52);
                    } else WAIT_V(1);
                }
            }
            const __amdgpu_buffer_rsrc_t rs = mk_rsrc(p.ws[wid], WS_NEED);
            const unsigned xo = stg + (unsigned)(wid * 8) * KT_A + (unsigned)sb * 2048u + voff;
            const unsigned wro = stg + W_REL + (unsigned)(rank * 8) * KT_B + (unsigned)sb * 8192u + voff;
            const unsigned wlo = stg + B_REL + (unsigned)(wid * 8) * KT_B + (unsigned)sb * 8192u + voff;
            unsigned ts = 0;
            auto need = [&](int word, int bit) {
                while (!((lds_ld(ctl + word) >> bit) & 1u)) __builtin_amdgcn_s_sleep(1);
                asm volatile("" ::: "memory");
            };
            need(4 + (wid >> 2), (wid & 3) * 8);
            {
                unsigned long long t = __builtin_amdgcn_s_memrealtime();
                const unsigned long long tp = t64 + slots<TAU_ST>(12u);
                if (t < tp + SLACK) t = tp;
                ts = (unsigned)t - slots<TAU_IN>(12u);
                if (lane == 0) __hip_atomic_store((unsigned*)(myws + LNK_OFF) + wid * 32 + sb, (unsigned)t + slots<TAU_IN>(164u), __ATOMIC_RELAXED, __HIP_MEMORY_SCOPE_AGENT);
            }
            unsigned next = ts;
            u4 d[2] = {};
            unsigned lo[2] = {OOB, OOB};
            auto progress = [&](unsigned v) { if (lane == 0) __hip_atomic_store(ctl + 24 + wid, v, __ATOMIC_RELAXED, __HIP_MEMORY_SCOPE_WORKGROUP); };
            auto slot = [&](int i, unsigned n, unsigned ro, unsigned l) {
                pace_soft(ts + slots<TAU_IN>(n), next, GAP_IN);
                lo[i & 1] = l;
                d[i & 1] = ld16(rs, ro);
                asm volatile("" ::: "memory");
                st16(d[(i & 1) ^ 1], rws, lo[(i & 1) ^ 1]);
                asm volatile("" ::: "memory");
            };
#pragma unroll
            for (int i = 12; i < 20; ++i) {
                slot(i, i, wro + (i - 12) * 1024, wlo + (i - 12) * 1024);
                if (i == 12) { WAIT_V(2); lds_add(ctl + 0, xe ? 257u : 1u, lane); }
            }
            for (int c = 1; c < 8; ++c) {
                need(4 + (wid >> 2), (wid & 3) * 8 + c);
#pragma unroll
                for (int i = 0; i < 20; ++i) {
                    const int ks = i / 10, r = i % 10;
                    if (r < 6) { const unsigned o = xo + c * KT_A + xg[r] * 65536u + ks * 1024; slot(i, c * 20 + i, o, o); }
                    else slot(i, c * 20 + i, wro + c * KT_B + (ks * 4 + r - 6) * 1024, wlo + c * KT_B + (ks * 4 + r - 6) * 1024);
                    if (i == 1) { WAIT_V(2); progress(c); STAMPW(7, c); }
                    if (i == 11 && c == 7) { WAIT_V(2); progress(8); }
                }
            }
            need(6, wid);
#pragma unroll
            for (int i = 0; i < 16; ++i) {
                const unsigned o = xo + (unsigned)(i >> 1) * KT_A + (unsigned)rank * 65536u + (i & 1) * 1024;
                slot(i, 160 + i, o, o);
                if (i == 0) { WAIT_V(0); progress(9); STAMPW(wid, 56 + wid); STAMPW(7, 8); }
            }
            st16(d[1], rws, lo[1]);
            WAIT_V(0);
            progress(10);
            STAMPW(7, 9);
        }
        BAR();
    }

    const int wm = wid >> 1, wn = wid & 1;
    const int lvoff = wid * 1024 + voff;
    int fw0, fw1, tsrc;
    unsigned taba, tabb;
    {
        const int src = lane >> 3, c = lane & 7;
        fw0 = fw1 = orel == 0 && src != rank ? 64 + src : lane;
        tsrc = src * 17;
        taba = stg + (unsigned)lane * KT_A + (unsigned)(owner * 2 + hh) * STAGE_A;
        tabb = stg + B_REL + (unsigned)lane * KT_B + (unsigned)nt * STAGE_B;
        if (orel != 0 && src == owner) {
            fw0 = fw1 = rank * 8 + c;
            tsrc = rank * 17;
            taba = stg + (unsigned)(rank * 8 + c) * KT_A + (unsigned)(rank * 2 + hh) * STAGE_A;
            tabb = stg + W_REL + (unsigned)lane * KT_B + (unsigned)nt * STAGE_B;
        } else if (orel != 0 && c == 7 && src != rank) {
            const int t = (src - rank) & 7, k = t < orel ? t - 1 : t - 2, ka = k & ~1, kb = k | 1, h = k & 1;
            const int sa = (rank + (ka + 1 < orel ? ka + 1 : ka + 2)) & 7, sb2 = (rank + (kb + 1 < orel ? kb + 1 : kb + 2)) & 7;
            fw0 = h ? sa * 8 + 7 : 72 + sa; fw1 = h ? sb2 * 8 + 7 : 72 + sb2;
            tsrc = sa | sb2 << 4;
            taba += (unsigned)((((wid & 1) ? sb2 : sa) - src) * 8) * KT_A + (unsigned)((h - (wid & 1)) * 1024);
            tabb += (unsigned)((((wid >> 2) ? sb2 : sa) - src) * 8) * KT_B + (unsigned)((h - (wid >> 2)) * 4096);
        }
    }
    const unsigned* const myflags = (const unsigned*)(myws + FLG_OFF + xcd * FLG_REP);
    f4 acc[4][4], acc2[4][4];
#pragma unroll
    for (int m = 0; m < 4; ++m)
#pragma unroll
        for (int n = 0; n < 4; ++n) { acc[m][n] = (f4){0.f, 0.f, 0.f, 0.f}; acc2[m][n] = (f4){0.f, 0.f, 0.f, 0.f}; }

    auto stage1 = [&](unsigned ao, unsigned bo, int buf, int i) {
        LAS char* const l = (LAS char*)smem + buf * STAGE + wid * 1024 + i * 8192;
        __builtin_amdgcn_raw_ptr_buffer_load_lds(rws, l, 16, lvoff, i < 4 ? ao + i * 8192 : bo + (i - 4) * 8192, 0, AUX_LD);
    };
    auto stage = [&](int kt, int buf) {
        const unsigned ao = __builtin_amdgcn_readlane(taba, kt), bo = __builtin_amdgcn_readlane(tabb, kt);
#pragma unroll
        for (int i = 0; i < 6; ++i) stage1(ao, bo, buf, i);
    };
    auto frag = [&](h8 (&af)[4], h8 (&bf)[4], int buf, int ks, int g) {
        const LAS char* const l = (const LAS char*)smem + buf * STAGE + voff;
        if (g < 4) bf[g] = *(const LAS h8*)(l + STAGE_A + wn * 8192 + ks * 4096 + g * 1024);
        else af[g - 4] = *(const LAS h8*)(l + wm * 8192 + ks * 1024 + (g - 4) * 2048);
    };
    auto kstep = [&](f4 (&c)[4][4], const h8 (&af)[4], const h8 (&bf)[4], h8 (&an)[4], h8 (&bn)[4], int rbuf, int rks,
                     bool dma, unsigned ao, unsigned bo, int sbuf) {
#pragma unroll
        for (int g = 0; g < 8; ++g) {
            const int m = g >> 1, n0 = (g & 1) * 2;
            c[m][n0] = __builtin_amdgcn_mfma_f32_16x16x32_f16(af[m], bf[n0], c[m][n0], 0, 0, 0);
            if (g < 2) { frag(an, bn, rbuf, rks, 2 * g); frag(an, bn, rbuf, rks, 2 * g + 1); }
            else if (g < 6) frag(an, bn, rbuf, rks, g + 2);
            c[m][n0 + 1] = __builtin_amdgcn_mfma_f32_16x16x32_f16(af[m], bf[n0 + 1], c[m][n0 + 1], 0, 0, 0);
            if (dma && g >= 2) stage1(ao, bo, sbuf, g - 2);
            __builtin_amdgcn_sched_barrier(0);
        }
    };
    int buf = 0;
    unsigned long long done = 0, seen = 0;
    const unsigned* const pf0 = myflags + fw0;
    const unsigned* const pf1 = myflags + fw1;
    auto run = [&](f4 (&c)[4][4], unsigned long long kind) {
        unsigned long long avail = seen & kind & ~done;
        done |= avail;
        stage(__builtin_ctzll(avail), buf); avail &= avail - 1;
        int sbuf = buf + 1 == NSTAGE ? 0 : buf + 1, ahead = 0, par = 0;
        if (avail) { stage(__builtin_ctzll(avail), sbuf); avail &= avail - 1; sbuf = sbuf + 1 == NSTAGE ? 0 : sbuf + 1; ahead = 1; WAIT_V(6); }
        else {
            if (wid == 0) {
                __builtin_amdgcn_s_sleep(PRO_SLEEP);
                const unsigned v0 = __hip_atomic_load(pf0, __ATOMIC_RELAXED, __HIP_MEMORY_SCOPE_AGENT);
                const unsigned v1 = __hip_atomic_load(pf1, __ATOMIC_RELAXED, __HIP_MEMORY_SCOPE_AGENT);
                const unsigned long long m = __ballot(v0 == call2 && v1 == call2);
                if (lane < 2) ctl[14 + lane] = (unsigned)(m >> (lane * 32));
            }
            WAIT_V(0);
        }
        BAR();
        h8 a0[4], b0[4], a1[4], b1f[4];
#pragma unroll
        for (int g = 0; g < 8; ++g) frag(a0, b0, buf, 0, g);
        if (ahead == 0) {
            avail = (((unsigned long long)lds_ld(ctl + 15) << 32) | lds_ld(ctl + 14)) & kind & ~done;
            done |= avail;
        }
        bool more;
        do {
            unsigned v0 = 0, v1 = 0;
            if (wid == 0) asm volatile("global_load_dword %0, %2, off sc1\n\tglobal_load_dword %1, %3, off sc1" : "=&v"(v0), "=&v"(v1) : "v"(pf0), "v"(pf1) : "memory");
            if (ahead == 0 && avail) { stage(__builtin_ctzll(avail), sbuf); avail &= avail - 1; sbuf = sbuf + 1 == NSTAGE ? 0 : sbuf + 1; ahead = 1; }
            more = ahead != 0;
            const int kt = __builtin_ctzll(avail | (1ull << 63));
            const unsigned ao = avail ? __builtin_amdgcn_readlane(taba, kt) : OOB, bo = avail ? __builtin_amdgcn_readlane(tabb, kt) : OOB;
            const int b1 = buf + 1 == NSTAGE ? 0 : buf + 1;
            kstep(c, a0, b0, a1, b1f, buf, 1, true, ao, bo, sbuf);
            if (avail) { avail &= avail - 1; sbuf = sbuf + 1 == NSTAGE ? 0 : sbuf + 1; } else ahead = 0;
            WAIT_V(6);
            if (wid == 0) {
                asm volatile("" : "+v"(v0), "+v"(v1));
                const unsigned long long m = __ballot(v0 == call2 && v1 == call2);
                if (lane < 2) ctl[8 + par + lane] = (unsigned)(m >> (lane * 32));
            }
            BAR();
            const unsigned mlo = ctl[8 + par], mhi = ctl[9 + par];
            kstep(c, a1, b1f, a0, b0, b1, 0, false, 0, 0, 0);
            seen = ((unsigned long long)__builtin_amdgcn_readfirstlane(mhi) << 32) | (unsigned)__builtin_amdgcn_readfirstlane(mlo);
            if (more) { avail |= seen & kind & ~done; done |= avail; }
            par ^= 2;
            buf = b1;
        } while (more);
    };

    auto run_scaled = [&](f4 (&c)[4][4], unsigned long long todo, int jx, int jw) {
        done |= todo;
        while (todo) {
            const int kt = __builtin_ctzll(todo);
            todo &= todo - 1;
            stage(kt, buf);
            const int src2 = __builtin_amdgcn_readlane(tsrc, kt);
            int e[2];
#pragma unroll
            for (int ks = 0; ks < 2; ++ks) {
                const int* ep = (const int*)(p.ws[(src2 >> (ks * 4)) & 7] + expo);
                const int idx = (lane & 7) < 4 ? ((kt & 7) * 32 + hh * 16 + wm * 4 + (lane & 3)) * 8 + jx : ((8 + (kt & 7)) * 32 + nt * 2 + wn) * 8 + jw;
                e[ks] = __hip_atomic_load(ep + idx, __ATOMIC_RELAXED, __HIP_MEMORY_SCOPE_SYSTEM);
            }
            WAIT_V(0);
            BAR();
#pragma unroll
            for (int ks = 0; ks < 2; ++ks) {
                h8 af[4], bf[4];
#pragma unroll
                for (int g = 0; g < 8; ++g) frag(af, bf, buf, ks, g);
#pragma unroll
                for (int m = 0; m < 4; ++m) {
                    const int em = __builtin_amdgcn_readlane(e[ks], m) + __builtin_amdgcn_readlane(e[ks], 4);
#pragma unroll
                    for (int n = 0; n < 4; ++n) {
                        const f4 t = __builtin_amdgcn_mfma_f32_16x16x32_f16(af[m], bf[n], (f4){0.f, 0.f, 0.f, 0.f}, 0, 0, 0);
#pragma unroll
                        for (int r = 0; r < 4; ++r) c[m][n][r] += ldexpf(t[r], em);
                    }
                }
            }
            BAR();
        }
    };

    const unsigned long long locm = orel == 0 ? 0ull : 0xffull << (owner * 8);
    bool bad = false;
    int nbatch = 0;
    while (done != ~0ull) {
        unsigned long long scaled = 0;
        if (!(seen & ~done)) {
            if (wid == 0) {
                const unsigned long long t0 = __builtin_amdgcn_s_memrealtime();
                unsigned long long m, ms;
                for (unsigned it = 1;; ++it) {
                    const unsigned v0 = __hip_atomic_load(pf0, __ATOMIC_RELAXED, __HIP_MEMORY_SCOPE_AGENT);
                    const unsigned v1 = __hip_atomic_load(pf1, __ATOMIC_RELAXED, __HIP_MEMORY_SCOPE_AGENT);
                    m = __ballot(v0 == call2 && v1 == call2);
                    ms = __ballot((v0 | 1u) == call2 + 1u && (v1 | 1u) == call2 + 1u);
                    if ((ms & ~done) || ((it & 63u) == 0 && give_up(tmo, call, t0, lane))) break;
                    __builtin_amdgcn_s_sleep(POLL_SLEEP);
                }
                if (lane == 0) { ctl[16] = (unsigned)m; ctl[17] = (unsigned)(m >> 32); ctl[20] = (unsigned)ms; ctl[21] = (unsigned)(ms >> 32); }
            }
            BAR();
            seen = ((unsigned long long)lds_ld(ctl + 17) << 32) | lds_ld(ctl + 16);
            scaled = (((unsigned long long)lds_ld(ctl + 21) << 32) | lds_ld(ctl + 20)) & ~done;
            if (!scaled) { bad = true; done = ~0ull; }
            scaled &= ~seen;
        }
        if (seen & ~done & locm) run(acc2, locm);
        if (seen & ~done & ~locm) run(acc, ~locm);
        if (__builtin_expect((scaled & locm) != 0, 0)) run_scaled(acc2, scaled & locm, rank, owner);
        if (__builtin_expect((scaled & ~locm) != 0, 0)) run_scaled(acc, scaled & ~locm, owner, rank);
    }
    STAMP(10);

    const float qn = __builtin_nanf("");
    const __amdgpu_buffer_rsrc_t rout = mk_rsrc(p.out, OUT_BYTES);
    const unsigned oo = ((unsigned)(hh * 256 + wm * 64 + (lane >> 4) * 4) * NG + (unsigned)(owner * NS + nt * 128 + wn * 64 + (lane & 15) * 4)) * 4u;
    auto put = [&](int m) {
#pragma unroll
        for (int r = 0; r < 4; ++r) {
            f4 v = {acc[m][0][r] + acc2[m][0][r], acc[m][1][r] + acc2[m][1][r], acc[m][2][r] + acc2[m][2][r], acc[m][3][r] + acc2[m][3][r]};
            if (bad) v = (f4){qn, qn, qn, qn};
            st16(__builtin_bit_cast(u4, v), rout, oo + (unsigned)(m * 16 + r) * (NG * 4u));
        }
    };
    if (orel != 0) {
        const unsigned tile = (unsigned)(hh * 16 + nt);
        {
            if (wid > 0) pace<TAU_LD>(now32(), wid * 16 - LEAD);
            if (bad) {
#pragma unroll
                for (int q = 0; q < 16; ++q) acc[q >> 2][q & 3] = (f4){qn, qn, qn, qn};
            }
            const unsigned po = stg + P_REL + (((unsigned)owner * 32u + tile) * 8u + (unsigned)wid) * 16384u + voff;
#pragma unroll
            for (int q = 0; q < 16; ++q) st16(__builtin_bit_cast(u4, acc[q >> 2][q & 3]), rws, po + q * 1024);
            WAIT_V(0);
            if (lane == 0) __hip_atomic_store((unsigned*)(p.ws[owner] + PFL_OFF + ((unsigned)rank * 32u + tile) * 128u) + wid, call, __ATOMIC_RELAXED, __HIP_MEMORY_SCOPE_SYSTEM);
        }
        STAMP(11);
        {
            const unsigned* f = (const unsigned*)(myws + PFL_OFF + ((unsigned)owner * 32u + tile) * 128u) + wid;
            const unsigned long long t0 = __builtin_amdgcn_s_memrealtime();
            unsigned tv;
            for (unsigned it = 1;; ++it) {
                const unsigned v = __hip_atomic_load(f, __ATOMIC_RELAXED, __HIP_MEMORY_SCOPE_SYSTEM);
                const unsigned tm = __hip_atomic_load(tmo, __ATOMIC_RELAXED, __HIP_MEMORY_SCOPE_AGENT);
                tv = __hip_atomic_load((unsigned*)(myws + LNK_OFF) + owner * 32 + (lane & 31), __ATOMIC_RELAXED, __HIP_MEMORY_SCOPE_AGENT);
                if (__builtin_amdgcn_readfirstlane(tm) == call) { bad = true; break; }
                if ((int)(__builtin_amdgcn_readfirstlane(v) - call) >= 0) break;
                if ((it & 15u) == 0 && give_up(tmo, call, t0, lane)) { bad = true; break; }
                __builtin_amdgcn_s_sleep(1);
            }
            const unsigned tf = now32();
            if (tv - tf + 20000u > 40000u) tv = tf;
            const unsigned tl = __builtin_amdgcn_readlane(tv, tile);
            while (lds_ld(ctl + 12) < (unsigned)wid) __builtin_amdgcn_s_sleep(1);
            unsigned te = wid > 0 ? lds_ld(ctl + 13) : tl;
            if ((int)(tf - te - slots<TAU_LD>(wid * 16)) > 0) te = tf - slots<TAU_LD>(wid * 16);
            unsigned tlast = te + 30000u;
            if (wid == 7) {
                int dm = (int)(tv - tf);
#pragma unroll
                for (int o = 16; o > 0; o >>= 1) dm = min(dm, __shfl_xor(dm, o));
                tlast = tf + (unsigned)__builtin_amdgcn_readfirstlane(dm) + slots<TAU_LD>(128u - ENDB);
            }
            unsigned next = te;
            STAMP(12);
            const __amdgpu_buffer_rsrc_t ro = mk_rsrc(p.ws[owner], WS_NEED);
            const unsigned pi = stg + P_REL + (((unsigned)rank * 32u + tile) * 8u + (unsigned)wid) * 16384u + voff;
#pragma unroll
            for (int q = 0; q < 16; ++q) {
                const unsigned due = te + slots<TAU_LD>(wid * 16 + q);
                if ((int)(due - tlast) > 0) wait_until(tlast); else pace_soft(due, next, GAP_OUT);
                acc[q >> 2][q & 3] = __builtin_bit_cast(f4, ld16(ro, pi + q * 1024));
                if ((q & 3) == 1 && q > 4) WAIT_V(5); else WAIT_V(1);
                if ((q & 3) == 0 && q > 0) put((q >> 2) - 1);
            }
            if (lane == 0) {
                __hip_atomic_store(ctl + 13, te, __ATOMIC_RELAXED, __HIP_MEMORY_SCOPE_WORKGROUP);
                __hip_atomic_store(ctl + 12, (unsigned)wid + 1u, __ATOMIC_RELAXED, __HIP_MEMORY_SCOPE_WORKGROUP);
            }
            STAMPW(7, 13);
            put(3);
        }
    } else {
        if (!bad) bad = __builtin_amdgcn_readfirstlane(__hip_atomic_load(tmo, __ATOMIC_RELAXED, __HIP_MEMORY_SCOPE_AGENT)) == call;
#pragma unroll
        for (int m = 0; m < 4; ++m) put(m);
    }
}

}

extern "C" void kernel_launch(int rank, int nranks, ncclComm_t comm,
                              hipStream_t stream,
                              void* const* d_in, const int* in_sizes, int n_in,
                              void* d_out, int out_size,
                              const int* in_dtypes, int out_dtype,
                              void* const* d_out_all,
                              void* const* d_ws_all, size_t ws_bytes) {
    (void)comm; (void)in_dtypes; (void)out_dtype; (void)d_out_all;
    if (nranks != NR || (unsigned)rank >= (unsigned)NR || n_in < 2 || in_sizes[0] != MG * KS || in_sizes[1] != KS * NG
        || out_size != MS * NG || ws_bytes < WS_NEED) {
        fprintf(stderr, "kernel_launch: unexpected problem shape\n");
        return;
    }
    Params p;
    p.x = (const float*)d_in[0]; p.w = (const float*)d_in[1]; p.out = (float*)d_out; p.rank = rank;
    for (int i = 0; i < NR; ++i) p.ws[i] = (char*)d_ws_all[i];
    hipLaunchKernelGGL(gemm_rs_kernel, dim3(NWG), dim3(512), 0, stream, p);
    HIP_CHECK(hipGetLastError());
}
```

```cpp
#include <hip/hip_runtime.h>
#include <rccl/rccl.h>
#include <cstdio>

#define HIP_CHECK(expr) do { hipError_t _e = (expr); if (_e != hipSuccess) { \
    fprintf(stderr, "HIP error %s at %s:%d\n", hipGetErrorString(_e), __FILE__, __LINE__); \
    return; } } while (0)

namespace {

typedef unsigned u4 __attribute__((ext_vector_type(4)));
typedef float f4 __attribute__((ext_vector_type(4)));
typedef _Float16 h8 __attribute__((ext_vector_type(8)));
#define LAS __attribute__((address_space(3)))
#define GAS __attribute__((address_space(1)))
#define INL __attribute__((always_inline))

constexpr int NR = 8;
constexpr int MG = 4096, NG = 16384, KS = 512, NS = 2048, MS = 512;
constexpr int NWG = 256, NOWN = 32;
constexpr int STAGE_A = 32768, STAGE_B = 16384, STAGE = STAGE_A + STAGE_B, NSTAGE = 3;
constexpr int CTL = NSTAGE * STAGE;
constexpr int LDS_BYTES = CTL + 128;
constexpr unsigned TMO_OFF = 0;
constexpr unsigned LNK_OFF = 3072;
constexpr unsigned ORG_OFF = 1024;
constexpr unsigned SEQ_OFF = 4096;
constexpr unsigned AGG_OFF = 36864;
constexpr unsigned PFL_OFF = 65536;
constexpr unsigned FLG_OFF = 98304, FLG_REP = 4352;
constexpr unsigned RDY_OFF = 147456;
constexpr unsigned EXP_OFF = 196608;
constexpr unsigned STAMP_OFF = 262144;
constexpr unsigned STG_OFF = 1u << 20;
constexpr unsigned A_BYTES = 32u << 20;
constexpr unsigned B_BYTES = 16u << 20;
constexpr unsigned W_BYTES = 16u << 20;
constexpr unsigned P_BYTES = 32u << 20;
constexpr unsigned B_REL = A_BYTES, W_REL = B_REL + B_BYTES, P_REL = W_REL + W_BYTES, SLOT_BYTES = P_REL + P_BYTES;
constexpr unsigned WS_NEED = STG_OFF + 2 * SLOT_BYTES;
constexpr unsigned KT_A = 16u * STAGE_A, KT_B = 16u * STAGE_B;
constexpr unsigned OOB = 0x80000000u;
constexpr unsigned OUT_BYTES = (unsigned)MS * NG * 4;
constexpr unsigned long long DEADLINE = 100000000ull;
constexpr int AUX_SYS = 17;
constexpr int AUX_LD = 16;

struct Params {
    const float* x; char* myws; int rank, pad; const float* w;
    char* ws[NR]; float* out;
};

#define WAIT_V(n) asm volatile("s_waitcnt vmcnt(" #n ")" ::: "memory")
#define WAIT_VN(n) asm volatile("s_waitcnt vmcnt(%0)" :: "i"(n) : "memory")
#define GBS 58.0
#define GBS_OUT 61.0
#define T_START 400
#define CONV_TICKS 192
#define POLL_SLEEP 4
#define RDY_LAT 150
#define SLACK 60
#define PRO_SLEEP 12
#define ENDB 1
#define LOW_EXP 9
constexpr unsigned LOW_BITS = (127u - LOW_EXP) << 23;
#define GAP_IN 51
#define GAP_OUT 51
#define LEAD 12
#define GBS_IN 60.6
constexpr unsigned TAU_ST = (unsigned)(32768.0 / GBS / 10.0 * 256.0), TAU_IN = (unsigned)(32768.0 / GBS_IN / 10.0 * 256.0), TAU_LD = (unsigned)(32768.0 / GBS_OUT / 10.0 * 256.0);
#define BAR() do { asm volatile("" ::: "memory"); __builtin_amdgcn_s_waitcnt(0xC07F);   __builtin_amdgcn_s_barrier(); asm volatile("" ::: "memory"); } while (0)

#define STAMPW(w, i) do {} while (0)
#define STAMP(i) STAMPW(0, i)

__device__ __forceinline__ __amdgpu_buffer_rsrc_t mk_rsrc(const void* p, unsigned bytes) {
    return __builtin_amdgcn_make_buffer_rsrc((void*)p, 0, (int)bytes, 0x00020000);
}
__device__ __forceinline__ void st16(u4 v, __amdgpu_buffer_rsrc_t r, unsigned off) {
    __builtin_amdgcn_raw_buffer_store_b128(v, r, (int)off, 0, AUX_SYS);
}
__device__ __forceinline__ u4 ld16(__amdgpu_buffer_rsrc_t r, unsigned off) {
    return __builtin_amdgcn_raw_buffer_load_b128(r, (int)off, 0, AUX_SYS);
}
__device__ __forceinline__ unsigned now32() { return (unsigned)__builtin_amdgcn_s_memrealtime(); }
template <unsigned TAU> __device__ __forceinline__ unsigned slots(unsigned n) { return (n * TAU) >> 8; }
__device__ __forceinline__ void wait_until(unsigned due) {
    while ((int)(now32() - due) < 0) __builtin_amdgcn_s_sleep(1);
    asm volatile("" ::: "memory");
}
template <unsigned TAU> __device__ __forceinline__ void pace(unsigned t0, unsigned n) { wait_until(t0 + slots<TAU>(n)); }
__device__ __forceinline__ void pace_soft(unsigned due, unsigned& next, unsigned gap) {
    if ((int)(next - due) > 0) due = next;
    const unsigned now = now32();
    if ((int)(now - due) > 0) due = now; else wait_until(due);
    asm volatile("" ::: "memory");
    next = due + gap;
}
__device__ __forceinline__ unsigned lds_ld(LAS unsigned* p) {
    return __builtin_amdgcn_readfirstlane(__hip_atomic_load(p, __ATOMIC_RELAXED, __HIP_MEMORY_SCOPE_WORKGROUP));
}
__device__ __forceinline__ unsigned lds_add(LAS unsigned* p, unsigned v, int lane) {
    unsigned old = 0;
    if (lane == 0) old = __hip_atomic_fetch_add(p, v, __ATOMIC_RELAXED, __HIP_MEMORY_SCOPE_WORKGROUP);
    return __builtin_amdgcn_readfirstlane(old) + v;
}
__device__ __forceinline__ u4 cvt8(f4 a, f4 b) {
    h8 h = {(_Float16)a[0], (_Float16)a[1], (_Float16)a[2], (_Float16)a[3], (_Float16)b[0], (_Float16)b[1], (_Float16)b[2], (_Float16)b[3]};
    return __builtin_bit_cast(u4, h);
}
template <int N> __device__ __forceinline__ int block_scale(f4 (&v)[N]) {
    float m = 0.f;
#pragma unroll
    for (int i = 0; i < N; ++i) m = fmaxf(fmaxf(m, fmaxf(fabsf(v[i][0]), fabsf(v[i][1]))), fmaxf(fabsf(v[i][2]), fabsf(v[i][3])));
    int e = 0;
    if (__builtin_expect((__builtin_amdgcn_ballot_w64(m >= 32768.f) != 0) | (__builtin_amdgcn_ballot_w64(m >= __uint_as_float(LOW_BITS)) == 0), 0)) {
        unsigned mb = 0;
#pragma unroll
        for (int i = 0; i < N; ++i)
#pragma unroll
            for (int j = 0; j < 4; ++j) { const unsigned t = __float_as_uint(v[i][j]) & 0x7fffffffu; if (t < 0x7f800000u && t > mb) mb = t; }
#pragma unroll
        for (int o = 32; o > 0; o >>= 1) { const unsigned t = (unsigned)__shfl_xor((int)mb, o); mb = t > mb ? t : mb; }
        mb = __builtin_amdgcn_readfirstlane(mb);
        const int ex = mb >= 0x00800000u ? (int)(mb >> 23) - 127 : -118 - __builtin_clz(mb | 1u);
        e = mb == 0u || (mb >= LOW_BITS && mb < 0x47000000u) ? 0 : (ex - 6) & ~7;
#pragma unroll
        for (int i = 0; i < N; ++i)
#pragma unroll
            for (int j = 0; j < 4; ++j) v[i][j] = ldexpf(v[i][j], -e);
    }
    return e;
}
__device__ __forceinline__ bool give_up(unsigned* tmo, unsigned call, unsigned long long t0, int lane) {
    if (__builtin_amdgcn_readfirstlane(__hip_atomic_load(tmo, __ATOMIC_RELAXED, __HIP_MEMORY_SCOPE_AGENT)) == call
        || __builtin_amdgcn_s_memrealtime() - t0 > DEADLINE) {
        if (lane == 0) __hip_atomic_store(tmo, call, __ATOMIC_RELAXED, __HIP_MEMORY_SCOPE_AGENT);
        return true;
    }
    return false;
}

__global__ __launch_bounds__(512, 2) void gemm_rs_kernel(const Params p) {
    __shared__ __attribute__((aligned(16))) char smem[LDS_BYTES];
    const int tid = threadIdx.x, lane = tid & 63;
    const int wid = __builtin_amdgcn_readfirstlane(tid >> 6);
    const int b = blockIdx.x, rank = p.rank;
    char* const myws = p.myws;
    LAS unsigned* const ctl = (LAS unsigned*)((LAS char*)smem + CTL);
    STAMP(0);
    const unsigned long long tstart = __builtin_amdgcn_s_memrealtime();
    const int xcd = b & 7, slot = b >> 3;
    const int orel = slot >> 2;
    const int owner = (rank + orel) & 7;
    const int hh = xcd & 1, nt = (xcd >> 1) * 4 + (slot & 3);
    const __amdgpu_buffer_rsrc_t rws = mk_rsrc(myws, WS_NEED);
    const int voff = lane * 16;
    unsigned* const tmo = (unsigned*)(myws + TMO_OFF);

    const int sb = b & 31;
    unsigned long long* const torg = (unsigned long long*)(myws + ORG_OFF + sb * 64);
    const float* const xs = p.x + (size_t)((wid * 32 + sb) * 16 + (lane & 15)) * KS + (lane >> 4) * 8;
    const float* const wsrc = p.w + (size_t)((lane >> 4) * 8) * NG + wid * NS + sb * 64 + (lane & 15) * 4;
    u4 xh[2] = {}, wh[8] = {};
    f4 xr[4], wr[16];
    int xe = 0, we = 0;
    auto load_x = [&](int c) INL {
#pragma unroll
        for (int i = 0; i < 4; ++i) xr[i] = *(const f4*)(xs + c * 64 + (i >> 1) * 32 + (i & 1) * 4);
    };
    auto load_w = [&](int c) INL {
#pragma unroll
        for (int i = 0; i < 16; ++i) wr[i] = *(const f4*)(wsrc + (size_t)(c * 64 + (i >> 3) * 32 + (i & 7)) * NG);
    };
    auto cvt_x = [&]() INL { xe = block_scale(xr); xh[0] = cvt8(xr[0], xr[1]); xh[1] = cvt8(xr[2], xr[3]); };
    auto cvt_w = [&]() INL {
        we = block_scale(wr);
#pragma unroll
        for (int ks = 0; ks < 2; ++ks)
#pragma unroll
            for (int jj = 0; jj < 4; ++jj) {
                const f4 lo = {wr[ks * 8][jj], wr[ks * 8 + 1][jj], wr[ks * 8 + 2][jj], wr[ks * 8 + 3][jj]};
                const f4 hi = {wr[ks * 8 + 4][jj], wr[ks * 8 + 5][jj], wr[ks * 8 + 6][jj], wr[ks * 8 + 7][jj]};
                wh[ks * 4 + jj] = cvt8(lo, hi);
            }
    };
    const int uc = (b >> 5) == 7 ? 0 : b >> 5;
    const bool ux = b < 224;
    auto ticket = [&]() INL {
        if (wid == rank) {
            if (lane == 0) ctl[18] = __hip_atomic_fetch_add((unsigned*)(myws + SEQ_OFF + b * 128), 1u, __ATOMIC_RELAXED, __HIP_MEMORY_SCOPE_AGENT) + 1u;
            if (lane < 32 && lane != 18) ctl[lane] = 0;
        }
    };
    unsigned call, call2, stg, rdyo, expo, xoff;
    auto sync_call = [&]() INL {
        BAR();
        call = lds_ld(ctl + 18); call2 = 2u * call;
        stg = STG_OFF + (call & 1u) * SLOT_BYTES;
        rdyo = RDY_OFF + (call & 1u) * 16384u + (unsigned)sb * 512u; expo = EXP_OFF + (call & 1u) * 16384u;
        xoff = stg + (unsigned)(rank * 8) * KT_A + (unsigned)(wid * 32 + sb) * 2048u + voff;
    };

    auto arrive = [&](bool on, int fw, unsigned weight, bool scaled) INL {
        if (on) {
            unsigned long long* cnt = (unsigned long long*)(myws + AGG_OFF + fw * 128);
            const unsigned long long add = weight | ((unsigned long long)scaled << 32);
            const unsigned long long sum = __hip_atomic_fetch_add(cnt, add, __ATOMIC_RELAXED, __HIP_MEMORY_SCOPE_AGENT) + add;
            if (((unsigned)sum & 0x7fffffffu) == ((call * 64u) & 0x7fffffffu)) {
#pragma unroll
                for (int r = 0; r < 8; ++r)
                    __hip_atomic_store((unsigned*)(myws + FLG_OFF + r * FLG_REP) + fw, call2 + (unsigned)((sum >> 32) != 0), __ATOMIC_RELAXED, __HIP_MEMORY_SCOPE_AGENT);
                if (sum >> 31) __hip_atomic_fetch_and(cnt, 0x7fffffffull, __ATOMIC_RELAXED, __HIP_MEMORY_SCOPE_AGENT);
            }
        }
    };
    auto tell_peers = [&](int word, bool scaled) INL {
        char* w = p.ws[0];
#pragma unroll
        for (int i = 1; i < NR; ++i) if (lane == i) w = p.ws[i];
        if (lane < NR && lane != rank) __hip_atomic_store((GAS unsigned*)(w + rdyo) + word, call2 + (unsigned)scaled, __ATOMIC_RELAXED, __HIP_MEMORY_SCOPE_SYSTEM);
    };
    auto store_e = [&](int which, int c, int e) INL {
        if (lane == 0) __hip_atomic_store((int*)(myws + expo) + ((which * 8 + c) * 32 + sb) * 8 + wid, e, __ATOMIC_RELAXED, __HIP_MEMORY_SCOPE_SYSTEM);
    };
    auto store_x = [&](int c) INL { st16(xh[0], rws, xoff + c * KT_A); st16(xh[1], rws, xoff + c * KT_A + 1024); store_e(0, c, xe); };
    auto store_w = [&](int c) INL {
        const unsigned wo = stg + (wid == rank ? B_REL : W_REL) + (unsigned)(wid * 8 + c) * KT_B + (unsigned)sb * 8192u + voff;
#pragma unroll
        for (int q = 0; q < 8; ++q) st16(wh[q], rws, wo + q * 1024);
        store_e(1, c, we);
    };

    if (__builtin_expect(b >= NOWN, 0)) {
        if (b >= 64 && b < 224) {
            if (wid == ((rank + 1) & 7)) {
                const int j = (b >> 3) & 7;
                const __amdgpu_buffer_rsrc_t rj = b < 192 ? mk_rsrc(p.ws[j], WS_NEED) : mk_rsrc(p.x, (unsigned)MG * KS * 4);
                const unsigned par = STG_OFF + (lane & 1) * SLOT_BYTES, q = lane >> 1;
                unsigned o;
                if (b < 128) o = q == 0 ? par + (unsigned)(rank * 8) * KT_A : q == 1 ? par + (unsigned)(rank * 8 + 1) * KT_A - 4u : OOB;
                else if (b >= 192) o = lane < 2 ? (unsigned)(3 - j) * 0x200000u + lane * 0x1ffffcu : OOB;
                else if (j == rank) o = STG_OFF + (unsigned)lane * 0x200000u;
                else o = q < 2 ? par + W_REL + (unsigned)(rank * 8) * KT_B + q * 0x1ffffcu : q == 2 ? RDY_OFF : q < 6 ? par + (unsigned)(j * 8) * KT_A + (q - 3) * 0x1ffffcu
                       : q < 9 ? par + P_REL + (unsigned)rank * 0x400000u + (q - 6) * 0x1ffffcu : OOB;
                unsigned t = __builtin_amdgcn_raw_buffer_load_b32(rj, (int)o, 0, AUX_SYS);
                if (b >= 128 && b < 192 && j == rank) {
                    t += __builtin_amdgcn_raw_buffer_load_b32(rj, (int)(o + 64u * 0x200000u), 0, AUX_SYS);
                    t += __builtin_amdgcn_raw_buffer_load_b32(mk_rsrc(p.out, OUT_BYTES), lane * 0x1ffffc, 0, AUX_SYS);
                    t += __builtin_amdgcn_raw_buffer_load_b32(mk_rsrc(p.w, (unsigned)KS * NG * 4), lane * 0x1ffffc, 0, 0);
                }
                asm volatile("" :: "v"(t));
                STAMPW(wid, 56);
            }
            wait_until((unsigned)tstart + T_START + (unsigned)(b - 64) * CONV_TICKS / 16u);
        }
        ticket();
        if (ux) load_x(uc);
        load_w(uc);
        sync_call();
        if (ux) { cvt_x(); store_x(uc); }
        cvt_w(); store_w(uc);
        WAIT_V(0);
        unsigned sum = lds_add(ctl + 0, (xe | we) ? 257u : 1u, lane);
        if ((sum & 255u) == 8u) {
            arrive(lane == 0, rank * 8 + uc, ux ? 2u : 1u, sum > 255u);
            if (!ux) {
                unsigned long long o;
                bool late = false;
                while ((o = __hip_atomic_load(torg, __ATOMIC_RELAXED, __HIP_MEMORY_SCOPE_AGENT)) + 1000ull < tstart) {
                    if (give_up(tmo, call, tstart, lane)) { late = true; break; }
                    __builtin_amdgcn_s_sleep(4);
                }
                if (!late) wait_until(__builtin_amdgcn_readfirstlane((unsigned)o) + slots<TAU_ST>(12u) - RDY_LAT);
            }
            tell_peers(rank * 8 + uc, sum > 255u);
        }
        if (!ux) {
            wait_until((unsigned)tstart + T_START + (unsigned)(b - 64) * CONV_TICKS / 16u);
            load_x(7); load_w(7);
            cvt_x(); store_x(7); cvt_w(); store_w(7);
            WAIT_V(0);
            sum = lds_add(ctl + 1, (xe | we) ? 257u : 1u, lane);
            if ((sum & 255u) == 8u) { arrive(lane == 0, rank * 8 + 7, 2u, sum > 255u); tell_peers(rank * 8 + 7, sum > 255u); }
        }
        STAMP(15);
    } else {
        ticket();
        load_x(0);
        const int u = (wid - rank - 1) & 7;
        unsigned xg[6];
        char* pw[6];
#pragma unroll
        for (int s = 0; s < 6; ++s) { int a = u + 1 + s; if (a >= 7) a -= 7; xg[s] = (unsigned)(rank + 1 + a) & 7u; pw[s] = p.ws[xg[s]]; }
        asm volatile("" : "+s"(pw[0]), "+s"(pw[1]), "+s"(pw[2]), "+s"(pw[3]), "+s"(pw[4]), "+s"(pw[5]));
        sync_call();
        cvt_x();
        asm volatile("" : "+v"(xh[0]), "+v"(xh[1]));
        const unsigned long long t64 = __builtin_amdgcn_s_memrealtime();
        if (__builtin_expect(wid == rank, 0)) {
            store_x(0);
            WAIT_V(0);
            lds_add(ctl + 0, xe ? 257u : 1u, lane);
            const unsigned* f = (const unsigned*)(myws + rdyo);
            bool told = false, all = false;
            unsigned fx = 0, ann = lane < NR && lane != rank ? 0u : 10u;
            unsigned long long scl = 0;
            pace<TAU_ST>((unsigned)t64, 6);
            for (unsigned it = 1;; ++it) {
                if (!told) {
                    const unsigned sum = lds_ld(ctl + 0);
                    if ((sum & 255u) == 8u) { told = true; tell_peers(64 + rank, sum > 255u); arrive(lane == 0, rank * 8, 1u, sum > 255u); }
                }
                if (!all && (fx != 0xffu || (it & 7u) == 0)) {
                    const unsigned v = __hip_atomic_load(f + lane, __ATOMIC_RELAXED, __HIP_MEMORY_SCOPE_SYSTEM);
                    const unsigned vx = __hip_atomic_load(f + 64 + (lane >> 3), __ATOMIC_RELAXED, __HIP_MEMORY_SCOPE_SYSTEM);
                    unsigned long long full = __ballot((lane >> 3) == rank || (int)(v - call2) >= 0);
                    const unsigned long long fx8 = __ballot((lane >> 3) == rank || (int)(vx - call2) >= 0);
                    scl = __ballot(v == call2 + 1u || ((lane & 7) == 0 && vx == call2 + 1u));
                    fx = (unsigned)__ballot((fx8 >> ((lane & 7) * 8)) & 1u) & 0xffu;
                    if ((it & 31u) == 0 && give_up(tmo, call, tstart, lane)) { full = ~0ull; fx = 0xffu; }
                    if (lane < 3) __hip_atomic_store(ctl + 4 + lane, lane == 2 ? fx : (unsigned)(full >> (lane * 32)), __ATOMIC_RELAXED, __HIP_MEMORY_SCOPE_WORKGROUP);
                    all = full == ~0ull && fx == 0xffu;
                }
                const unsigned prog = __hip_atomic_load(ctl + 24 + (lane & 7), __ATOMIC_RELAXED, __HIP_MEMORY_SCOPE_WORKGROUP);
                const bool on = prog > ann && ((fx >> (lane & 7)) & 1u);
                const unsigned sc = (unsigned)(scl >> ((lane & 7) * 8)) & 0xffu;
                arrive(on, ann < 7u ? lane * 8 + (int)ann : ann == 7u ? 72 + lane : ann == 8u ? lane * 8 + 7 : 64 + lane, 2u,
                       ann < 9u ? (sc >> (ann < 7u ? ann : 7u)) & 1u : sc != 0u);
                if (on) ++ann;
                if (told && __all(ann == 10u)) break;
                __builtin_amdgcn_s_sleep(1);
            }
        } else {
            {
                __amdgpu_buffer_rsrc_t rd[6];
#pragma unroll
                for (int s = 0; s < 6; ++s) rd[s] = mk_rsrc(pw[s], WS_NEED);
#pragma unroll
                for (int i = 0; i < 12; ++i) {
                    if (i > 0) pace<TAU_ST>((unsigned)t64, i);
                    st16(xh[i & 1], rd[i >> 1], xoff + (i & 1) * 1024);
                    if (i == 0) {
                        store_x(0);
                        if (u == 0 && lane == 0) __hip_atomic_store(torg, t64, __ATOMIC_RELAXED, __HIP_MEMORY_SCOPE_AGENT);
                        STAMPW(7, 52);
                    } else WAIT_V(1);
                }
            }
            const __amdgpu_buffer_rsrc_t rs = mk_rsrc(p.ws[wid], WS_NEED);
            const unsigned xo = stg + (unsigned)(wid * 8) * KT_A + (unsigned)sb * 2048u + voff;
            const unsigned wro = stg + W_REL + (unsigned)(rank * 8) * KT_B + (unsigned)sb * 8192u + voff;
            const unsigned wlo = stg + B_REL + (unsigned)(wid * 8) * KT_B + (unsigned)sb * 8192u + voff;
            unsigned ts = 0;
            auto need = [&](int word, int bit) INL {
                while (!((lds_ld(ctl + word) >> bit) & 1u)) __builtin_amdgcn_s_sleep(1);
                asm volatile("" ::: "memory");
            };
            STAMPW(7, 53);
            need(4 + (wid >> 2), (wid & 3) * 8);
            STAMPW(7, 54);
            {
                unsigned long long t = __builtin_amdgcn_s_memrealtime();
                const unsigned long long tp = t64 + slots<TAU_ST>(12u);
                if (t < tp + SLACK) t = tp;
                ts = (unsigned)t - slots<TAU_IN>(12u);
                if (lane == 0) __hip_atomic_store((unsigned*)(myws + LNK_OFF) + wid * 32 + sb, (unsigned)t + slots<TAU_IN>(164u), __ATOMIC_RELAXED, __HIP_MEMORY_SCOPE_AGENT);
            }
            unsigned next = ts;
            u4 d[2] = {};
            unsigned lo[2] = {OOB, OOB};
            auto progress = [&](unsigned v) INL { if (lane == 0) __hip_atomic_store(ctl + 24 + wid, v, __ATOMIC_RELAXED, __HIP_MEMORY_SCOPE_WORKGROUP); };
            auto slot = [&](int i, unsigned n, unsigned ro, unsigned l) INL {
                pace_soft(ts + slots<TAU_IN>(n), next, GAP_IN);
                lo[i & 1] = l;
                d[i & 1] = ld16(rs, ro);
                asm volatile("" ::: "memory");
                st16(d[(i & 1) ^ 1], rws, lo[(i & 1) ^ 1]);
                asm volatile("" ::: "memory");
            };
#pragma unroll
            for (int i = 12; i < 20; ++i) {
                slot(i, i, wro + (i - 12) * 1024, wlo + (i - 12) * 1024);
                if (i == 12) { WAIT_V(2); lds_add(ctl + 0, xe ? 257u : 1u, lane); }
            }
            for (int c = 1; c < 8; ++c) {
                need(4 + (wid >> 2), (wid & 3) * 8 + c);
#pragma unroll
                for (int i = 0; i < 20; ++i) {
                    const int ks = i / 10, r = i % 10;
                    if (r < 6) { const unsigned o = xo + c * KT_A + xg[r] * 65536u + ks * 1024; slot(i, c * 20 + i, o, o); }
                    else slot(i, c * 20 + i, wro + c * KT_B + (ks * 4 + r - 6) * 1024, wlo + c * KT_B + (ks * 4 + r - 6) * 1024);
                    if (i == 1) { WAIT_V(2); progress(c); STAMPW(7, c); }
                    if (i == 11 && c == 7) { WAIT_V(2); progress(8); }
                }
            }
            need(6, wid);
#pragma unroll
            for (int i = 0; i < 16; ++i) {
                const unsigned o = xo + (unsigned)(i >> 1) * KT_A + (unsigned)rank * 65536u + (i & 1) * 1024;
                slot(i, 160 + i, o, o);
                if (i == 0) { WAIT_V(0); progress(9); STAMPW(wid, 56 + wid); STAMPW(7, 8); }
            }
            st16(d[1], rws, lo[1]);
            WAIT_V(0);
            progress(10);
            STAMPW(7, 9);
        }
        BAR();
    }

    const int wm = wid >> 1, wn = wid & 1;
    const int lvoff = wid * 1024 + voff;
    int fw0, fw1, tsrc;
    unsigned taba, tabb;
    {
        const int src = lane >> 3, c = lane & 7;
        fw0 = fw1 = orel == 0 && src != rank ? 64 + src : lane;
        tsrc = src * 17;
        taba = stg + (unsigned)lane * KT_A + (unsigned)(owner * 2 + hh) * STAGE_A;
        tabb = stg + B_REL + (unsigned)lane * KT_B + (unsigned)nt * STAGE_B;
        if (orel != 0 && src == owner) {
            fw0 = fw1 = rank * 8 + c;
            tsrc = rank * 17;
            taba = stg + (unsigned)(rank * 8 + c) * KT_A + (unsigned)(rank * 2 + hh) * STAGE_A;
            tabb = stg + W_REL + (unsigned)lane * KT_B + (unsigned)nt * STAGE_B;
        } else if (orel != 0 && c == 7 && src != rank) {
            const int t = (src - rank) & 7, k = t < orel ? t - 1 : t - 2, ka = k & ~1, kb = k | 1, h = k & 1;
            const int sa = (rank + (ka + 1 < orel ? ka + 1 : ka + 2)) & 7, sb2 = (rank + (kb + 1 < orel ? kb + 1 : kb + 2)) & 7;
            fw0 = h ? sa * 8 + 7 : 72 + sa; fw1 = h ? sb2 * 8 + 7 : 72 + sb2;
            tsrc = sa | sb2 << 4;
            taba += (unsigned)((((wid & 1) ? sb2 : sa) - src) * 8) * KT_A + (unsigned)((h - (wid & 1)) * 1024);
            tabb += (unsigned)((((wid >> 2) ? sb2 : sa) - src) * 8) * KT_B + (unsigned)((h - (wid >> 2)) * 4096);
        }
    }
    const unsigned* const myflags = (const unsigned*)(myws + FLG_OFF + xcd * FLG_REP);
    f4 acc[4][4], acc2[4][4];
#pragma unroll
    for (int m = 0; m < 4; ++m)
#pragma unroll
        for (int n = 0; n < 4; ++n) { acc[m][n] = (f4){0.f, 0.f, 0.f, 0.f}; acc2[m][n] = (f4){0.f, 0.f, 0.f, 0.f}; }

    auto stage1 = [&](unsigned ao, unsigned bo, int buf, int i) INL {
        LAS char* const l = (LAS char*)smem + buf * STAGE + wid * 1024 + i * 8192;
        __builtin_amdgcn_raw_ptr_buffer_load_lds(rws, l, 16, lvoff, i < 4 ? ao + i * 8192 : bo + (i - 4) * 8192, 0, AUX_LD);
    };
    auto stage = [&](int kt, int buf) INL {
        const unsigned ao = __builtin_amdgcn_readlane(taba, kt), bo = __builtin_amdgcn_readlane(tabb, kt);
#pragma unroll
        for (int i = 0; i < 6; ++i) stage1(ao, bo, buf, i);
    };
    auto frag = [&](h8 (&af)[4], h8 (&bf)[4], int buf, int ks, int g) INL {
        const LAS char* const l = (const LAS char*)smem + buf * STAGE + voff;
        if (g < 4) bf[g] = *(const LAS h8*)(l + STAGE_A + wn * 8192 + ks * 4096 + g * 1024);
        else af[g - 4] = *(const LAS h8*)(l + wm * 8192 + ks * 1024 + (g - 4) * 2048);
    };
    auto kstep = [&](f4 (&c)[4][4], const h8 (&af)[4], const h8 (&bf)[4], h8 (&an)[4], h8 (&bn)[4], int rbuf, int rks,
                     bool dma, unsigned ao, unsigned bo, int sbuf) INL {
#pragma unroll
        for (int g = 0; g < 8; ++g) {
            const int m = g >> 1, n0 = (g & 1) * 2;
            c[m][n0] = __builtin_amdgcn_mfma_f32_16x16x32_f16(af[m], bf[n0], c[m][n0], 0, 0, 0);
            if (g < 2) { frag(an, bn, rbuf, rks, 2 * g); frag(an, bn, rbuf, rks, 2 * g + 1); }
            else if (g < 6) frag(an, bn, rbuf, rks, g + 2);
            c[m][n0 + 1] = __builtin_amdgcn_mfma_f32_16x16x32_f16(af[m], bf[n0 + 1], c[m][n0 + 1], 0, 0, 0);
            if (dma && g >= 2) stage1(ao, bo, sbuf, g - 2);
            __builtin_amdgcn_sched_barrier(0);
        }
    };
    int buf = 0;
    unsigned long long done = 0, seen = 0;
    const unsigned* const pf0 = myflags + fw0;
    const unsigned* const pf1 = myflags + fw1;
    auto run = [&](f4 (&c)[4][4], unsigned long long kind) INL {
        unsigned long long avail = seen & kind & ~done;
        done |= avail;
        stage(__builtin_ctzll(avail), buf); avail &= avail - 1;
        int sbuf = buf + 1 == NSTAGE ? 0 : buf + 1, ahead = 0, par = 0;
        if (avail) { stage(__builtin_ctzll(avail), sbuf); avail &= avail - 1; sbuf = sbuf + 1 == NSTAGE ? 0 : sbuf + 1; ahead = 1; WAIT_V(6); }
        else {
            if (wid == 0) {
                __builtin_amdgcn_s_sleep(PRO_SLEEP);
                const unsigned v0 = __hip_atomic_load(pf0, __ATOMIC_RELAXED, __HIP_MEMORY_SCOPE_AGENT);
                const unsigned v1 = __hip_atomic_load(pf1, __ATOMIC_RELAXED, __HIP_MEMORY_SCOPE_AGENT);
                const unsigned long long m = __ballot(v0 == call2 && v1 == call2);
                if (lane < 2) ctl[14 + lane] = (unsigned)(m >> (lane * 32));
            }
            WAIT_V(0);
        }
        BAR();
        h8 a0[4], b0[4], a1[4], b1f[4];
#pragma unroll
        for (int g = 0; g < 8; ++g) frag(a0, b0, buf, 0, g);
        if (ahead == 0) {
            avail = (((unsigned long long)lds_ld(ctl + 15) << 32) | lds_ld(ctl + 14)) & kind & ~done;
            done |= avail;
        }
        bool more;
        do {
            unsigned v0 = 0, v1 = 0;
            if (wid == 0) asm volatile("global_load_dword %0, %2, off sc1\n\tglobal_load_dword %1, %3, off sc1" : "=&v"(v0), "=&v"(v1) : "v"(pf0), "v"(pf1) : "memory");
            if (ahead == 0 && avail) { stage(__builtin_ctzll(avail), sbuf); avail &= avail - 1; sbuf = sbuf + 1 == NSTAGE ? 0 : sbuf + 1; ahead = 1; }
            more = ahead != 0;
            const int kt = __builtin_ctzll(avail | (1ull << 63));
            const unsigned ao = avail ? __builtin_amdgcn_readlane(taba, kt) : OOB, bo = avail ? __builtin_amdgcn_readlane(tabb, kt) : OOB;
            const int b1 = buf + 1 == NSTAGE ? 0 : buf + 1;
            kstep(c, a0, b0, a1, b1f, buf, 1, true, ao, bo, sbuf);
            if (avail) { avail &= avail - 1; sbuf = sbuf + 1 == NSTAGE ? 0 : sbuf + 1; } else ahead = 0;
            WAIT_V(6);
            if (wid == 0) {
                asm volatile("" : "+v"(v0), "+v"(v1));
                const unsigned long long m = __ballot(v0 == call2 && v1 == call2);
                if (lane < 2) ctl[8 + par + lane] = (unsigned)(m >> (lane * 32));
            }
            BAR();
            const unsigned mlo = ctl[8 + par], mhi = ctl[9 + par];
            kstep(c, a1, b1f, a0, b0, b1, 0, false, 0, 0, 0);
            seen = ((unsigned long long)__builtin_amdgcn_readfirstlane(mhi) << 32) | (unsigned)__builtin_amdgcn_readfirstlane(mlo);
            if (more) { avail |= seen & kind & ~done; done |= avail; }
            par ^= 2;
            buf = b1;
        } while (more);
    };

    auto run_scaled = [&](f4 (&c)[4][4], unsigned long long todo, int jx, int jw) INL {
        done |= todo;
        while (todo) {
            const int kt = __builtin_ctzll(todo);
            todo &= todo - 1;
            stage(kt, buf);
            const int src2 = __builtin_amdgcn_readlane(tsrc, kt);
            int e[2];
#pragma unroll
            for (int ks = 0; ks < 2; ++ks) {
                const int* ep = (const int*)(p.ws[(src2 >> (ks * 4)) & 7] + expo);
                const int idx = (lane & 7) < 4 ? ((kt & 7) * 32 + hh * 16 + wm * 4 + (lane & 3)) * 8 + jx : ((8 + (kt & 7)) * 32 + nt * 2 + wn) * 8 + jw;
                e[ks] = __hip_atomic_load(ep + idx, __ATOMIC_RELAXED, __HIP_MEMORY_SCOPE_SYSTEM);
            }
            WAIT_V(0);
            BAR();
#pragma unroll
            for (int ks = 0; ks < 2; ++ks) {
                h8 af[4], bf[4];
#pragma unroll
                for (int g = 0; g < 8; ++g) frag(af, bf, buf, ks, g);
#pragma unroll
                for (int m = 0; m < 4; ++m) {
                    const int em = __builtin_amdgcn_readlane(e[ks], m) + __builtin_amdgcn_readlane(e[ks], 4);
#pragma unroll
                    for (int n = 0; n < 4; ++n) {
                        const f4 t = __builtin_amdgcn_mfma_f32_16x16x32_f16(af[m], bf[n], (f4){0.f, 0.f, 0.f, 0.f}, 0, 0, 0);
#pragma unroll
                        for (int r = 0; r < 4; ++r) c[m][n][r] += ldexpf(t[r], em);
                    }
                }
            }
            BAR();
        }
    };

    const unsigned long long locm = orel == 0 ? 0ull : 0xffull << (owner * 8);
    bool bad = false;
    int nbatch = 0;
    while (done != ~0ull) {
        unsigned long long scaled = 0;
        if (!(seen & ~done)) {
            if (wid == 0) {
                const unsigned long long t0 = __builtin_amdgcn_s_memrealtime();
                unsigned long long m, ms;
                for (unsigned it = 1;; ++it) {
                    const unsigned v0 = __hip_atomic_load(pf0, __ATOMIC_RELAXED, __HIP_MEMORY_SCOPE_AGENT);
                    const unsigned v1 = __hip_atomic_load(pf1, __ATOMIC_RELAXED, __HIP_MEMORY_SCOPE_AGENT);
                    m = __ballot(v0 == call2 && v1 == call2);
                    ms = __ballot((v0 | 1u) == call2 + 1u && (v1 | 1u) == call2 + 1u);
                    if ((ms & ~done) || ((it & 63u) == 0 && give_up(tmo, call, t0, lane))) break;
                    __builtin_amdgcn_s_sleep(POLL_SLEEP);
                }
                if (lane == 0) { ctl[16] = (unsigned)m; ctl[17] = (unsigned)(m >> 32); ctl[20] = (unsigned)ms; ctl[21] = (unsigned)(ms >> 32); }
            }
            BAR();
            seen = ((unsigned long long)lds_ld(ctl + 17) << 32) | lds_ld(ctl + 16);
            scaled = (((unsigned long long)lds_ld(ctl + 21) << 32) | lds_ld(ctl + 20)) & ~done;
            if (!scaled) { bad = true; done = ~0ull; }
            scaled &= ~seen;
        }
        if (seen & ~done & locm) run(acc2, locm);
        if (seen & ~done & ~locm) run(acc, ~locm);
        if (__builtin_expect((scaled & locm) != 0, 0)) run_scaled(acc2, scaled & locm, rank, owner);
        if (__builtin_expect((scaled & ~locm) != 0, 0)) run_scaled(acc, scaled & ~locm, owner, rank);
    }
    STAMP(10);

    const float qn = __builtin_nanf("");
    const __amdgpu_buffer_rsrc_t rout = mk_rsrc(p.out, OUT_BYTES);
    const unsigned oo = ((unsigned)(hh * 256 + wm * 64 + (lane >> 4) * 4) * NG + (unsigned)(owner * NS + nt * 128 + wn * 64 + (lane & 15) * 4)) * 4u;
    auto put = [&](int m) INL {
#pragma unroll
        for (int r = 0; r < 4; ++r) {
            f4 v = {acc[m][0][r] + acc2[m][0][r], acc[m][1][r] + acc2[m][1][r], acc[m][2][r] + acc2[m][2][r], acc[m][3][r] + acc2[m][3][r]};
            if (bad) v = (f4){qn, qn, qn, qn};
            st16(__builtin_bit_cast(u4, v), rout, oo + (unsigned)(m * 16 + r) * (NG * 4u));
        }
    };
    if (orel != 0) {
        const unsigned tile = (unsigned)(hh * 16 + nt);
        {
            if (wid > 0) pace<TAU_LD>(now32(), wid * 16 - LEAD);
            if (bad) {
#pragma unroll
                for (int q = 0; q < 16; ++q) acc[q >> 2][q & 3] = (f4){qn, qn, qn, qn};
            }
            const unsigned po = stg + P_REL + (((unsigned)owner * 32u + tile) * 8u + (unsigned)wid) * 16384u + voff;
#pragma unroll
            for (int q = 0; q < 16; ++q) st16(__builtin_bit_cast(u4, acc[q >> 2][q & 3]), rws, po + q * 1024);
            WAIT_V(0);
            if (lane == 0) __hip_atomic_store((unsigned*)(p.ws[owner] + PFL_OFF + ((unsigned)rank * 32u + tile) * 128u) + wid, call, __ATOMIC_RELAXED, __HIP_MEMORY_SCOPE_SYSTEM);
        }
        STAMP(11);
        {
            const unsigned* f = (const unsigned*)(myws + PFL_OFF + ((unsigned)owner * 32u + tile) * 128u) + wid;
            const unsigned long long t0 = __builtin_amdgcn_s_memrealtime();
            unsigned tv;
            for (unsigned it = 1;; ++it) {
                const unsigned v = __hip_atomic_load(f, __ATOMIC_RELAXED, __HIP_MEMORY_SCOPE_SYSTEM);
                const unsigned tm = __hip_atomic_load(tmo, __ATOMIC_RELAXED, __HIP_MEMORY_SCOPE_AGENT);
                tv = __hip_atomic_load((unsigned*)(myws + LNK_OFF) + owner * 32 + (lane & 31), __ATOMIC_RELAXED, __HIP_MEMORY_SCOPE_AGENT);
                if (__builtin_amdgcn_readfirstlane(tm) == call) { bad = true; break; }
                if ((int)(__builtin_amdgcn_readfirstlane(v) - call) >= 0) break;
                if ((it & 15u) == 0 && give_up(tmo, call, t0, lane)) { bad = true; break; }
                __builtin_amdgcn_s_sleep(1);
            }
            const unsigned tf = now32();
            if (tv - tf + 20000u > 40000u) tv = tf;
            const unsigned tl = __builtin_amdgcn_readlane(tv, tile);
            while (lds_ld(ctl + 12) < (unsigned)wid) __builtin_amdgcn_s_sleep(1);
            unsigned te = wid > 0 ? lds_ld(ctl + 13) : tl;
            if ((int)(tf - te - slots<TAU_LD>(wid * 16)) > 0) te = tf - slots<TAU_LD>(wid * 16);
            unsigned tlast = te + 30000u;
            if (wid == 7) {
                int dm = (int)(tv - tf);
#pragma unroll
                for (int o = 16; o > 0; o >>= 1) dm = min(dm, __shfl_xor(dm, o));
                tlast = tf + (unsigned)__builtin_amdgcn_readfirstlane(dm) + slots<TAU_LD>(128u - ENDB);
            }
            unsigned next = te;
            STAMP(12);
            const __amdgpu_buffer_rsrc_t ro = mk_rsrc(p.ws[owner], WS_NEED);
            const unsigned pi = stg + P_REL + (((unsigned)rank * 32u + tile) * 8u + (unsigned)wid) * 16384u + voff;
#pragma unroll
            for (int q = 0; q < 16; ++q) {
                const unsigned due = te + slots<TAU_LD>(wid * 16 + q);
                if ((int)(due - tlast) > 0) wait_until(tlast); else pace_soft(due, next, GAP_OUT);
                acc[q >> 2][q & 3] = __builtin_bit_cast(f4, ld16(ro, pi + q * 1024));
                if ((q & 3) == 1 && q > 4) WAIT_V(5); else WAIT_V(1);
                if ((q & 3) == 0 && q > 0) put((q >> 2) - 1);
            }
            if (lane == 0) {
                __hip_atomic_store(ctl + 13, te, __ATOMIC_RELAXED, __HIP_MEMORY_SCOPE_WORKGROUP);
                __hip_atomic_store(ctl + 12, (unsigned)wid + 1u, __ATOMIC_RELAXED, __HIP_MEMORY_SCOPE_WORKGROUP);
            }
            STAMPW(7, 13);
            put(3);
        }
    } else {
        if (!bad) bad = __builtin_amdgcn_readfirstlane(__hip_atomic_load(tmo, __ATOMIC_RELAXED, __HIP_MEMORY_SCOPE_AGENT)) == call;
#pragma unroll
        for (int m = 0; m < 4; ++m) put(m);
    }
}

}

extern "C" void kernel_launch(int rank, int nranks, ncclComm_t comm,
                              hipStream_t stream,
                              void* const* d_in, const int* in_sizes, int n_in,
                              void* d_out, int out_size,
                              const int* in_dtypes, int out_dtype,
                              void* const* d_out_all,
                              void* const* d_ws_all, size_t ws_bytes) {
    (void)comm; (void)in_dtypes; (void)out_dtype; (void)d_out_all;
    if (nranks != NR || (unsigned)rank >= (unsigned)NR || n_in < 2 || in_sizes[0] != MG * KS || in_sizes[1] != KS * NG
        || out_size != MS * NG || ws_bytes < WS_NEED) {
        fprintf(stderr, "kernel_launch: unexpected problem shape\n");
        return;
    }
    Params p;
    p.x = (const float*)d_in[0]; p.w = (const float*)d_in[1]; p.out = (float*)d_out; p.rank = rank; p.pad = 0; p.myws = (char*)d_ws_all[rank];
    for (int i = 0; i < NR; ++i) p.ws[i] = (char*)d_ws_all[i];
    hipLaunchKernelGGL(gemm_rs_kernel, dim3(NWG), dim3(512), 0, stream, p);
    HIP_CHECK(hipGetLastError());
}
```

```cpp
#include <hip/hip_runtime.h>
#include <rccl/rccl.h>
#include <cstdio>

#define HIP_CHECK(expr) do { hipError_t _e = (expr); if (_e != hipSuccess) { \
    fprintf(stderr, "HIP error %s at %s:%d\n", hipGetErrorString(_e), __FILE__, __LINE__); \
    return; } } while (0)

namespace {

typedef unsigned u4 __attribute__((ext_vector_type(4)));
typedef float f4 __attribute__((ext_vector_type(4)));
typedef _Float16 h8 __attribute__((ext_vector_type(8)));
#define LAS __attribute__((address_space(3)))
#define GAS __attribute__((address_space(1)))
#define INL __attribute__((always_inline))

constexpr int NR = 8;
constexpr int MG = 4096, NG = 16384, KS = 512, NS = 2048, MS = 512;
constexpr int NWG = 256, NOWN = 32;
constexpr int STAGE_A = 32768, STAGE_B = 16384, STAGE = STAGE_A + STAGE_B, NSTAGE = 3;
constexpr int CTL = NSTAGE * STAGE;
constexpr int LDS_BYTES = CTL + 128;
constexpr unsigned TMO_OFF = 0;
constexpr unsigned LNK_OFF = 3072;
constexpr unsigned ORG_OFF = 1024;
constexpr unsigned SEQ_OFF = 4096;
constexpr unsigned AGG_OFF = 36864;
constexpr unsigned PFL_OFF = 65536;
constexpr unsigned FLG_OFF = 98304, FLG_REP = 4352;
constexpr unsigned RDY_OFF = 147456;
constexpr unsigned EXP_OFF = 196608;
constexpr unsigned STG_OFF = 1u << 20;
constexpr unsigned A_BYTES = 32u << 20;
constexpr unsigned B_BYTES = 16u << 20;
constexpr unsigned W_BYTES = 16u << 20;
constexpr unsigned P_BYTES = 32u << 20;
constexpr unsigned B_REL = A_BYTES, W_REL = B_REL + B_BYTES, P_REL = W_REL + W_BYTES, SLOT_BYTES = P_REL + P_BYTES;
constexpr unsigned WS_NEED = STG_OFF + 2 * SLOT_BYTES;
constexpr unsigned KT_A = 16u * STAGE_A, KT_B = 16u * STAGE_B;
constexpr unsigned OOB = 0x80000000u;
constexpr unsigned OUT_BYTES = (unsigned)MS * NG * 4;
constexpr unsigned long long DEADLINE = 100000000ull;
constexpr int AUX_SYS = 17;
constexpr int AUX_LD = 16;

struct Params {
    const float* x; char* myws; int rank, pad; const float* w;
    char* ws[NR]; float* out;
};

#define WAIT_V(n) asm volatile("s_waitcnt vmcnt(" #n ")" ::: "memory")
constexpr unsigned tau(double gbs) { return (unsigned)(32768.0 / gbs / 10.0 * 256.0); }
constexpr unsigned TAU_ST = tau(58.0), TAU_IN = tau(60.8), TAU_LD = tau(61.0);
constexpr unsigned GAP_IN = 52, GAP_OUT = 51;
constexpr unsigned RDY_LAT = 220, SLACK = 60;
constexpr unsigned T_START = 400, CONV_TICKS = 192;
constexpr int LEAD = 8;
constexpr unsigned LOW_BITS = (127u - 9u) << 23;
#define BAR() do { asm volatile("" ::: "memory"); __builtin_amdgcn_s_waitcnt(0xC07F);   __builtin_amdgcn_s_barrier(); asm volatile("" ::: "memory"); } while (0)

__device__ __forceinline__ __amdgpu_buffer_rsrc_t mk_rsrc(const void* p, unsigned bytes) {
    return __builtin_amdgcn_make_buffer_rsrc((void*)p, 0, (int)bytes, 0x00020000);
}
__device__ __forceinline__ void st16(u4 v, __amdgpu_buffer_rsrc_t r, unsigned off) {
    __builtin_amdgcn_raw_buffer_store_b128(v, r, (int)off, 0, AUX_SYS);
}
__device__ __forceinline__ u4 ld16(__amdgpu_buffer_rsrc_t r, unsigned off) {
    return __builtin_amdgcn_raw_buffer_load_b128(r, (int)off, 0, AUX_SYS);
}
__device__ __forceinline__ unsigned now32() { return (unsigned)__builtin_amdgcn_s_memrealtime(); }
template <unsigned TAU> __device__ __forceinline__ unsigned slots(unsigned n) { return (n * TAU) >> 8; }
__device__ __forceinline__ void wait_until(unsigned due) {
    while ((int)(now32() - due) < 0) __builtin_amdgcn_s_sleep(1);
    asm volatile("" ::: "memory");
}
template <unsigned TAU> __device__ __forceinline__ void pace(unsigned t0, unsigned n) { wait_until(t0 + slots<TAU>(n)); }
__device__ __forceinline__ void pace_soft(unsigned due, unsigned& next, unsigned gap) {
    if ((int)(next - due) > 0) due = next;
    const unsigned now = now32();
    if ((int)(now - due) > 0) due = now; else wait_until(due);
    asm volatile("" ::: "memory");
    next = due + gap;
}
__device__ __forceinline__ unsigned lds_ld(LAS unsigned* p) {
    return __builtin_amdgcn_readfirstlane(__hip_atomic_load(p, __ATOMIC_RELAXED, __HIP_MEMORY_SCOPE_WORKGROUP));
}
__device__ __forceinline__ unsigned lds_add(LAS unsigned* p, unsigned v, int lane) {
    unsigned old = 0;
    if (lane == 0) old = __hip_atomic_fetch_add(p, v, __ATOMIC_RELAXED, __HIP_MEMORY_SCOPE_WORKGROUP);
    return __builtin_amdgcn_readfirstlane(old) + v;
}
__device__ __forceinline__ u4 cvt8(f4 a, f4 b) {
    h8 h = {(_Float16)a[0], (_Float16)a[1], (_Float16)a[2], (_Float16)a[3], (_Float16)b[0], (_Float16)b[1], (_Float16)b[2], (_Float16)b[3]};
    return __builtin_bit_cast(u4, h);
}
template <int N> __device__ __forceinline__ int block_scale(f4 (&v)[N]) {
    float m = 0.f;
#pragma unroll
    for (int i = 0; i < N; ++i) m = fmaxf(fmaxf(m, fmaxf(fabsf(v[i][0]), fabsf(v[i][1]))), fmaxf(fabsf(v[i][2]), fabsf(v[i][3])));
    int e = 0;
    if (__builtin_expect((__builtin_amdgcn_ballot_w64(m >= 32768.f) != 0) | (__builtin_amdgcn_ballot_w64(m >= __uint_as_float(LOW_BITS)) == 0), 0)) {
        unsigned mb = 0;
#pragma unroll
        for (int i = 0; i < N; ++i)
#pragma unroll
            for (int j = 0; j < 4; ++j) { const unsigned t = __float_as_uint(v[i][j]) & 0x7fffffffu; if (t < 0x7f800000u && t > mb) mb = t; }
#pragma unroll
        for (int o = 32; o > 0; o >>= 1) { const unsigned t = (unsigned)__shfl_xor((int)mb, o); mb = t > mb ? t : mb; }
        mb = __builtin_amdgcn_readfirstlane(mb);
        const int ex = mb >= 0x00800000u ? (int)(mb >> 23) - 127 : -118 - __builtin_clz(mb | 1u);
        e = mb == 0u || (mb >= LOW_BITS && mb < 0x47000000u) ? 0 : (ex - 6) & ~7;
#pragma unroll
        for (int i = 0; i < N; ++i)
#pragma unroll
            for (int j = 0; j < 4; ++j) v[i][j] = ldexpf(v[i][j], -e);
    }
    return e;
}
__device__ __forceinline__ bool give_up(unsigned* tmo, unsigned call, unsigned long long t0, int lane) {
    if (__builtin_amdgcn_readfirstlane(__hip_atomic_load(tmo, __ATOMIC_RELAXED, __HIP_MEMORY_SCOPE_AGENT)) == call
        || __builtin_amdgcn_s_memrealtime() - t0 > DEADLINE) {
        if (lane == 0) __hip_atomic_store(tmo, call, __ATOMIC_RELAXED, __HIP_MEMORY_SCOPE_AGENT);
        return true;
    }
    return false;
}

__global__ __launch_bounds__(512, 2) void gemm_rs_kernel(const Params p) {
    __shared__ __attribute__((aligned(16))) char smem[LDS_BYTES];
    const int tid = threadIdx.x, lane = tid & 63;
    const int wid = __builtin_amdgcn_readfirstlane(tid >> 6);
    const int b = blockIdx.x, rank = p.rank;
    char* const myws = p.myws;
    LAS unsigned* const ctl = (LAS unsigned*)((LAS char*)smem + CTL);
    const unsigned long long tstart = __builtin_amdgcn_s_memrealtime();
    const int xcd = b & 7, slot = b >> 3;
    const int orel = slot >> 2;
    const int owner = (rank + orel) & 7;
    const int hh = xcd & 1, nt = (xcd >> 1) * 4 + (slot & 3);
    const __amdgpu_buffer_rsrc_t rws = mk_rsrc(myws, WS_NEED);
    const int voff = lane * 16;
    unsigned* const tmo = (unsigned*)(myws + TMO_OFF);

    const int sb = b & 31;
    unsigned long long* const torg = (unsigned long long*)(myws + ORG_OFF + sb * 64);
    const float* const xs = p.x + (size_t)((wid * 32 + sb) * 16 + (lane >> 2)) * KS + ((lane & 3) ^ ((lane >> 5) << 1)) * 4;
    const float* const wsrc = p.w + (size_t)((lane >> 4) * 4) * NG + wid * NS + sb * 64 + (lane & 15) * 4;
    u4 xh[2] = {}, wh[8] = {};
    f4 xr[4], wr[16];
    int xe = 0, we = 0;
    auto load_x = [&](int c) INL {
#pragma unroll
        for (int i = 0; i < 4; ++i) xr[i] = *(const f4*)(xs + c * 64 + i * 16);
    };
    auto load_w = [&](int c) INL {
#pragma unroll
        for (int i = 0; i < 16; ++i) wr[i] = *(const f4*)(wsrc + (size_t)(c * 64 + (i >> 2) * 16 + (i & 3)) * NG);
    };
    auto cvt_x = [&]() INL { xe = block_scale(xr); xh[0] = cvt8(xr[0], xr[1]); xh[1] = cvt8(xr[2], xr[3]); };
    auto cvt_w = [&]() INL {
        we = block_scale(wr);
#pragma unroll
        for (int ks = 0; ks < 2; ++ks)
#pragma unroll
            for (int jj = 0; jj < 4; ++jj) {
                const f4 lo = {wr[ks * 8][jj], wr[ks * 8 + 1][jj], wr[ks * 8 + 2][jj], wr[ks * 8 + 3][jj]};
                const f4 hi = {wr[ks * 8 + 4][jj], wr[ks * 8 + 5][jj], wr[ks * 8 + 6][jj], wr[ks * 8 + 7][jj]};
                wh[ks * 4 + jj] = cvt8(lo, hi);
            }
    };
    const int uc = (b >> 5) == 7 ? 0 : b >> 5;
    const bool ux = b < 224;
    auto ticket = [&]() INL {
        if (wid == rank) {
            if (lane == 0) ctl[18] = __hip_atomic_fetch_add((unsigned*)(myws + SEQ_OFF + b * 128), 1u, __ATOMIC_RELAXED, __HIP_MEMORY_SCOPE_AGENT) + 1u;
            if (lane < 32 && lane != 18) ctl[lane] = 0;
        }
    };
    unsigned call, call2, stg, rdyo, expo, xoff;
    auto sync_call = [&]() INL {
        BAR();
        call = lds_ld(ctl + 18); call2 = 2u * call;
        stg = STG_OFF + (call & 1u) * SLOT_BYTES;
        rdyo = RDY_OFF + (call & 1u) * 16384u + (unsigned)sb * 512u; expo = EXP_OFF + (call & 1u) * 16384u;
        xoff = stg + (unsigned)(rank * 8) * KT_A + (unsigned)(wid * 32 + sb) * 2048u + voff;
    };

    auto arrive = [&](bool on, int fw, unsigned weight, bool scaled) INL {
        if (on) {
            unsigned long long* cnt = (unsigned long long*)(myws + AGG_OFF + fw * 128);
            const unsigned long long add = weight | ((unsigned long long)scaled << 32);
            const unsigned long long sum = __hip_atomic_fetch_add(cnt, add, __ATOMIC_RELAXED, __HIP_MEMORY_SCOPE_AGENT) + add;
            if (((unsigned)sum & 0x7fffffffu) == ((call * 64u) & 0x7fffffffu)) {
#pragma unroll
                for (int r = 0; r < 8; ++r)
                    __hip_atomic_store((unsigned*)(myws + FLG_OFF + r * FLG_REP) + fw, call2 + (unsigned)((sum >> 32) != 0), __ATOMIC_RELAXED, __HIP_MEMORY_SCOPE_AGENT);
                if (sum >> 31) __hip_atomic_fetch_and(cnt, 0x7fffffffull, __ATOMIC_RELAXED, __HIP_MEMORY_SCOPE_AGENT);
            }
        }
    };
    auto tell_peers = [&](int word, bool scaled) INL {
        char* w = p.ws[0];
#pragma unroll
        for (int i = 1; i < NR; ++i) if (lane == i) w = p.ws[i];
        if (lane < NR && lane != rank) __hip_atomic_store((GAS unsigned*)(w + rdyo) + word, call2 + (unsigned)scaled, __ATOMIC_RELAXED, __HIP_MEMORY_SCOPE_SYSTEM);
    };
    auto store_e = [&](int which, int c, int e) INL {
        if (lane == 0) __hip_atomic_store((int*)(myws + expo) + ((which * 8 + c) * 32 + sb) * 8 + wid, e, __ATOMIC_RELAXED, __HIP_MEMORY_SCOPE_SYSTEM);
    };
    auto store_x = [&](int c) INL { st16(xh[0], rws, xoff + c * KT_A); st16(xh[1], rws, xoff + c * KT_A + 1024); store_e(0, c, xe); };
    auto store_w = [&](int c) INL {
        const unsigned wo = stg + (wid == rank ? B_REL : W_REL) + (unsigned)(wid * 8 + c) * KT_B + (unsigned)sb * 8192u + voff;
#pragma unroll
        for (int q = 0; q < 8; ++q) st16(wh[q], rws, wo + q * 1024);
        store_e(1, c, we);
    };

    if (__builtin_expect(b >= NOWN, 0)) {
        if (b >= 64 && b < 224) {
            if (wid == ((rank + 1) & 7)) {
                const int j = (b >> 3) & 7;
                const __amdgpu_buffer_rsrc_t rj = b < 192 ? mk_rsrc(p.ws[j], WS_NEED) : mk_rsrc(p.x, (unsigned)MG * KS * 4);
                const unsigned par = STG_OFF + (lane & 1) * SLOT_BYTES, q = lane >> 1;
                unsigned o;
                if (b < 128) o = q == 0 ? par + (unsigned)(rank * 8) * KT_A : q == 1 ? par + (unsigned)(rank * 8 + 1) * KT_A - 4u : OOB;
                else if (b >= 192) o = lane < 2 ? (unsigned)(3 - j) * 0x200000u + lane * 0x1ffffcu : OOB;
                else if (j == rank) o = STG_OFF + (unsigned)lane * 0x200000u;
                else o = q < 2 ? par + W_REL + (unsigned)(rank * 8) * KT_B + q * 0x1ffffcu : q == 2 ? RDY_OFF : q < 6 ? par + (unsigned)(j * 8) * KT_A + (q - 3) * 0x1ffffcu
                       : q < 9 ? par + P_REL + (unsigned)rank * 0x400000u + (q - 6) * 0x1ffffcu : OOB;
                unsigned t = __builtin_amdgcn_raw_buffer_load_b32(rj, (int)o, 0, AUX_SYS);
                if (b >= 128 && b < 192 && j == rank) {
                    t += __builtin_amdgcn_raw_buffer_load_b32(rj, (int)(o + 64u * 0x200000u), 0, AUX_SYS);
                    t += __builtin_amdgcn_raw_buffer_load_b32(mk_rsrc(p.out, OUT_BYTES), lane * 0x1ffffc, 0, AUX_SYS);
                    t += __builtin_amdgcn_raw_buffer_load_b32(mk_rsrc(p.w, (unsigned)KS * NG * 4), lane * 0x1ffffc, 0, 0);
                }
                asm volatile("" :: "v"(t));
            }
            wait_until((unsigned)tstart + T_START + (unsigned)(b - 64) * CONV_TICKS / 16u);
        }
        ticket();
        if (ux) load_x(uc);
        load_w(uc);
        sync_call();
        if (ux) { cvt_x(); store_x(uc); }
        cvt_w(); store_w(uc);
        WAIT_V(0);
        unsigned sum = lds_add(ctl + 0, (xe | we) ? 257u : 1u, lane);
        if ((sum & 255u) == 8u) {
            arrive(lane == 0, rank * 8 + uc, ux ? 2u : 1u, sum > 255u);
            if (!ux) {
                unsigned long long o;
                bool late = false;
                while ((unsigned)((o = __hip_atomic_load(torg, __ATOMIC_RELAXED, __HIP_MEMORY_SCOPE_AGENT)) >> 32) != call) {
                    if (give_up(tmo, call, tstart, lane)) { late = true; break; }
                    __builtin_amdgcn_s_sleep(4);
                }
                if (!late) wait_until(__builtin_amdgcn_readfirstlane((unsigned)o) + slots<TAU_ST>(12u) - RDY_LAT);
            }
            tell_peers(rank * 8 + uc, sum > 255u);
        }
        if (!ux) {
            wait_until((unsigned)tstart + T_START + (unsigned)(b - 64) * CONV_TICKS / 16u);
            load_x(7); load_w(7);
            cvt_x(); store_x(7); cvt_w(); store_w(7);
            WAIT_V(0);
            sum = lds_add(ctl + 1, (xe | we) ? 257u : 1u, lane);
            if ((sum & 255u) == 8u) { arrive(lane == 0, rank * 8 + 7, 2u, sum > 255u); tell_peers(rank * 8 + 7, sum > 255u); }
        }
    } else {
        ticket();
        load_x(0);
        const int u = (wid - rank - 1) & 7;
        unsigned xg[6];
        char* pw[6];
#pragma unroll
        for (int s = 0; s < 6; ++s) { int a = u + 1 + s; if (a >= 7) a -= 7; xg[s] = (unsigned)(rank + 1 + a) & 7u; pw[s] = p.ws[xg[s]]; }
        asm volatile("" : "+s"(pw[0]), "+s"(pw[1]), "+s"(pw[2]), "+s"(pw[3]), "+s"(pw[4]), "+s"(pw[5]));
        sync_call();
        cvt_x();
        asm volatile("" : "+v"(xh[0]), "+v"(xh[1]));
        const unsigned long long t64 = __builtin_amdgcn_s_memrealtime();
        if (__builtin_expect(wid == rank, 0)) {
            store_x(0);
            WAIT_V(0);
            lds_add(ctl + 0, xe ? 257u : 1u, lane);
            const unsigned* f = (const unsigned*)(myws + rdyo);
            bool told = false, all = false;
            unsigned fx = 0, ann = lane < NR && lane != rank ? 0u : 10u;
            unsigned long long scl = 0;
            pace<TAU_ST>((unsigned)t64, 6);
            for (unsigned it = 1;; ++it) {
                if (!told) {
                    const unsigned sum = lds_ld(ctl + 0);
                    if ((sum & 255u) == 8u) { told = true; tell_peers(64 + rank, sum > 255u); arrive(lane == 0, rank * 8, 1u, sum > 255u); }
                }
                if (!all && (fx != 0xffu || (it & 7u) == 0)) {
                    const unsigned v = __hip_atomic_load(f + lane, __ATOMIC_RELAXED, __HIP_MEMORY_SCOPE_SYSTEM);
                    const unsigned vx = __hip_atomic_load(f + 64 + (lane >> 3), __ATOMIC_RELAXED, __HIP_MEMORY_SCOPE_SYSTEM);
                    unsigned long long full = __ballot((lane >> 3) == rank || (int)(v - call2) >= 0);
                    const unsigned long long fx8 = __ballot((lane >> 3) == rank || (int)(vx - call2) >= 0);
                    scl = __ballot(v == call2 + 1u || ((lane & 7) == 0 && vx == call2 + 1u));
                    fx = (unsigned)__ballot((fx8 >> ((lane & 7) * 8)) & 1u) & 0xffu;
                    if ((it & 31u) == 0 && give_up(tmo, call, tstart, lane)) { full = ~0ull; fx = 0xffu; }
                    if (lane < 3) __hip_atomic_store(ctl + 4 + lane, lane == 2 ? fx : (unsigned)(full >> (lane * 32)), __ATOMIC_RELAXED, __HIP_MEMORY_SCOPE_WORKGROUP);
                    all = full == ~0ull && fx == 0xffu;
                }
                const unsigned prog = __hip_atomic_load(ctl + 24 + (lane & 7), __ATOMIC_RELAXED, __HIP_MEMORY_SCOPE_WORKGROUP);
                const bool on = prog > ann && ((fx >> (lane & 7)) & 1u);
                const unsigned sc = (unsigned)(scl >> ((lane & 7) * 8)) & 0xffu;
                arrive(on, ann < 7u ? lane * 8 + (int)ann : ann == 7u ? 72 + lane : ann == 8u ? lane * 8 + 7 : 64 + lane, 2u,
                       ann < 9u ? (sc >> (ann < 7u ? ann : 7u)) & 1u : sc != 0u);
                if (on) ++ann;
                if (told && __all(ann == 10u)) break;
                __builtin_amdgcn_s_sleep(1);
            }
        } else {
            {
                __amdgpu_buffer_rsrc_t rd[6];
#pragma unroll
                for (int s = 0; s < 6; ++s) rd[s] = mk_rsrc(pw[s], WS_NEED);
#pragma unroll
                for (int i = 0; i < 12; ++i) {
                    if (i > 0) pace<TAU_ST>((unsigned)t64, i);
                    st16(xh[i & 1], rd[i >> 1], xoff + (i & 1) * 1024);
                    if (i == 0) {
                        store_x(0);
                        if (u == 0 && lane == 0) __hip_atomic_store(torg, (unsigned long long)call << 32 | (unsigned)t64, __ATOMIC_RELAXED, __HIP_MEMORY_SCOPE_AGENT);
                    } else WAIT_V(1);
                }
            }
            const __amdgpu_buffer_rsrc_t rs = mk_rsrc(p.ws[wid], WS_NEED);
            const unsigned xo = stg + (unsigned)(wid * 8) * KT_A + (unsigned)sb * 2048u + voff;
            const unsigned wro = stg + W_REL + (unsigned)(rank * 8) * KT_B + (unsigned)sb * 8192u + voff;
            const unsigned wlo = stg + B_REL + (unsigned)(wid * 8) * KT_B + (unsigned)sb * 8192u + voff;
            unsigned ts = 0;
            auto need = [&](int word, int bit) INL {
                while (!((lds_ld(ctl + word) >> bit) & 1u)) __builtin_amdgcn_s_sleep(1);
                asm volatile("" ::: "memory");
            };
            need(4 + (wid >> 2), (wid & 3) * 8);
            {
                unsigned long long t = __builtin_amdgcn_s_memrealtime();
                const unsigned long long tp = t64 + slots<TAU_ST>(12u);
                if (t < tp + SLACK) t = tp;
                ts = (unsigned)t - slots<TAU_IN>(12u);
                if (lane == 0) __hip_atomic_store((unsigned*)(myws + LNK_OFF) + wid * 32 + sb, (unsigned)t + slots<TAU_IN>(164u), __ATOMIC_RELAXED, __HIP_MEMORY_SCOPE_AGENT);
            }
            unsigned next = ts;
            u4 d[2] = {};
            unsigned lo[2] = {OOB, OOB};
            auto progress = [&](unsigned v) INL { if (lane == 0) __hip_atomic_store(ctl + 24 + wid, v, __ATOMIC_RELAXED, __HIP_MEMORY_SCOPE_WORKGROUP); };
            auto slot = [&](int i, unsigned n, unsigned ro, unsigned l) INL {
                pace_soft(ts + slots<TAU_IN>(n), next, GAP_IN);
                lo[i & 1] = l;
                d[i & 1] = ld16(rs, ro);
                asm volatile("" ::: "memory");
                st16(d[(i & 1) ^ 1], rws, lo[(i & 1) ^ 1]);
                asm volatile("" ::: "memory");
            };
#pragma unroll
            for (int i = 12; i < 20; ++i) {
                slot(i, i, wro + (i - 12) * 1024, wlo + (i - 12) * 1024);
                if (i == 12) { WAIT_V(2); lds_add(ctl + 0, xe ? 257u : 1u, lane); }
            }
            for (int c = 1; c < 8; ++c) {
                need(4 + (wid >> 2), (wid & 3) * 8 + c);
#pragma unroll
                for (int i = 0; i < 20; ++i) {
                    const int ks = i / 10, r = i % 10;
                    if (r < 6) { const unsigned o = xo + c * KT_A + xg[r] * 65536u + ks * 1024; slot(i, c * 20 + i, o, o); }
                    else slot(i, c * 20 + i, wro + c * KT_B + (ks * 4 + r - 6) * 1024, wlo + c * KT_B + (ks * 4 + r - 6) * 1024);
                    if (i == 1) { WAIT_V(2); progress(c); }
                    if (i == 11 && c == 7) { WAIT_V(2); progress(8); }
                }
            }
            need(6, wid);
#pragma unroll
            for (int i = 0; i < 16; ++i) {
                const unsigned o = xo + (unsigned)(i >> 1) * KT_A + (unsigned)rank * 65536u + (i & 1) * 1024;
                slot(i, 160 + i, o, o);
                if (i == 0) { WAIT_V(0); progress(9); }
            }
            st16(d[1], rws, lo[1]);
            WAIT_V(0);
            progress(10);
        }
        BAR();
    }

    const int wm = wid >> 1, wn = wid & 1;
    const int lvoff = wid * 1024 + voff;
    const int aoff = (lane & 15) * 64 + (((lane >> 4) ^ ((lane >> 2) & 2)) << 4);
    int fw0, fw1, tsrc;
    unsigned taba, tabb;
    {
        const int src = lane >> 3, c = lane & 7;
        fw0 = fw1 = orel == 0 && src != rank ? 64 + src : lane;
        tsrc = src * 17;
        taba = stg + (unsigned)lane * KT_A + (unsigned)(owner * 2 + hh) * STAGE_A;
        tabb = stg + B_REL + (unsigned)lane * KT_B + (unsigned)nt * STAGE_B;
        if (orel != 0 && src == owner) {
            fw0 = fw1 = rank * 8 + c;
            tsrc = rank * 17;
            taba = stg + (unsigned)(rank * 8 + c) * KT_A + (unsigned)(rank * 2 + hh) * STAGE_A;
            tabb = stg + W_REL + (unsigned)lane * KT_B + (unsigned)nt * STAGE_B;
        } else if (orel != 0 && c == 7 && src != rank) {
            const int t = (src - rank) & 7, k = t < orel ? t - 1 : t - 2, ka = k & ~1, kb = k | 1, h = k & 1;
            const int sa = (rank + (ka + 1 < orel ? ka + 1 : ka + 2)) & 7, sb2 = (rank + (kb + 1 < orel ? kb + 1 : kb + 2)) & 7;
            fw0 = h ? sa * 8 + 7 : 72 + sa; fw1 = h ? sb2 * 8 + 7 : 72 + sb2;
            tsrc = sa | sb2 << 4;
            taba += (unsigned)((((wid & 1) ? sb2 : sa) - src) * 8) * KT_A + (unsigned)((h - (wid & 1)) * 1024);
            tabb += (unsigned)((((wid >> 2) ? sb2 : sa) - src) * 8) * KT_B + (unsigned)((h - (wid >> 2)) * 4096);
        }
    }
    const unsigned* const myflags = (const unsigned*)(myws + FLG_OFF + xcd * FLG_REP);
    f4 acc[4][4], acc2[4][4];
#pragma unroll
    for (int m = 0; m < 4; ++m)
#pragma unroll
        for (int n = 0; n < 4; ++n) { acc[m][n] = (f4){0.f, 0.f, 0.f, 0.f}; acc2[m][n] = (f4){0.f, 0.f, 0.f, 0.f}; }

    auto stage1 = [&](unsigned ao, unsigned bo, int buf, int i) INL {
        LAS char* const l = (LAS char*)smem + buf * STAGE + wid * 1024 + i * 8192;
        __builtin_amdgcn_raw_ptr_buffer_load_lds(rws, l, 16, lvoff, i < 4 ? ao + i * 8192 : bo + (i - 4) * 8192, 0, AUX_LD);
    };
    auto stage = [&](int kt, int buf) INL {
        const unsigned ao = __builtin_amdgcn_readlane(taba, kt), bo = __builtin_amdgcn_readlane(tabb, kt);
#pragma unroll
        for (int i = 0; i < 6; ++i) stage1(ao, bo, buf, i);
    };
    auto frag = [&](h8 (&af)[4], h8 (&bf)[4], int buf, int ks, int g) INL {
        const LAS char* const l = (const LAS char*)smem + buf * STAGE;
        if (g < 4) bf[g] = *(const LAS h8*)(l + STAGE_A + wn * 8192 + ks * 4096 + g * 1024 + voff);
        else af[g - 4] = *(const LAS h8*)(l + wm * 8192 + ks * 1024 + (g - 4) * 2048 + aoff);
    };
    auto kstep = [&](f4 (&c)[4][4], const h8 (&af)[4], const h8 (&bf)[4], h8 (&an)[4], h8 (&bn)[4], int rbuf, int rks,
                     bool dma, unsigned ao, unsigned bo, int sbuf) INL {
#pragma unroll
        for (int g = 0; g < 8; ++g) {
            const int m = g >> 1, n0 = (g & 1) * 2;
            c[m][n0] = __builtin_amdgcn_mfma_f32_16x16x32_f16(af[m], bf[n0], c[m][n0], 0, 0, 0);
            if (g < 2) { frag(an, bn, rbuf, rks, 2 * g); frag(an, bn, rbuf, rks, 2 * g + 1); }
            else if (g < 6) frag(an, bn, rbuf, rks, g + 2);
            c[m][n0 + 1] = __builtin_amdgcn_mfma_f32_16x16x32_f16(af[m], bf[n0 + 1], c[m][n0 + 1], 0, 0, 0);
            if (dma && g >= 2) stage1(ao, bo, sbuf, g - 2);
            __builtin_amdgcn_sched_barrier(0);
        }
    };
    int buf = 0;
    unsigned long long done = 0, seen = 0;
    const unsigned* const pf0 = myflags + fw0;
    const unsigned* const pf1 = myflags + fw1;
    auto run = [&](f4 (&c)[4][4], unsigned long long kind) INL {
        unsigned long long avail = seen & kind & ~done;
        done |= avail;
        stage(__builtin_ctzll(avail), buf); avail &= avail - 1;
        int sbuf = buf + 1 == NSTAGE ? 0 : buf + 1, ahead = 0, par = 0;
        if (avail) { stage(__builtin_ctzll(avail), sbuf); avail &= avail - 1; sbuf = sbuf + 1 == NSTAGE ? 0 : sbuf + 1; ahead = 1; WAIT_V(6); }
        else {
            if (wid == 0) {
                __builtin_amdgcn_s_sleep(12);
                const unsigned v0 = __hip_atomic_load(pf0, __ATOMIC_RELAXED, __HIP_MEMORY_SCOPE_AGENT);
                const unsigned v1 = __hip_atomic_load(pf1, __ATOMIC_RELAXED, __HIP_MEMORY_SCOPE_AGENT);
                const unsigned long long m = __ballot(v0 == call2 && v1 == call2);
                if (lane < 2) ctl[14 + lane] = (unsigned)(m >> (lane * 32));
            }
            WAIT_V(0);
        }
        BAR();
        h8 a0[4], b0[4], a1[4], b1f[4];
#pragma unroll
        for (int g = 0; g < 8; ++g) frag(a0, b0, buf, 0, g);
        if (ahead == 0) {
            avail = (((unsigned long long)lds_ld(ctl + 15) << 32) | lds_ld(ctl + 14)) & kind & ~done;
            done |= avail;
        }
        bool more;
        do {
            unsigned v0 = 0, v1 = 0;
            if (wid == 0) asm volatile("global_load_dword %0, %2, off sc1\n\tglobal_load_dword %1, %3, off sc1" : "=&v"(v0), "=&v"(v1) : "v"(pf0), "v"(pf1) : "memory");
            if (ahead == 0 && avail) { stage(__builtin_ctzll(avail), sbuf); avail &= avail - 1; sbuf = sbuf + 1 == NSTAGE ? 0 : sbuf + 1; ahead = 1; }
            more = ahead != 0;
            const int kt = __builtin_ctzll(avail | (1ull << 63));
            const unsigned ao = avail ? __builtin_amdgcn_readlane(taba, kt) : OOB, bo = avail ? __builtin_amdgcn_readlane(tabb, kt) : OOB;
            const int b1 = buf + 1 == NSTAGE ? 0 : buf + 1;
            kstep(c, a0, b0, a1, b1f, buf, 1, true, ao, bo, sbuf);
            if (avail) { avail &= avail - 1; sbuf = sbuf + 1 == NSTAGE ? 0 : sbuf + 1; } else ahead = 0;
            WAIT_V(6);
            if (wid == 0) {
                asm volatile("" : "+v"(v0), "+v"(v1));
                const unsigned long long m = __ballot(v0 == call2 && v1 == call2);
                if (lane < 2) ctl[8 + par + lane] = (unsigned)(m >> (lane * 32));
            }
            BAR();
            const unsigned mlo = ctl[8 + par], mhi = ctl[9 + par];
            kstep(c, a1, b1f, a0, b0, b1, 0, false, 0, 0, 0);
            seen = ((unsigned long long)__builtin_amdgcn_readfirstlane(mhi) << 32) | (unsigned)__builtin_amdgcn_readfirstlane(mlo);
            if (more) { avail |= seen & kind & ~done; done |= avail; }
            par ^= 2;
            buf = b1;
        } while (more);
    };

    auto run_scaled = [&](f4 (&c)[4][4], unsigned long long todo, int jx, int jw) INL {
        done |= todo;
        while (todo) {
            const int kt = __builtin_ctzll(todo);
            todo &= todo - 1;
            stage(kt, buf);
            const int src2 = __builtin_amdgcn_readlane(tsrc, kt);
            int e[2];
#pragma unroll
            for (int ks = 0; ks < 2; ++ks) {
                const int* ep = (const int*)(p.ws[(src2 >> (ks * 4)) & 7] + expo);
                const int idx = (lane & 7) < 4 ? ((kt & 7) * 32 + hh * 16 + wm * 4 + (lane & 3)) * 8 + jx : ((8 + (kt & 7)) * 32 + nt * 2 + wn) * 8 + jw;
                e[ks] = __hip_atomic_load(ep + idx, __ATOMIC_RELAXED, __HIP_MEMORY_SCOPE_SYSTEM);
            }
            WAIT_V(0);
            BAR();
#pragma unroll
            for (int ks = 0; ks < 2; ++ks) {
                h8 af[4], bf[4];
#pragma unroll
                for (int g = 0; g < 8; ++g) frag(af, bf, buf, ks, g);
#pragma unroll
                for (int m = 0; m < 4; ++m) {
                    const int em = __builtin_amdgcn_readlane(e[ks], m) + __builtin_amdgcn_readlane(e[ks], 4);
#pragma unroll
                    for (int n = 0; n < 4; ++n) {
                        const f4 t = __builtin_amdgcn_mfma_f32_16x16x32_f16(af[m], bf[n], (f4){0.f, 0.f, 0.f, 0.f}, 0, 0, 0);
#pragma unroll
                        for (int r = 0; r < 4; ++r) c[m][n][r] += ldexpf(t[r], em);
                    }
                }
            }
            BAR();
        }
    };

    const unsigned long long locm = orel == 0 ? 0ull : 0xffull << (owner * 8);
    bool bad = false;
    while (done != ~0ull) {
        unsigned long long scaled = 0;
        if (!(seen & ~done)) {
            if (wid == 0) {
                const unsigned long long t0 = __builtin_amdgcn_s_memrealtime();
                unsigned long long m, ms;
                for (unsigned it = 1;; ++it) {
                    const unsigned v0 = __hip_atomic_load(pf0, __ATOMIC_RELAXED, __HIP_MEMORY_SCOPE_AGENT);
                    const unsigned v1 = __hip_atomic_load(pf1, __ATOMIC_RELAXED, __HIP_MEMORY_SCOPE_AGENT);
                    m = __ballot(v0 == call2 && v1 == call2);
                    ms = __ballot((v0 | 1u) == call2 + 1u && (v1 | 1u) == call2 + 1u);
                    if ((ms & ~done) || ((it & 63u) == 0 && give_up(tmo, call, t0, lane))) break;
                    __builtin_amdgcn_s_sleep(4);
                }
                if (lane == 0) { ctl[16] = (unsigned)m; ctl[17] = (unsigned)(m >> 32); ctl[20] = (unsigned)ms; ctl[21] = (unsigned)(ms >> 32); }
            }
            BAR();
            seen = ((unsigned long long)lds_ld(ctl + 17) << 32) | lds_ld(ctl + 16);
            scaled = (((unsigned long long)lds_ld(ctl + 21) << 32) | lds_ld(ctl + 20)) & ~done;
            if (!scaled) { bad = true; done = ~0ull; }
            scaled &= ~seen;
        }
        if (seen & ~done & locm) run(acc2, locm);
        if (seen & ~done & ~locm) run(acc, ~locm);
        if (__builtin_expect((scaled & locm) != 0, 0)) run_scaled(acc2, scaled & locm, rank, owner);
        if (__builtin_expect((scaled & ~locm) != 0, 0)) run_scaled(acc, scaled & ~locm, owner, rank);
    }

    const float qn = __builtin_nanf("");
    const __amdgpu_buffer_rsrc_t rout = mk_rsrc(p.out, OUT_BYTES);
    const unsigned oo = ((unsigned)(hh * 256 + wm * 64 + (lane >> 4) * 4) * NG + (unsigned)(owner * NS + nt * 128 + wn * 64 + (lane & 15) * 4)) * 4u;
    auto put = [&](int m) INL {
#pragma unroll
        for (int r = 0; r < 4; ++r) {
            f4 v = {acc[m][0][r] + acc2[m][0][r], acc[m][1][r] + acc2[m][1][r], acc[m][2][r] + acc2[m][2][r], acc[m][3][r] + acc2[m][3][r]};
            if (bad) v = (f4){qn, qn, qn, qn};
            st16(__builtin_bit_cast(u4, v), rout, oo + (unsigned)(m * 16 + r) * (NG * 4u));
        }
    };
    if (orel != 0) {
        const unsigned tile = (unsigned)(hh * 16 + nt);
        {
            if (wid > 0) pace<TAU_LD>(now32(), wid * 16 - LEAD);
            if (bad) {
#pragma unroll
                for (int q = 0; q < 16; ++q) acc[q >> 2][q & 3] = (f4){qn, qn, qn, qn};
            }
            const unsigned po = stg + P_REL + (((unsigned)owner * 32u + tile) * 8u + (unsigned)wid) * 16384u + voff;
#pragma unroll
            for (int q = 0; q < 16; ++q) st16(__builtin_bit_cast(u4, acc[q >> 2][q & 3]), rws, po + q * 1024);
            WAIT_V(0);
            if (lane == 0) __hip_atomic_store((unsigned*)(p.ws[owner] + PFL_OFF + ((unsigned)rank * 32u + tile) * 128u) + wid, call, __ATOMIC_RELAXED, __HIP_MEMORY_SCOPE_SYSTEM);
        }
        {
            const unsigned* f = (const unsigned*)(myws + PFL_OFF + ((unsigned)owner * 32u + tile) * 128u) + wid;
            const unsigned long long t0 = __builtin_amdgcn_s_memrealtime();
            unsigned tv;
            for (unsigned it = 1;; ++it) {
                const unsigned v = __hip_atomic_load(f, __ATOMIC_RELAXED, __HIP_MEMORY_SCOPE_SYSTEM);
                const unsigned tm = __hip_atomic_load(tmo, __ATOMIC_RELAXED, __HIP_MEMORY_SCOPE_AGENT);
                tv = __hip_atomic_load((unsigned*)(myws + LNK_OFF) + owner * 32 + (lane & 31), __ATOMIC_RELAXED, __HIP_MEMORY_SCOPE_AGENT);
                if (__builtin_amdgcn_readfirstlane(tm) == call) { bad = true; break; }
                if ((int)(__builtin_amdgcn_readfirstlane(v) - call) >= 0) break;
                if ((it & 15u) == 0 && give_up(tmo, call, t0, lane)) { bad = true; break; }
                __builtin_amdgcn_s_sleep(1);
            }
            const unsigned tf = now32();
            if (tv - tf + 20000u > 40000u) tv = tf;
            const unsigned tl = __builtin_amdgcn_readlane(tv, tile);
            while (lds_ld(ctl + 12) < (unsigned)wid) __builtin_amdgcn_s_sleep(1);
            unsigned te = wid > 0 ? lds_ld(ctl + 13) : tl;
            if ((int)(tf - te - slots<TAU_LD>(wid * 16)) > 0) te = tf - slots<TAU_LD>(wid * 16);
            unsigned tlast = te + 30000u;
            if (wid == 7) {
                int dm = (int)(tv - tf);
#pragma unroll
                for (int o = 16; o > 0; o >>= 1) dm = min(dm, __shfl_xor(dm, o));
                tlast = tf + (unsigned)__builtin_amdgcn_readfirstlane(dm) + slots<TAU_LD>(127u);
            }
            unsigned next = te;
            const __amdgpu_buffer_rsrc_t ro = mk_rsrc(p.ws[owner], WS_NEED);
            const unsigned pi = stg + P_REL + (((unsigned)rank * 32u + tile) * 8u + (unsigned)wid) * 16384u + voff;
#pragma unroll
            for (int q = 0; q < 16; ++q) {
                const unsigned due = te + slots<TAU_LD>(wid * 16 + q);
                if ((int)(due - tlast) > 0) wait_until(tlast); else pace_soft(due, next, GAP_OUT);
                acc[q >> 2][q & 3] = __builtin_bit_cast(f4, ld16(ro, pi + q * 1024));
                if ((q & 3) == 1 && q > 4) WAIT_V(5); else WAIT_V(1);
                if ((q & 3) == 0 && q > 0) put((q >> 2) - 1);
            }
            if (lane == 0) {
                __hip_atomic_store(ctl + 13, te, __ATOMIC_RELAXED, __HIP_MEMORY_SCOPE_WORKGROUP);
                __hip_atomic_store(ctl + 12, (unsigned)wid + 1u, __ATOMIC_RELAXED, __HIP_MEMORY_SCOPE_WORKGROUP);
            }
            put(3);
        }
    } else {
        if (!bad) bad = __builtin_amdgcn_readfirstlane(__hip_atomic_load(tmo, __ATOMIC_RELAXED, __HIP_MEMORY_SCOPE_AGENT)) == call;
#pragma unroll
        for (int m = 0; m < 4; ++m) put(m);
    }
}

}

extern "C" void kernel_launch(int rank, int nranks, ncclComm_t comm,
                              hipStream_t stream,
                              void* const* d_in, const int* in_sizes, int n_in,
                              void* d_out, int out_size,
                              const int* in_dtypes, int out_dtype,
                              void* const* d_out_all,
                              void* const* d_ws_all, size_t ws_bytes) {
    (void)comm; (void)in_dtypes; (void)out_dtype; (void)d_out_all;
    if (nranks != NR || (unsigned)rank >= (unsigned)NR || n_in < 2 || in_sizes[0] != MG * KS || in_sizes[1] != KS * NG
        || out_size != MS * NG || ws_bytes < WS_NEED) {
        fprintf(stderr, "kernel_launch: unexpected problem shape\n");
        return;
    }
    Params p;
    p.x = (const float*)d_in[0]; p.w = (const float*)d_in[1]; p.out = (float*)d_out; p.rank = rank; p.pad = 0; p.myws = (char*)d_ws_all[rank];
    for (int i = 0; i < NR; ++i) p.ws[i] = (char*)d_ws_all[i];
    hipLaunchKernelGGL(gemm_rs_kernel, dim3(NWG), dim3(512), 0, stream, p);
    HIP_CHECK(hipGetLastError());
}
```

```cpp
#include <hip/hip_runtime.h>
#include <rccl/rccl.h>
#include <cstdio>

#define HIP_CHECK(expr) do { hipError_t _e = (expr); if (_e != hipSuccess) { \
    fprintf(stderr, "HIP error %s at %s:%d\n", hipGetErrorString(_e), __FILE__, __LINE__); \
    return; } } while (0)

namespace {

typedef unsigned u4 __attribute__((ext_vector_type(4)));
typedef float f4 __attribute__((ext_vector_type(4)));
typedef _Float16 h8 __attribute__((ext_vector_type(8)));
#define LAS __attribute__((address_space(3)))
#define GAS __attribute__((address_space(1)))
#define INL __attribute__((always_inline))

constexpr int NR = 8;
constexpr int MG = 4096, NG = 16384, KS = 512, NS = 2048, MS = 512;
constexpr int NWG = 256, NOWN = 32;
constexpr int STAGE_A = 32768, STAGE_B = 16384, STAGE = STAGE_A + STAGE_B, NSTAGE = 3;
constexpr int CTL = NSTAGE * STAGE;
constexpr int LDS_BYTES = CTL + 128;
constexpr unsigned TMO_OFF = 0;
constexpr unsigned LNK_OFF = 3072;
constexpr unsigned ORG_OFF = 1024;
constexpr unsigned SEQ_OFF = 4096;
constexpr unsigned AGG_OFF = 36864;
constexpr unsigned PFL_OFF = 65536;
constexpr unsigned FLG_OFF = 98304, FLG_REP = 4352;
constexpr unsigned RDY_OFF = 147456;
constexpr unsigned EXP_OFF = 196608;
constexpr unsigned STG_OFF = 1u << 20;
constexpr unsigned A_BYTES = 32u << 20;
constexpr unsigned B_BYTES = 16u << 20;
constexpr unsigned W_BYTES = 16u << 20;
constexpr unsigned P_BYTES = 32u << 20;
constexpr unsigned B_REL = A_BYTES, W_REL = B_REL + B_BYTES, P_REL = W_REL + W_BYTES, SLOT_BYTES = P_REL + P_BYTES;
constexpr unsigned WS_NEED = STG_OFF + 2 * SLOT_BYTES;
constexpr unsigned KT_A = 16u * STAGE_A, KT_B = 16u * STAGE_B;
constexpr unsigned OOB = 0x80000000u;
constexpr unsigned OUT_BYTES = (unsigned)MS * NG * 4;
constexpr unsigned long long DEADLINE = 100000000ull;
constexpr int AUX_SYS = 17;
constexpr int AUX_LD = 16;

struct Params {
    const float* x; char* myws; int rank, pad; const float* w;
    char* ws[NR]; float* out;
};

#define WAIT_V(n) asm volatile("s_waitcnt vmcnt(" #n ")" ::: "memory")
constexpr unsigned tau(double gbs) { return (unsigned)(32768.0 / gbs / 10.0 * 256.0); }
constexpr unsigned TAU_ST = tau(58.0), TAU_IN = tau(60.8), TAU_LD = tau(61.0);
constexpr unsigned GAP_ST = 45, GAP_IN = 52, GAP_OUT = 51;
constexpr unsigned RDY_LAT = 220, SLACK = 60;
constexpr unsigned T_START = 400, CONV_TICKS = 192;
constexpr int LEAD = 8;
constexpr unsigned LOW_BITS = (127u - 9u) << 23;
#define BAR() do { asm volatile("" ::: "memory"); __builtin_amdgcn_s_waitcnt(0xC07F);   __builtin_amdgcn_s_barrier(); asm volatile("" ::: "memory"); } while (0)

__device__ __forceinline__ __amdgpu_buffer_rsrc_t mk_rsrc(const void* p, unsigned bytes) {
    return __builtin_amdgcn_make_buffer_rsrc((void*)p, 0, (int)bytes, 0x00020000);
}
__device__ __forceinline__ void st16(u4 v, __amdgpu_buffer_rsrc_t r, unsigned off) {
    __builtin_amdgcn_raw_buffer_store_b128(v, r, (int)off, 0, AUX_SYS);
}
__device__ __forceinline__ u4 ld16(__amdgpu_buffer_rsrc_t r, unsigned off) {
    return __builtin_amdgcn_raw_buffer_load_b128(r, (int)off, 0, AUX_SYS);
}
__device__ __forceinline__ unsigned now32() { return (unsigned)__builtin_amdgcn_s_memrealtime(); }
template <unsigned TAU> __device__ __forceinline__ unsigned slots(unsigned n) { return (n * TAU) >> 8; }
__device__ __forceinline__ void wait_until(unsigned due) {
    while ((int)(now32() - due) < 0) __builtin_amdgcn_s_sleep(1);
    asm volatile("" ::: "memory");
}
template <unsigned TAU> __device__ __forceinline__ void pace(unsigned t0, unsigned n) { wait_until(t0 + slots<TAU>(n)); }
__device__ __forceinline__ void pace_soft(unsigned due, unsigned& next, unsigned gap) {
    if ((int)(next - due) > 0) due = next;
    const unsigned now = now32();
    if ((int)(now - due) > 0) due = now; else wait_until(due);
    asm volatile("" ::: "memory");
    next = due + gap;
}
__device__ __forceinline__ unsigned lds_ld(LAS unsigned* p) {
    return __builtin_amdgcn_readfirstlane(__hip_atomic_load(p, __ATOMIC_RELAXED, __HIP_MEMORY_SCOPE_WORKGROUP));
}
__device__ __forceinline__ unsigned lds_add(LAS unsigned* p, unsigned v, int lane) {
    unsigned old = 0;
    if (lane == 0) old = __hip_atomic_fetch_add(p, v, __ATOMIC_RELAXED, __HIP_MEMORY_SCOPE_WORKGROUP);
    return __builtin_amdgcn_readfirstlane(old) + v;
}
__device__ __forceinline__ u4 cvt8(f4 a, f4 b) {
    h8 h = {(_Float16)a[0], (_Float16)a[1], (_Float16)a[2], (_Float16)a[3], (_Float16)b[0], (_Float16)b[1], (_Float16)b[2], (_Float16)b[3]};
    return __builtin_bit_cast(u4, h);
}
template <int N> __device__ __forceinline__ int block_scale(f4 (&v)[N]) {
    float m = 0.f;
#pragma unroll
    for (int i = 0; i < N; ++i) m = fmaxf(fmaxf(m, fmaxf(fabsf(v[i][0]), fabsf(v[i][1]))), fmaxf(fabsf(v[i][2]), fabsf(v[i][3])));
    int e = 0;
    if (__builtin_expect((__builtin_amdgcn_ballot_w64(m >= 32768.f) != 0) | (__builtin_amdgcn_ballot_w64(m >= __uint_as_float(LOW_BITS)) == 0), 0)) {
        unsigned mb = 0;
#pragma unroll
        for (int i = 0; i < N; ++i)
#pragma unroll
            for (int j = 0; j < 4; ++j) { const unsigned t = __float_as_uint(v[i][j]) & 0x7fffffffu; if (t < 0x7f800000u && t > mb) mb = t; }
#pragma unroll
        for (int o = 32; o > 0; o >>= 1) { const unsigned t = (unsigned)__shfl_xor((int)mb, o); mb = t > mb ? t : mb; }
        mb = __builtin_amdgcn_readfirstlane(mb);
        const int ex = mb >= 0x00800000u ? (int)(mb >> 23) - 127 : -118 - __builtin_clz(mb | 1u);
        e = mb == 0u || (mb >= LOW_BITS && mb < 0x47000000u) ? 0 : (ex - 6) & ~7;
#pragma unroll
        for (int i = 0; i < N; ++i)
#pragma unroll
            for (int j = 0; j < 4; ++j) v[i][j] = ldexpf(v[i][j], -e);
    }
    return e;
}
__device__ __forceinline__ bool give_up(unsigned* tmo, unsigned call, unsigned long long t0, int lane) {
    if (__builtin_amdgcn_readfirstlane(__hip_atomic_load(tmo, __ATOMIC_RELAXED, __HIP_MEMORY_SCOPE_AGENT)) == call
        || __builtin_amdgcn_s_memrealtime() - t0 > DEADLINE) {
        if (lane == 0) __hip_atomic_store(tmo, call, __ATOMIC_RELAXED, __HIP_MEMORY_SCOPE_AGENT);
        return true;
    }
    return false;
}

__global__ __launch_bounds__(512, 2) void gemm_rs_kernel(const Params p) {
    __shared__ __attribute__((aligned(16))) char smem[LDS_BYTES];
    const int tid = threadIdx.x, lane = tid & 63;
    const int wid = __builtin_amdgcn_readfirstlane(tid >> 6);
    const int b = blockIdx.x, rank = p.rank;
    char* const myws = p.myws;
    LAS unsigned* const ctl = (LAS unsigned*)((LAS char*)smem + CTL);
    const unsigned long long tstart = __builtin_amdgcn_s_memrealtime();
    const int xcd = b & 7, slot = b >> 3;
    const int orel = slot >> 2;
    const int owner = (rank + orel) & 7;
    const int hh = xcd & 1, nt = (xcd >> 1) * 4 + (slot & 3);
    const __amdgpu_buffer_rsrc_t rws = mk_rsrc(myws, WS_NEED);
    const int voff = lane * 16;
    unsigned* const tmo = (unsigned*)(myws + TMO_OFF);

    const int sb = b & 31;
    unsigned long long* const torg = (unsigned long long*)(myws + ORG_OFF + sb * 64);
    const float* const xs = p.x + (size_t)((wid * 32 + sb) * 16 + (lane >> 2)) * KS + ((lane & 3) ^ ((lane >> 5) << 1)) * 4;
    const float* const wsrc = p.w + (size_t)((lane >> 4) * 4) * NG + wid * NS + sb * 64 + (lane & 15) * 4;
    u4 xh[2] = {}, wh[8] = {};
    f4 xr[4], wr[16];
    int xe = 0, we = 0;
    auto load_x = [&](int c) INL {
#pragma unroll
        for (int i = 0; i < 4; ++i) xr[i] = *(const f4*)(xs + c * 64 + i * 16);
    };
    auto load_w = [&](int c) INL {
#pragma unroll
        for (int i = 0; i < 16; ++i) wr[i] = *(const f4*)(wsrc + (size_t)(c * 64 + (i >> 2) * 16 + (i & 3)) * NG);
    };
    auto cvt_x = [&]() INL { xe = block_scale(xr); xh[0] = cvt8(xr[0], xr[1]); xh[1] = cvt8(xr[2], xr[3]); };
    auto cvt_w = [&]() INL {
        we = block_scale(wr);
#pragma unroll
        for (int ks = 0; ks < 2; ++ks)
#pragma unroll
            for (int jj = 0; jj < 4; ++jj) {
                const f4 lo = {wr[ks * 8][jj], wr[ks * 8 + 1][jj], wr[ks * 8 + 2][jj], wr[ks * 8 + 3][jj]};
                const f4 hi = {wr[ks * 8 + 4][jj], wr[ks * 8 + 5][jj], wr[ks * 8 + 6][jj], wr[ks * 8 + 7][jj]};
                wh[ks * 4 + jj] = cvt8(lo, hi);
            }
    };
    const int uc = (b >> 5) == 7 ? 0 : b >> 5;
    const bool ux = b < 224;
    unsigned call, call2, stg, rdyo, expo, xoff;
    auto read_call = [&]() INL {
        asm volatile("s_load_dword %0, %1, 0x0\n\ts_waitcnt lgkmcnt(0)" : "=s"(call) : "s"(myws + SEQ_OFF + b * 128) : "memory");
        ++call; call2 = 2u * call;
        stg = STG_OFF + (call & 1u) * SLOT_BYTES;
        rdyo = RDY_OFF + (call & 1u) * 16384u + (unsigned)sb * 512u; expo = EXP_OFF + (call & 1u) * 16384u;
        xoff = stg + (unsigned)(rank * 8) * KT_A + (unsigned)(wid * 32 + sb) * 2048u + voff;
    };
    auto first_barrier = [&]() INL {
        if (wid == rank && lane < 32) ctl[lane] = 0;
        BAR();
        if (wid == rank && lane == 0) __hip_atomic_store((unsigned*)(myws + SEQ_OFF + b * 128), call, __ATOMIC_RELAXED, __HIP_MEMORY_SCOPE_AGENT);
    };

    auto arrive = [&](bool on, int fw, unsigned weight, bool scaled) INL {
        if (on) {
            unsigned long long* cnt = (unsigned long long*)(myws + AGG_OFF + fw * 128);
            const unsigned long long add = weight | ((unsigned long long)scaled << 32);
            const unsigned long long sum = __hip_atomic_fetch_add(cnt, add, __ATOMIC_RELAXED, __HIP_MEMORY_SCOPE_AGENT) + add;
            if (((unsigned)sum & 0x7fffffffu) == ((call * 64u) & 0x7fffffffu)) {
#pragma unroll
                for (int r = 0; r < 8; ++r)
                    __hip_atomic_store((unsigned*)(myws + FLG_OFF + r * FLG_REP) + fw, call2 + (unsigned)((sum >> 32) != 0), __ATOMIC_RELAXED, __HIP_MEMORY_SCOPE_AGENT);
                if (sum >> 31) __hip_atomic_fetch_and(cnt, 0x7fffffffull, __ATOMIC_RELAXED, __HIP_MEMORY_SCOPE_AGENT);
            }
        }
    };
    auto tell_peers = [&](int word, bool scaled) INL {
        char* w = p.ws[0];
#pragma unroll
        for (int i = 1; i < NR; ++i) if (lane == i) w = p.ws[i];
        if (lane < NR && lane != rank) __hip_atomic_store((GAS unsigned*)(w + rdyo) + word, call2 + (unsigned)scaled, __ATOMIC_RELAXED, __HIP_MEMORY_SCOPE_SYSTEM);
    };
    auto store_e = [&](int which, int c, int e) INL {
        if (lane == 0) __hip_atomic_store((int*)(myws + expo) + ((which * 8 + c) * 32 + sb) * 8 + wid, e, __ATOMIC_RELAXED, __HIP_MEMORY_SCOPE_SYSTEM);
    };
    auto store_x = [&](int c) INL { st16(xh[0], rws, xoff + c * KT_A); st16(xh[1], rws, xoff + c * KT_A + 1024); store_e(0, c, xe); };
    auto store_w = [&](int c) INL {
        const unsigned wo = stg + (wid == rank ? B_REL : W_REL) + (unsigned)(wid * 8 + c) * KT_B + (unsigned)sb * 8192u + voff;
#pragma unroll
        for (int q = 0; q < 8; ++q) st16(wh[q], rws, wo + q * 1024);
        store_e(1, c, we);
    };

    if (__builtin_expect(b >= NOWN, 0)) {
        if (b >= 64 && b < 224) {
            if (wid == ((rank + 1) & 7)) {
                const int j = (b >> 3) & 7;
                const __amdgpu_buffer_rsrc_t rj = b < 192 ? mk_rsrc(p.ws[j], WS_NEED) : mk_rsrc(p.x, (unsigned)MG * KS * 4);
                const unsigned par = STG_OFF + (lane & 1) * SLOT_BYTES, q = lane >> 1;
                unsigned o;
                if (b < 128) o = q == 0 ? par + (unsigned)(rank * 8) * KT_A : q == 1 ? par + (unsigned)(rank * 8 + 1) * KT_A - 4u : OOB;
                else if (b >= 192) o = lane < 2 ? (unsigned)(3 - j) * 0x200000u + lane * 0x1ffffcu : OOB;
                else if (j == rank) o = STG_OFF + (unsigned)lane * 0x200000u;
                else o = q < 2 ? par + W_REL + (unsigned)(rank * 8) * KT_B + q * 0x1ffffcu : q == 2 ? RDY_OFF : q < 6 ? par + (unsigned)(j * 8) * KT_A + (q - 3) * 0x1ffffcu
                       : q < 9 ? par + P_REL + (unsigned)rank * 0x400000u + (q - 6) * 0x1ffffcu : OOB;
                unsigned t = __builtin_amdgcn_raw_buffer_load_b32(rj, (int)o, 0, AUX_SYS);
                if (b >= 128 && b < 192 && j == rank) {
                    t += __builtin_amdgcn_raw_buffer_load_b32(rj, (int)(o + 64u * 0x200000u), 0, AUX_SYS);
                    t += __builtin_amdgcn_raw_buffer_load_b32(mk_rsrc(p.out, OUT_BYTES), lane * 0x1ffffc, 0, AUX_SYS);
                    t += __builtin_amdgcn_raw_buffer_load_b32(mk_rsrc(p.w, (unsigned)KS * NG * 4), lane * 0x1ffffc, 0, 0);
                }
                asm volatile("" :: "v"(t));
            }
            wait_until((unsigned)tstart + T_START + (unsigned)(b - 64) * CONV_TICKS / 16u);
        }
        if (ux) load_x(uc);
        load_w(uc);
        read_call();
        first_barrier();
        if (ux) { cvt_x(); store_x(uc); }
        cvt_w(); store_w(uc);
        WAIT_V(0);
        unsigned sum = lds_add(ctl + 0, (xe | we) ? 257u : 1u, lane);
        if ((sum & 255u) == 8u) {
            arrive(lane == 0, rank * 8 + uc, ux ? 2u : 1u, sum > 255u);
            if (!ux) {
                unsigned long long o;
                bool late = false;
                while ((unsigned)((o = __hip_atomic_load(torg, __ATOMIC_RELAXED, __HIP_MEMORY_SCOPE_AGENT)) >> 32) != call) {
                    if (give_up(tmo, call, tstart, lane)) { late = true; break; }
                    __builtin_amdgcn_s_sleep(4);
                }
                if (!late) wait_until(__builtin_amdgcn_readfirstlane((unsigned)o) + slots<TAU_ST>(12u) - RDY_LAT);
            }
            tell_peers(rank * 8 + uc, sum > 255u);
        }
        if (!ux) {
            wait_until((unsigned)tstart + T_START + (unsigned)(b - 64) * CONV_TICKS / 16u);
            load_x(7); load_w(7);
            cvt_x(); store_x(7); cvt_w(); store_w(7);
            WAIT_V(0);
            sum = lds_add(ctl + 1, (xe | we) ? 257u : 1u, lane);
            if ((sum & 255u) == 8u) { arrive(lane == 0, rank * 8 + 7, 2u, sum > 255u); tell_peers(rank * 8 + 7, sum > 255u); }
        }
    } else {
        load_x(0);
        const int u = (wid - rank - 1) & 7;
        unsigned xg[6];
        char* pw[6];
#pragma unroll
        for (int s = 0; s < 6; ++s) { int a = u + 1 + s; if (a >= 7) a -= 7; xg[s] = (unsigned)(rank + 1 + a) & 7u; pw[s] = p.ws[xg[s]]; }
        asm volatile("" : "+s"(pw[0]), "+s"(pw[1]), "+s"(pw[2]), "+s"(pw[3]), "+s"(pw[4]), "+s"(pw[5]));
        read_call();
        cvt_x();
        asm volatile("" : "+v"(xh[0]), "+v"(xh[1]));
        const unsigned long long t64 = __builtin_amdgcn_s_memrealtime();
        if (__builtin_expect(wid == rank, 0)) {
            store_x(0);
            first_barrier();
            WAIT_V(0);
            lds_add(ctl + 0, xe ? 257u : 1u, lane);
            const unsigned* f = (const unsigned*)(myws + rdyo);
            bool told = false, all = false;
            unsigned fx = 0, ann = lane < NR && lane != rank ? 0u : 10u;
            unsigned long long scl = 0;
            pace<TAU_ST>((unsigned)t64, 6);
            for (unsigned it = 1;; ++it) {
                if (!told) {
                    const unsigned sum = lds_ld(ctl + 0);
                    if ((sum & 255u) == 8u) { told = true; tell_peers(64 + rank, sum > 255u); arrive(lane == 0, rank * 8, 1u, sum > 255u); }
                }
                if (!all && (fx != 0xffu || (it & 7u) == 0)) {
                    const unsigned v = __hip_atomic_load(f + lane, __ATOMIC_RELAXED, __HIP_MEMORY_SCOPE_SYSTEM);
                    const unsigned vx = __hip_atomic_load(f + 64 + (lane >> 3), __ATOMIC_RELAXED, __HIP_MEMORY_SCOPE_SYSTEM);
                    unsigned long long full = __ballot((lane >> 3) == rank || (int)(v - call2) >= 0);
                    const unsigned long long fx8 = __ballot((lane >> 3) == rank || (int)(vx - call2) >= 0);
                    scl = __ballot(v == call2 + 1u || ((lane & 7) == 0 && vx == call2 + 1u));
                    fx = (unsigned)__ballot((fx8 >> ((lane & 7) * 8)) & 1u) & 0xffu;
                    if ((it & 31u) == 0 && give_up(tmo, call, tstart, lane)) { full = ~0ull; fx = 0xffu; }
                    if (lane < 3) __hip_atomic_store(ctl + 4 + lane, lane == 2 ? fx : (unsigned)(full >> (lane * 32)), __ATOMIC_RELAXED, __HIP_MEMORY_SCOPE_WORKGROUP);
                    all = full == ~0ull && fx == 0xffu;
                }
                const unsigned prog = __hip_atomic_load(ctl + 24 + (lane & 7), __ATOMIC_RELAXED, __HIP_MEMORY_SCOPE_WORKGROUP);
                const bool on = prog > ann && ((fx >> (lane & 7)) & 1u);
                const unsigned sc = (unsigned)(scl >> ((lane & 7) * 8)) & 0xffu;
                arrive(on, ann < 7u ? lane * 8 + (int)ann : ann == 7u ? 72 + lane : ann == 8u ? lane * 8 + 7 : 64 + lane, 2u,
                       ann < 9u ? (sc >> (ann < 7u ? ann : 7u)) & 1u : sc != 0u);
                if (on) ++ann;
                if (told && __all(ann == 10u)) break;
                __builtin_amdgcn_s_sleep(1);
            }
        } else {
            {
                unsigned nx = (unsigned)t64;
                __amdgpu_buffer_rsrc_t rd[6];
#pragma unroll
                for (int s = 0; s < 6; ++s) rd[s] = mk_rsrc(pw[s], WS_NEED);
#pragma unroll
                for (int i = 0; i < 12; ++i) {
                    if (i > 0) pace_soft((unsigned)t64 + slots<TAU_ST>(i), nx, GAP_ST);
                    st16(xh[i & 1], rd[i >> 1], xoff + (i & 1) * 1024);
                    if (i == 0) {
                        store_x(0);
                        if (u == 0 && lane == 0) __hip_atomic_store(torg, (unsigned long long)call << 32 | (unsigned)t64, __ATOMIC_RELAXED, __HIP_MEMORY_SCOPE_AGENT);
                    } else WAIT_V(1);
                    if (i == 3) first_barrier();
                }
            }
            const __amdgpu_buffer_rsrc_t rs = mk_rsrc(p.ws[wid], WS_NEED);
            const unsigned xo = stg + (unsigned)(wid * 8) * KT_A + (unsigned)sb * 2048u + voff;
            const unsigned wro = stg + W_REL + (unsigned)(rank * 8) * KT_B + (unsigned)sb * 8192u + voff;
            const unsigned wlo = stg + B_REL + (unsigned)(wid * 8) * KT_B + (unsigned)sb * 8192u + voff;
            unsigned ts = 0;
            auto need = [&](int word, int bit) INL {
                while (!((lds_ld(ctl + word) >> bit) & 1u)) __builtin_amdgcn_s_sleep(1);
                asm volatile("" ::: "memory");
            };
            need(4 + (wid >> 2), (wid & 3) * 8);
            {
                unsigned long long t = __builtin_amdgcn_s_memrealtime();
                const unsigned long long tp = t64 + slots<TAU_ST>(12u);
                if (t < tp + SLACK) t = tp;
                ts = (unsigned)t - slots<TAU_IN>(12u);
                if (lane == 0) __hip_atomic_store((unsigned*)(myws + LNK_OFF) + wid * 32 + sb, (unsigned)t + slots<TAU_IN>(164u), __ATOMIC_RELAXED, __HIP_MEMORY_SCOPE_AGENT);
            }
            unsigned next = ts;
            u4 d[2] = {};
            unsigned lo[2] = {OOB, OOB};
            auto progress = [&](unsigned v) INL { if (lane == 0) __hip_atomic_store(ctl + 24 + wid, v, __ATOMIC_RELAXED, __HIP_MEMORY_SCOPE_WORKGROUP); };
            auto slot = [&](int i, unsigned n, unsigned ro, unsigned l) INL {
                pace_soft(ts + slots<TAU_IN>(n), next, GAP_IN);
                lo[i & 1] = l;
                d[i & 1] = ld16(rs, ro);
                asm volatile("" ::: "memory");
                st16(d[(i & 1) ^ 1], rws, lo[(i & 1) ^ 1]);
                asm volatile("" ::: "memory");
            };
#pragma unroll
            for (int i = 12; i < 20; ++i) {
                slot(i, i, wro + (i - 12) * 1024, wlo + (i - 12) * 1024);
                if (i == 12) { WAIT_V(2); lds_add(ctl + 0, xe ? 257u : 1u, lane); }
            }
            for (int c = 1; c < 8; ++c) {
                need(4 + (wid >> 2), (wid & 3) * 8 + c);
#pragma unroll
                for (int i = 0; i < 20; ++i) {
                    const int ks = i / 10, r = i % 10;
                    if (r < 6) { const unsigned o = xo + c * KT_A + xg[r] * 65536u + ks * 1024; slot(i, c * 20 + i, o, o); }
                    else slot(i, c * 20 + i, wro + c * KT_B + (ks * 4 + r - 6) * 1024, wlo + c * KT_B + (ks * 4 + r - 6) * 1024);
                    if (i == 1) { WAIT_V(2); progress(c); }
                    if (i == 11 && c == 7) { WAIT_V(2); progress(8); }
                }
            }
            need(6, wid);
#pragma unroll
            for (int i = 0; i < 16; ++i) {
                const unsigned o = xo + (unsigned)(i >> 1) * KT_A + (unsigned)rank * 65536u + (i & 1) * 1024;
                slot(i, 160 + i, o, o);
                if (i == 0) { WAIT_V(0); progress(9); }
            }
            st16(d[1], rws, lo[1]);
            WAIT_V(0);
            progress(10);
        }
        BAR();
    }

    const int wm = wid >> 1, wn = wid & 1;
    const int lvoff = wid * 1024 + voff;
    const int aoff = (lane & 15) * 64 + (((lane >> 4) ^ ((lane >> 2) & 2)) << 4);
    int fw0, fw1, tsrc;
    unsigned taba, tabb;
    {
        const int src = lane >> 3, c = lane & 7;
        fw0 = fw1 = orel == 0 && src != rank ? 64 + src : lane;
        tsrc = src * 17;
        taba = stg + (unsigned)lane * KT_A + (unsigned)(owner * 2 + hh) * STAGE_A;
        tabb = stg + B_REL + (unsigned)lane * KT_B + (unsigned)nt * STAGE_B;
        if (orel != 0 && src == owner) {
            fw0 = fw1 = rank * 8 + c;
            tsrc = rank * 17;
            taba = stg + (unsigned)(rank * 8 + c) * KT_A + (unsigned)(rank * 2 + hh) * STAGE_A;
            tabb = stg + W_REL + (unsigned)lane * KT_B + (unsigned)nt * STAGE_B;
        } else if (orel != 0 && c == 7 && src != rank) {
            const int t = (src - rank) & 7, k = t < orel ? t - 1 : t - 2, ka = k & ~1, kb = k | 1, h = k & 1;
            const int sa = (rank + (ka + 1 < orel ? ka + 1 : ka + 2)) & 7, sb2 = (rank + (kb + 1 < orel ? kb + 1 : kb + 2)) & 7;
            fw0 = h ? sa * 8 + 7 : 72 + sa; fw1 = h ? sb2 * 8 + 7 : 72 + sb2;
            tsrc = sa | sb2 << 4;
            taba += (unsigned)((((wid & 1) ? sb2 : sa) - src) * 8) * KT_A + (unsigned)((h - (wid & 1)) * 1024);
            tabb += (unsigned)((((wid >> 2) ? sb2 : sa) - src) * 8) * KT_B + (unsigned)((h - (wid >> 2)) * 4096);
        }
    }
    const unsigned* const myflags = (const unsigned*)(myws + FLG_OFF + xcd * FLG_REP);
    f4 acc[4][4], acc2[4][4];
#pragma unroll
    for (int m = 0; m < 4; ++m)
#pragma unroll
        for (int n = 0; n < 4; ++n) { acc[m][n] = (f4){0.f, 0.f, 0.f, 0.f}; acc2[m][n] = (f4){0.f, 0.f, 0.f, 0.f}; }

    auto stage1 = [&](unsigned ao, unsigned bo, int buf, int i) INL {
        LAS char* const l = (LAS char*)smem + buf * STAGE + wid * 1024 + i * 8192;
        __builtin_amdgcn_raw_ptr_buffer_load_lds(rws, l, 16, lvoff, i < 4 ? ao + i * 8192 : bo + (i - 4) * 8192, 0, AUX_LD);
    };
    auto stage = [&](int kt, int buf) INL {
        const unsigned ao = __builtin_amdgcn_readlane(taba, kt), bo = __builtin_amdgcn_readlane(tabb, kt);
#pragma unroll
        for (int i = 0; i < 6; ++i) stage1(ao, bo, buf, i);
    };
    auto frag = [&](h8 (&af)[4], h8 (&bf)[4], int buf, int ks, int g) INL {
        const LAS char* const l = (const LAS char*)smem + buf * STAGE;
        if (g < 4) bf[g] = *(const LAS h8*)(l + STAGE_A + wn * 8192 + ks * 4096 + g * 1024 + voff);
        else af[g - 4] = *(const LAS h8*)(l + wm * 8192 + ks * 1024 + (g - 4) * 2048 + aoff);
    };
    auto kstep = [&](f4 (&c)[4][4], const h8 (&af)[4], const h8 (&bf)[4], h8 (&an)[4], h8 (&bn)[4], int rbuf, int rks,
                     bool dma, unsigned ao, unsigned bo, int sbuf) INL {
#pragma unroll
        for (int g = 0; g < 8; ++g) {
            const int m = g >> 1, n0 = (g & 1) * 2;
            c[m][n0] = __builtin_amdgcn_mfma_f32_16x16x32_f16(af[m], bf[n0], c[m][n0], 0, 0, 0);
            if (g < 2) { frag(an, bn, rbuf, rks, 2 * g); frag(an, bn, rbuf, rks, 2 * g + 1); }
            else if (g < 6) frag(an, bn, rbuf, rks, g + 2);
            c[m][n0 + 1] = __builtin_amdgcn_mfma_f32_16x16x32_f16(af[m], bf[n0 + 1], c[m][n0 + 1], 0, 0, 0);
            if (dma && g >= 2) stage1(ao, bo, sbuf, g - 2);
            __builtin_amdgcn_sched_barrier(0);
        }
    };
    int buf = 0;
    unsigned long long done = 0, seen = 0;
    const unsigned* const pf0 = myflags + fw0;
    const unsigned* const pf1 = myflags + fw1;
    auto run = [&](f4 (&c)[4][4], unsigned long long kind) INL {
        unsigned long long avail = seen & kind & ~done;
        done |= avail;
        stage(__builtin_ctzll(avail), buf); avail &= avail - 1;
        int sbuf = buf + 1 == NSTAGE ? 0 : buf + 1, ahead = 0, par = 0;
        if (avail) { stage(__builtin_ctzll(avail), sbuf); avail &= avail - 1; sbuf = sbuf + 1 == NSTAGE ? 0 : sbuf + 1; ahead = 1; WAIT_V(6); }
        else {
            if (wid == 0) {
                __builtin_amdgcn_s_sleep(12);
                const unsigned v0 = __hip_atomic_load(pf0, __ATOMIC_RELAXED, __HIP_MEMORY_SCOPE_AGENT);
                const unsigned v1 = __hip_atomic_load(pf1, __ATOMIC_RELAXED, __HIP_MEMORY_SCOPE_AGENT);
                const unsigned long long m = __ballot(v0 == call2 && v1 == call2);
                if (lane < 2) ctl[14 + lane] = (unsigned)(m >> (lane * 32));
            }
            WAIT_V(0);
        }
        BAR();
        h8 a0[4], b0[4], a1[4], b1f[4];
#pragma unroll
        for (int g = 0; g < 8; ++g) frag(a0, b0, buf, 0, g);
        if (ahead == 0) {
            avail = (((unsigned long long)lds_ld(ctl + 15) << 32) | lds_ld(ctl + 14)) & kind & ~done;
            done |= avail;
        }
        bool more;
        do {
            unsigned v0 = 0, v1 = 0;
            if (wid == 0) asm volatile("global_load_dword %0, %2, off sc1\n\tglobal_load_dword %1, %3, off sc1" : "=&v"(v0), "=&v"(v1) : "v"(pf0), "v"(pf1) : "memory");
            if (ahead == 0 && avail) { stage(__builtin_ctzll(avail), sbuf); avail &= avail - 1; sbuf = sbuf + 1 == NSTAGE ? 0 : sbuf + 1; ahead = 1; }
            more = ahead != 0;
            const int kt = __builtin_ctzll(avail | (1ull << 63));
            const unsigned ao = avail ? __builtin_amdgcn_readlane(taba, kt) : OOB, bo = avail ? __builtin_amdgcn_readlane(tabb, kt) : OOB;
            const int b1 = buf + 1 == NSTAGE ? 0 : buf + 1;
            kstep(c, a0, b0, a1, b1f, buf, 1, true, ao, bo, sbuf);
            if (avail) { avail &= avail - 1; sbuf = sbuf + 1 == NSTAGE ? 0 : sbuf + 1; } else ahead = 0;
            WAIT_V(6);
            if (wid == 0) {
                asm volatile("" : "+v"(v0), "+v"(v1));
                const unsigned long long m = __ballot(v0 == call2 && v1 == call2);
                if (lane < 2) ctl[8 + par + lane] = (unsigned)(m >> (lane * 32));
            }
            BAR();
            const unsigned mlo = ctl[8 + par], mhi = ctl[9 + par];
            kstep(c, a1, b1f, a0, b0, b1, 0, false, 0, 0, 0);
            seen = ((unsigned long long)__builtin_amdgcn_readfirstlane(mhi) << 32) | (unsigned)__builtin_amdgcn_readfirstlane(mlo);
            if (more) { avail |= seen & kind & ~done; done |= avail; }
            par ^= 2;
            buf = b1;
        } while (more);
    };

    auto run_scaled = [&](f4 (&c)[4][4], unsigned long long todo, int jx, int jw) INL {
        done |= todo;
        while (todo) {
            const int kt = __builtin_ctzll(todo);
            todo &= todo - 1;
            stage(kt, buf);
            const int src2 = __builtin_amdgcn_readlane(tsrc, kt);
            int e[2];
#pragma unroll
            for (int ks = 0; ks < 2; ++ks) {
                const int* ep = (const int*)(p.ws[(src2 >> (ks * 4)) & 7] + expo);
                const int idx = (lane & 7) < 4 ? ((kt & 7) * 32 + hh * 16 + wm * 4 + (lane & 3)) * 8 + jx : ((8 + (kt & 7)) * 32 + nt * 2 + wn) * 8 + jw;
                e[ks] = __hip_atomic_load(ep + idx, __ATOMIC_RELAXED, __HIP_MEMORY_SCOPE_SYSTEM);
            }
            WAIT_V(0);
            BAR();
#pragma unroll
            for (int ks = 0; ks < 2; ++ks) {
                h8 af[4], bf[4];
#pragma unroll
                for (int g = 0; g < 8; ++g) frag(af, bf, buf, ks, g);
#pragma unroll
                for (int m = 0; m < 4; ++m) {
                    const int em = __builtin_amdgcn_readlane(e[ks], m) + __builtin_amdgcn_readlane(e[ks], 4);
#pragma unroll
                    for (int n = 0; n < 4; ++n) {
                        const f4 t = __builtin_amdgcn_mfma_f32_16x16x32_f16(af[m], bf[n], (f4){0.f, 0.f, 0.f, 0.f}, 0, 0, 0);
#pragma unroll
                        for (int r = 0; r < 4; ++r) c[m][n][r] += ldexpf(t[r], em);
                    }
                }
            }
            BAR();
        }
    };

    const unsigned long long locm = orel == 0 ? 0ull : 0xffull << (owner * 8);
    bool bad = false;
    while (done != ~0ull) {
        unsigned long long scaled = 0;
        if (!(seen & ~done)) {
            if (wid == 0) {
                const unsigned long long t0 = __builtin_amdgcn_s_memrealtime();
                unsigned long long m, ms;
                for (unsigned it = 1;; ++it) {
                    const unsigned v0 = __hip_atomic_load(pf0, __ATOMIC_RELAXED, __HIP_MEMORY_SCOPE_AGENT);
                    const unsigned v1 = __hip_atomic_load(pf1, __ATOMIC_RELAXED, __HIP_MEMORY_SCOPE_AGENT);
                    m = __ballot(v0 == call2 && v1 == call2);
                    ms = __ballot((v0 | 1u) == call2 + 1u && (v1 | 1u) == call2 + 1u);
                    if ((ms & ~done) || ((it & 63u) == 0 && give_up(tmo, call, t0, lane))) break;
                    __builtin_amdgcn_s_sleep(4);
                }
                if (lane == 0) { ctl[16] = (unsigned)m; ctl[17] = (unsigned)(m >> 32); ctl[20] = (unsigned)ms; ctl[21] = (unsigned)(ms >> 32); }
            }
            BAR();
            seen = ((unsigned long long)lds_ld(ctl + 17) << 32) | lds_ld(ctl + 16);
            scaled = (((unsigned long long)lds_ld(ctl + 21) << 32) | lds_ld(ctl + 20)) & ~done;
            if (!scaled) { bad = true; done = ~0ull; }
            scaled &= ~seen;
        }
        if (seen & ~done & locm) run(acc2, locm);
        if (seen & ~done & ~locm) run(acc, ~locm);
        if (__builtin_expect((scaled & locm) != 0, 0)) run_scaled(acc2, scaled & locm, rank, owner);
        if (__builtin_expect((scaled & ~locm) != 0, 0)) run_scaled(acc, scaled & ~locm, owner, rank);
    }

    const float qn = __builtin_nanf("");
    const __amdgpu_buffer_rsrc_t rout = mk_rsrc(p.out, OUT_BYTES);
    const unsigned oo = ((unsigned)(hh * 256 + wm * 64 + (lane >> 4) * 4) * NG + (unsigned)(owner * NS + nt * 128 + wn * 64 + (lane & 15) * 4)) * 4u;
    auto put = [&](int m) INL {
#pragma unroll
        for (int r = 0; r < 4; ++r) {
            f4 v = {acc[m][0][r] + acc2[m][0][r], acc[m][1][r] + acc2[m][1][r], acc[m][2][r] + acc2[m][2][r], acc[m][3][r] + acc2[m][3][r]};
            if (bad) v = (f4){qn, qn, qn, qn};
            st16(__builtin_bit_cast(u4, v), rout, oo + (unsigned)(m * 16 + r) * (NG * 4u));
        }
    };
    if (orel != 0) {
        const unsigned tile = (unsigned)(hh * 16 + nt);
        {
            if (wid > 0) pace<TAU_LD>(now32(), wid * 16 - LEAD);
            if (bad) {
#pragma unroll
                for (int q = 0; q < 16; ++q) acc[q >> 2][q & 3] = (f4){qn, qn, qn, qn};
            }
            const unsigned po = stg + P_REL + (((unsigned)owner * 32u + tile) * 8u + (unsigned)wid) * 16384u + voff;
#pragma unroll
            for (int q = 0; q < 16; ++q) st16(__builtin_bit_cast(u4, acc[q >> 2][q & 3]), rws, po + q * 1024);
            WAIT_V(0);
            if (lane == 0) __hip_atomic_store((unsigned*)(p.ws[owner] + PFL_OFF + ((unsigned)rank * 32u + tile) * 128u) + wid, call, __ATOMIC_RELAXED, __HIP_MEMORY_SCOPE_SYSTEM);
        }
        {
            const unsigned* f = (const unsigned*)(myws + PFL_OFF + ((unsigned)owner * 32u + tile) * 128u) + wid;
            const unsigned long long t0 = __builtin_amdgcn_s_memrealtime();
            unsigned tv;
            for (unsigned it = 1;; ++it) {
                const unsigned v = __hip_atomic_load(f, __ATOMIC_RELAXED, __HIP_MEMORY_SCOPE_SYSTEM);
                const unsigned tm = __hip_atomic_load(tmo, __ATOMIC_RELAXED, __HIP_MEMORY_SCOPE_AGENT);
                tv = __hip_atomic_load((unsigned*)(myws + LNK_OFF) + owner * 32 + (lane & 31), __ATOMIC_RELAXED, __HIP_MEMORY_SCOPE_AGENT);
                if (__builtin_amdgcn_readfirstlane(tm) == call) { bad = true; break; }
                if ((int)(__builtin_amdgcn_readfirstlane(v) - call) >= 0) break;
                if ((it & 15u) == 0 && give_up(tmo, call, t0, lane)) { bad = true; break; }
                __builtin_amdgcn_s_sleep(1);
            }
            const unsigned tf = now32();
            if (tv - tf + 20000u > 40000u) tv = tf;
            const unsigned tl = __builtin_amdgcn_readlane(tv, tile);
            while (lds_ld(ctl + 12) < (unsigned)wid) __builtin_amdgcn_s_sleep(1);
            unsigned te = wid > 0 ? lds_ld(ctl + 13) : tl;
            if ((int)(tf - te - slots<TAU_LD>(wid * 16)) > 0) te = tf - slots<TAU_LD>(wid * 16);
            unsigned tlast = te + 30000u;
            if (wid == 7) {
                int dm = (int)(tv - tf);
#pragma unroll
                for (int o = 16; o > 0; o >>= 1) dm = min(dm, __shfl_xor(dm, o));
                tlast = tf + (unsigned)__builtin_amdgcn_readfirstlane(dm) + slots<TAU_LD>(127u);
            }
            unsigned next = te;
            const __amdgpu_buffer_rsrc_t ro = mk_rsrc(p.ws[owner], WS_NEED);
            const unsigned pi = stg + P_REL + (((unsigned)rank * 32u + tile) * 8u + (unsigned)wid) * 16384u + voff;
#pragma unroll
            for (int q = 0; q < 16; ++q) {
                const unsigned due = te + slots<TAU_LD>(wid * 16 + q);
                if ((int)(due - tlast) > 0) wait_until(tlast); else pace_soft(due, next, GAP_OUT);
                acc[q >> 2][q & 3] = __builtin_bit_cast(f4, ld16(ro, pi + q * 1024));
                if ((q & 3) == 1 && q > 4) WAIT_V(5); else WAIT_V(1);
                if ((q & 3) == 0 && q > 0) put((q >> 2) - 1);
            }
            if (lane == 0) {
                __hip_atomic_store(ctl + 13, te, __ATOMIC_RELAXED, __HIP_MEMORY_SCOPE_WORKGROUP);
                __hip_atomic_store(ctl + 12, (unsigned)wid + 1u, __ATOMIC_RELAXED, __HIP_MEMORY_SCOPE_WORKGROUP);
            }
            put(3);
        }
    } else {
        if (!bad) bad = __builtin_amdgcn_readfirstlane(__hip_atomic_load(tmo, __ATOMIC_RELAXED, __HIP_MEMORY_SCOPE_AGENT)) == call;
#pragma unroll
        for (int m = 0; m < 4; ++m) put(m);
    }
}

}

extern "C" void kernel_launch(int rank, int nranks, ncclComm_t comm,
                              hipStream_t stream,
                              void* const* d_in, const int* in_sizes, int n_in,
                              void* d_out, int out_size,
                              const int* in_dtypes, int out_dtype,
                              void* const* d_out_all,
                              void* const* d_ws_all, size_t ws_bytes) {
    (void)comm; (void)in_dtypes; (void)out_dtype; (void)d_out_all;
    if (nranks != NR || (unsigned)rank >= (unsigned)NR || n_in < 2 || in_sizes[0] != MG * KS || in_sizes[1] != KS * NG
        || out_size != MS * NG || ws_bytes < WS_NEED) {
        fprintf(stderr, "kernel_launch: unexpected problem shape\n");
        return;
    }
    Params p;
    p.x = (const float*)d_in[0]; p.w = (const float*)d_in[1]; p.out = (float*)d_out; p.rank = rank; p.pad = 0; p.myws = (char*)d_ws_all[rank];
    for (int i = 0; i < NR; ++i) p.ws[i] = (char*)d_ws_all[i];
    hipLaunchKernelGGL(gemm_rs_kernel, dim3(NWG), dim3(512), 0, stream, p);
    HIP_CHECK(hipGetLastError());
}
```

```cpp
#include <hip/hip_runtime.h>
#include <rccl/rccl.h>
#include <cstdio>

#define HIP_CHECK(expr) do { hipError_t _e = (expr); if (_e != hipSuccess) { \
    fprintf(stderr, "HIP error %s at %s:%d\n", hipGetErrorString(_e), __FILE__, __LINE__); \
    return; } } while (0)

namespace {

typedef unsigned u4 __attribute__((ext_vector_type(4)));
typedef float f4 __attribute__((ext_vector_type(4)));
typedef _Float16 h8 __attribute__((ext_vector_type(8)));
#define LAS __attribute__((address_space(3)))
#define GAS __attribute__((address_space(1)))
#define INL __attribute__((always_inline))

constexpr int NR = 8;
constexpr int MG = 4096, NG = 16384, KS = 512, NS = 2048, MS = 512;
constexpr int NWG = 256, NOWN = 32;
constexpr int STAGE_A = 32768, STAGE_B = 16384, STAGE = STAGE_A + STAGE_B, NSTAGE = 3;
constexpr int CTL = NSTAGE * STAGE;
constexpr int LDS_BYTES = CTL + 128;
constexpr unsigned TMO_OFF = 0;
constexpr unsigned LNK_OFF = 3072;
constexpr unsigned ORG_OFF = 1024;
constexpr unsigned SEQ_OFF = 4096;
constexpr unsigned AGG_OFF = 36864;
constexpr unsigned PFL_OFF = 65536;
constexpr unsigned FLG_OFF = 98304, FLG_REP = 4352;
constexpr unsigned RDY_OFF = 147456;
constexpr unsigned EXP_OFF = 524288;
constexpr unsigned STG_OFF = 1u << 20;
constexpr unsigned A_BYTES = 32u << 20;
constexpr unsigned B_BYTES = 16u << 20;
constexpr unsigned W_BYTES = 16u << 20;
constexpr unsigned P_BYTES = 32u << 20;
constexpr unsigned B_REL = A_BYTES, W_REL = B_REL + B_BYTES, P_REL = W_REL + W_BYTES, SLOT_BYTES = P_REL + P_BYTES;
constexpr unsigned WS_NEED = STG_OFF + 2 * SLOT_BYTES;
constexpr unsigned KT_A = 16u * STAGE_A, KT_B = 16u * STAGE_B;
constexpr unsigned OOB = 0x80000000u;
constexpr unsigned OUT_BYTES = (unsigned)MS * NG * 4;
constexpr unsigned long long DEADLINE = 100000000ull;
constexpr int AUX_SYS = 17;
constexpr int AUX_LD = 16;

struct Params {
    const float* x; char* myws; int rank, pad; const float* w;
    char* ws[NR]; float* out;
};

#define WAIT_V(n) asm volatile("s_waitcnt vmcnt(" #n ")" ::: "memory")
constexpr unsigned tau(double gbs) { return (unsigned)(32768.0 / gbs / 10.0 * 256.0); }
constexpr unsigned TAU_ST = tau(58.0), TAU_IN = tau(60.8), TAU_LD = tau(61.0);
constexpr unsigned GAP_ST = 45, GAP_IN = 52, GAP_OUT = 51;
constexpr unsigned RDY_LAT = 180, SLACK = 30;
constexpr unsigned T_START = 400, CONV_TICKS = 192;
constexpr int LEAD = 8;
constexpr unsigned LOW_BITS = (127u - 9u) << 23;
#define BAR() do { asm volatile("" ::: "memory"); __builtin_amdgcn_s_waitcnt(0xC07F);   __builtin_amdgcn_s_barrier(); asm volatile("" ::: "memory"); } while (0)

__device__ __forceinline__ __amdgpu_buffer_rsrc_t mk_rsrc(const void* p, unsigned bytes) {
    return __builtin_amdgcn_make_buffer_rsrc((void*)p, 0, (int)bytes, 0x00020000);
}
__device__ __forceinline__ void st16(u4 v, __amdgpu_buffer_rsrc_t r, unsigned off) {
    __builtin_amdgcn_raw_buffer_store_b128(v, r, (int)off, 0, AUX_SYS);
}
__device__ __forceinline__ u4 ld16(__amdgpu_buffer_rsrc_t r, unsigned off) {
    return __builtin_amdgcn_raw_buffer_load_b128(r, (int)off, 0, AUX_SYS);
}
__device__ __forceinline__ unsigned now32() { return (unsigned)__builtin_amdgcn_s_memrealtime(); }
template <unsigned TAU> __device__ __forceinline__ unsigned slots(unsigned n) { return (n * TAU) >> 8; }
__device__ __forceinline__ void wait_until(unsigned due) {
    while ((int)(now32() - due) < 0) __builtin_amdgcn_s_sleep(1);
    asm volatile("" ::: "memory");
}
template <unsigned TAU> __device__ __forceinline__ void pace(unsigned t0, unsigned n) { wait_until(t0 + slots<TAU>(n)); }
__device__ __forceinline__ void pace_soft(unsigned due, unsigned& next, unsigned gap) {
    if ((int)(next - due) > 0) due = next;
    const unsigned now = now32();
    if ((int)(now - due) > 0) due = now; else wait_until(due);
    asm volatile("" ::: "memory");
    next = due + gap;
}
__device__ __forceinline__ unsigned lds_ld(LAS unsigned* p) {
    return __builtin_amdgcn_readfirstlane(__hip_atomic_load(p, __ATOMIC_RELAXED, __HIP_MEMORY_SCOPE_WORKGROUP));
}
__device__ __forceinline__ unsigned lds_add(LAS unsigned* p, unsigned v, int lane) {
    unsigned old = 0;
    if (lane == 0) old = __hip_atomic_fetch_add(p, v, __ATOMIC_RELAXED, __HIP_MEMORY_SCOPE_WORKGROUP);
    return __builtin_amdgcn_readfirstlane(old) + v;
}
__device__ __forceinline__ u4 cvt8(f4 a, f4 b) {
    h8 h = {(_Float16)a[0], (_Float16)a[1], (_Float16)a[2], (_Float16)a[3], (_Float16)b[0], (_Float16)b[1], (_Float16)b[2], (_Float16)b[3]};
    return __builtin_bit_cast(u4, h);
}
template <int N> __device__ __forceinline__ int block_scale(f4 (&v)[N]) {
    float m = 0.f;
#pragma unroll
    for (int i = 0; i < N; ++i) m = fmaxf(fmaxf(m, fmaxf(fabsf(v[i][0]), fabsf(v[i][1]))), fmaxf(fabsf(v[i][2]), fabsf(v[i][3])));
    int e = 0;
    if (__builtin_expect((__builtin_amdgcn_ballot_w64(m >= 32768.f) != 0) | (__builtin_amdgcn_ballot_w64(m >= __uint_as_float(LOW_BITS)) == 0), 0)) {
        unsigned mb = 0;
#pragma unroll
        for (int i = 0; i < N; ++i)
#pragma unroll
            for (int j = 0; j < 4; ++j) { const unsigned t = __float_as_uint(v[i][j]) & 0x7fffffffu; if (t < 0x7f800000u && t > mb) mb = t; }
#pragma unroll
        for (int o = 32; o > 0; o >>= 1) { const unsigned t = (unsigned)__shfl_xor((int)mb, o); mb = t > mb ? t : mb; }
        mb = __builtin_amdgcn_readfirstlane(mb);
        const int ex = mb >= 0x00800000u ? (int)(mb >> 23) - 127 : -118 - __builtin_clz(mb | 1u);
        e = mb == 0u || (mb >= LOW_BITS && mb < 0x47000000u) ? 0 : (ex - 6) & ~7;
#pragma unroll
        for (int i = 0; i < N; ++i)
#pragma unroll
            for (int j = 0; j < 4; ++j) v[i][j] = ldexpf(v[i][j], -e);
    }
    return e;
}
__device__ __forceinline__ bool give_up(unsigned* tmo, unsigned call, unsigned long long t0, int lane) {
    if (__builtin_amdgcn_readfirstlane(__hip_atomic_load(tmo, __ATOMIC_RELAXED, __HIP_MEMORY_SCOPE_AGENT)) == call
        || __builtin_amdgcn_s_memrealtime() - t0 > DEADLINE) {
        if (lane == 0) __hip_atomic_store(tmo, call, __ATOMIC_RELAXED, __HIP_MEMORY_SCOPE_AGENT);
        return true;
    }
    return false;
}

__global__ __launch_bounds__(512, 2) void gemm_rs_kernel(const Params p) {
    __shared__ __attribute__((aligned(16))) char smem[LDS_BYTES];
    const int tid = threadIdx.x, lane = tid & 63;
    const int wid = __builtin_amdgcn_readfirstlane(tid >> 6);
    const int b = blockIdx.x, rank = p.rank;
    char* const myws = p.myws;
    LAS unsigned* const ctl = (LAS unsigned*)((LAS char*)smem + CTL);
    const unsigned long long tstart = __builtin_amdgcn_s_memrealtime();
    const int xcd = b & 7, slot = b >> 3;
    const int orel = slot >> 2;
    const int owner = (rank + orel) & 7;
    const int hh = xcd & 1, nt = (xcd >> 1) * 4 + (slot & 3);
    const __amdgpu_buffer_rsrc_t rws = mk_rsrc(myws, WS_NEED);
    const int voff = lane * 16;
    unsigned* const tmo = (unsigned*)(myws + TMO_OFF);

    const int sb = b & 31;
    unsigned long long* const torg = (unsigned long long*)(myws + ORG_OFF + sb * 64);
    const float* const xs = p.x + (size_t)((wid * 32 + sb) * 16 + (lane >> 2)) * KS + ((lane & 3) ^ ((lane >> 5) << 1)) * 4;
    const float* const wsrc = p.w + (size_t)((lane >> 4) * 4) * NG + wid * NS + sb * 64 + (lane & 15) * 4;
    u4 xh[2] = {}, wh[8] = {};
    f4 xr[4], wr[16];
    int xe = 0, we = 0;
    auto load_x = [&](int c) INL {
#pragma unroll
        for (int i = 0; i < 4; ++i) xr[i] = *(const f4*)(xs + c * 64 + i * 16);
    };
    auto load_w = [&](int c) INL {
#pragma unroll
        for (int i = 0; i < 16; ++i) wr[i] = *(const f4*)(wsrc + (size_t)(c * 64 + (i >> 2) * 16 + (i & 3)) * NG);
    };
    auto cvt_x = [&]() INL { xe = block_scale(xr); xh[0] = cvt8(xr[0], xr[1]); xh[1] = cvt8(xr[2], xr[3]); };
    auto cvt_w = [&]() INL {
        we = block_scale(wr);
#pragma unroll
        for (int ks = 0; ks < 2; ++ks)
#pragma unroll
            for (int jj = 0; jj < 4; ++jj) {
                const f4 lo = {wr[ks * 8][jj], wr[ks * 8 + 1][jj], wr[ks * 8 + 2][jj], wr[ks * 8 + 3][jj]};
                const f4 hi = {wr[ks * 8 + 4][jj], wr[ks * 8 + 5][jj], wr[ks * 8 + 6][jj], wr[ks * 8 + 7][jj]};
                wh[ks * 4 + jj] = cvt8(lo, hi);
            }
    };
    const int uc = (b >> 5) == 7 ? 0 : b >> 5;
    const bool ux = b < 224;
    unsigned call, call2, stg, rdyo, expo, xoff;
    auto read_call = [&]() INL {
        asm volatile("s_load_dword %0, %1, 0x0\n\ts_waitcnt lgkmcnt(0)" : "=s"(call) : "s"(myws + SEQ_OFF + b * 128) : "memory");
        ++call; call2 = 2u * call;
        stg = STG_OFF + (call & 1u) * SLOT_BYTES;
        rdyo = RDY_OFF + (call & 1u) * 16384u + (unsigned)sb * 512u; expo = EXP_OFF + (call & 1u) * 262144u;
        xoff = stg + (unsigned)(rank * 8) * KT_A + (unsigned)(wid * 32 + sb) * 2048u + voff;
    };
    auto first_barrier = [&]() INL {
        if (wid == rank && lane < 32) ctl[lane] = 0;
        BAR();
        if (wid == rank && lane == 0) __hip_atomic_store((unsigned*)(myws + SEQ_OFF + b * 128), call, __ATOMIC_RELAXED, __HIP_MEMORY_SCOPE_AGENT);
    };

    auto arrive = [&](bool on, int fw, unsigned weight, bool scaled) INL {
        if (on) {
            unsigned long long* cnt = (unsigned long long*)(myws + AGG_OFF + fw * 128);
            const unsigned long long add = weight | ((unsigned long long)scaled << 32);
            const unsigned long long sum = __hip_atomic_fetch_add(cnt, add, __ATOMIC_RELAXED, __HIP_MEMORY_SCOPE_AGENT) + add;
            if (((unsigned)sum & 0x7fffffffu) == ((call * 64u) & 0x7fffffffu)) {
#pragma unroll
                for (int r = 0; r < 8; ++r)
                    __hip_atomic_store((unsigned*)(myws + FLG_OFF + r * FLG_REP) + fw, call2 + (unsigned)((sum >> 32) != 0), __ATOMIC_RELAXED, __HIP_MEMORY_SCOPE_AGENT);
                if (sum >> 31) __hip_atomic_fetch_and(cnt, 0x7fffffffull, __ATOMIC_RELAXED, __HIP_MEMORY_SCOPE_AGENT);
            }
        }
    };
    auto lane_ws = [&]() INL {
        char* w = p.ws[0];
#pragma unroll
        for (int i = 1; i < NR; ++i) if (lane == i) w = p.ws[i];
        return w;
    };
    auto tell_peers = [&](int word, bool scaled) INL {
        char* w = lane_ws();
        if (lane < NR && lane != rank) __hip_atomic_store((GAS unsigned*)(w + rdyo) + word, call2 + (unsigned)scaled, __ATOMIC_RELAXED, __HIP_MEMORY_SCOPE_SYSTEM);
    };
    auto store_e = [&](int which, int c, int e) INL {
        if (__builtin_expect(e != 0, 0)) {
            char* w = lane_ws();
            if (lane < NR) __hip_atomic_store((GAS unsigned long long*)(w + expo) + (((rank * 2 + which) * 8 + c) * 32 + sb) * 8 + wid,
                                              (unsigned long long)call << 32 | (unsigned)e, __ATOMIC_RELAXED, __HIP_MEMORY_SCOPE_SYSTEM);
        }
    };
    auto store_x = [&](int c) INL { st16(xh[0], rws, xoff + c * KT_A); st16(xh[1], rws, xoff + c * KT_A + 1024); store_e(0, c, xe); };
    auto store_w = [&](int c) INL {
        const unsigned wo = stg + (wid == rank ? B_REL : W_REL) + (unsigned)(wid * 8 + c) * KT_B + (unsigned)sb * 8192u + voff;
#pragma unroll
        for (int q = 0; q < 8; ++q) st16(wh[q], rws, wo + q * 1024);
        store_e(1, c, we);
    };

    if (__builtin_expect(b >= NOWN, 0)) {
        if (b >= 64 && b < 224) {
            if (wid == ((rank + 1) & 7)) {
                const int j = (b >> 3) & 7;
                const __amdgpu_buffer_rsrc_t rj = b < 192 ? mk_rsrc(p.ws[j], WS_NEED) : mk_rsrc(p.x, (unsigned)MG * KS * 4);
                const unsigned par = STG_OFF + (lane & 1) * SLOT_BYTES, q = lane >> 1;
                unsigned o;
                if (b < 128) o = q == 0 ? par + (unsigned)(rank * 8) * KT_A : q == 1 ? par + (unsigned)(rank * 8 + 1) * KT_A - 4u : OOB;
                else if (b >= 192) o = lane < 2 ? (unsigned)(3 - j) * 0x200000u + lane * 0x1ffffcu : OOB;
                else if (j == rank) o = STG_OFF + (unsigned)lane * 0x200000u;
                else o = q < 2 ? par + W_REL + (unsigned)(rank * 8) * KT_B + q * 0x1ffffcu : q == 2 ? RDY_OFF : q < 6 ? par + (unsigned)(j * 8) * KT_A + (q - 3) * 0x1ffffcu
                       : q < 9 ? par + P_REL + (unsigned)rank * 0x400000u + (q - 6) * 0x1ffffcu : OOB;
                unsigned t = __builtin_amdgcn_raw_buffer_load_b32(rj, (int)o, 0, AUX_SYS);
                if (b >= 128 && b < 192 && j == rank) {
                    t += __builtin_amdgcn_raw_buffer_load_b32(rj, (int)(o + 64u * 0x200000u), 0, AUX_SYS);
                    t += __builtin_amdgcn_raw_buffer_load_b32(mk_rsrc(p.out, OUT_BYTES), lane * 0x1ffffc, 0, AUX_SYS);
                    t += __builtin_amdgcn_raw_buffer_load_b32(mk_rsrc(p.w, (unsigned)KS * NG * 4), lane * 0x1ffffc, 0, 0);
                }
                asm volatile("" :: "v"(t));
            }
            wait_until((unsigned)tstart + T_START + (unsigned)(b - 64) * CONV_TICKS / 16u);
        }
        if (ux) load_x(uc);
        load_w(uc);
        read_call();
        first_barrier();
        if (ux) { cvt_x(); store_x(uc); }
        cvt_w(); store_w(uc);
        WAIT_V(0);
        unsigned sum = lds_add(ctl + 0, (xe | we) ? 257u : 1u, lane);
        if ((sum & 255u) == 8u) {
            arrive(lane == 0, rank * 8 + uc, ux ? 2u : 1u, sum > 255u);
            if (!ux) {
                unsigned long long o;
                bool late = false;
                while ((unsigned)((o = __hip_atomic_load(torg, __ATOMIC_RELAXED, __HIP_MEMORY_SCOPE_AGENT)) >> 32) != call) {
                    if (give_up(tmo, call, tstart, lane)) { late = true; break; }
                    __builtin_amdgcn_s_sleep(4);
                }
                if (!late) wait_until(__builtin_amdgcn_readfirstlane((unsigned)o) + slots<TAU_ST>(12u) - RDY_LAT);
            }
            tell_peers(rank * 8 + uc, sum > 255u);
        }
        if (!ux) {
            wait_until((unsigned)tstart + T_START + (unsigned)(b - 64) * CONV_TICKS / 16u);
            load_x(7); load_w(7);
            cvt_x(); store_x(7); cvt_w(); store_w(7);
            WAIT_V(0);
            sum = lds_add(ctl + 1, (xe | we) ? 257u : 1u, lane);
            if ((sum & 255u) == 8u) { arrive(lane == 0, rank * 8 + 7, 2u, sum > 255u); tell_peers(rank * 8 + 7, sum > 255u); }
        }
    } else {
        load_x(0);
        const int u = (wid - rank - 1) & 7;
        unsigned xg[6];
        char* pw[6];
#pragma unroll
        for (int s = 0; s < 6; ++s) { int a = u + 1 + s; if (a >= 7) a -= 7; xg[s] = (unsigned)(rank + 1 + a) & 7u; pw[s] = p.ws[xg[s]]; }
        asm volatile("" : "+s"(pw[0]), "+s"(pw[1]), "+s"(pw[2]), "+s"(pw[3]), "+s"(pw[4]), "+s"(pw[5]));
        read_call();
        cvt_x();
        asm volatile("" : "+v"(xh[0]), "+v"(xh[1]));
        const unsigned long long t64 = __builtin_amdgcn_s_memrealtime();
        if (__builtin_expect(wid == rank, 0)) {
            store_x(0);
            first_barrier();
            WAIT_V(0);
            lds_add(ctl + 0, xe ? 257u : 1u, lane);
            const unsigned* f = (const unsigned*)(myws + rdyo);
            bool told = false, all = false;
            unsigned fx = 0, ann = lane < NR && lane != rank ? 0u : 10u;
            unsigned long long scl = 0;
            pace<TAU_ST>((unsigned)t64, 6);
            for (unsigned it = 1;; ++it) {
                if (!told) {
                    const unsigned sum = lds_ld(ctl + 0);
                    if ((sum & 255u) == 8u) { told = true; tell_peers(64 + rank, sum > 255u); arrive(lane == 0, rank * 8, 1u, sum > 255u); }
                }
                if (!all && (fx != 0xffu || (it & 7u) == 0)) {
                    const unsigned v = __hip_atomic_load(f + lane, __ATOMIC_RELAXED, __HIP_MEMORY_SCOPE_SYSTEM);
                    const unsigned vx = __hip_atomic_load(f + 64 + (lane >> 3), __ATOMIC_RELAXED, __HIP_MEMORY_SCOPE_SYSTEM);
                    unsigned long long full = __ballot((lane >> 3) == rank || (int)(v - call2) >= 0);
                    const unsigned long long fx8 = __ballot((lane >> 3) == rank || (int)(vx - call2) >= 0);
                    scl = __ballot(v == call2 + 1u || ((lane & 7) == 0 && vx == call2 + 1u));
                    fx = (unsigned)__ballot((fx8 >> ((lane & 7) * 8)) & 1u) & 0xffu;
                    if ((it & 31u) == 0 && give_up(tmo, call, tstart, lane)) { full = ~0ull; fx = 0xffu; }
                    if (lane < 3) __hip_atomic_store(ctl + 4 + lane, lane == 2 ? fx : (unsigned)(full >> (lane * 32)), __ATOMIC_RELAXED, __HIP_MEMORY_SCOPE_WORKGROUP);
                    all = full == ~0ull && fx == 0xffu;
                }
                const unsigned prog = __hip_atomic_load(ctl + 24 + (lane & 7), __ATOMIC_RELAXED, __HIP_MEMORY_SCOPE_WORKGROUP);
                const bool on = prog > ann && ((fx >> (lane & 7)) & 1u);
                const unsigned sc = (unsigned)(scl >> ((lane & 7) * 8)) & 0xffu;
                arrive(on, ann < 7u ? lane * 8 + (int)ann : ann == 7u ? 72 + lane : ann == 8u ? lane * 8 + 7 : 64 + lane, 2u,
                       ann < 9u ? (sc >> (ann < 7u ? ann : 7u)) & 1u : sc != 0u);
                if (on) ++ann;
                if (told && __all(ann == 10u)) break;
                __builtin_amdgcn_s_sleep(1);
            }
        } else {
            {
                unsigned nx = (unsigned)t64;
                __amdgpu_buffer_rsrc_t rd[6];
#pragma unroll
                for (int s = 0; s < 6; ++s) rd[s] = mk_rsrc(pw[s], WS_NEED);
#pragma unroll
                for (int i = 0; i < 12; ++i) {
                    if (i > 0) pace_soft((unsigned)t64 + slots<TAU_ST>(i), nx, GAP_ST);
                    st16(xh[i & 1], rd[i >> 1], xoff + (i & 1) * 1024);
                    if (i == 0) {
                        store_x(0);
                        if (u == 0 && lane == 0) __hip_atomic_store(torg, (unsigned long long)call << 32 | (unsigned)t64, __ATOMIC_RELAXED, __HIP_MEMORY_SCOPE_AGENT);
                    } else WAIT_V(1);
                    if (i == 3) first_barrier();
                }
            }
            const __amdgpu_buffer_rsrc_t rs = mk_rsrc(p.ws[wid], WS_NEED);
            const unsigned xo = stg + (unsigned)(wid * 8) * KT_A + (unsigned)sb * 2048u + voff;
            const unsigned wro = stg + W_REL + (unsigned)(rank * 8) * KT_B + (unsigned)sb * 8192u + voff;
            const unsigned wlo = stg + B_REL + (unsigned)(wid * 8) * KT_B + (unsigned)sb * 8192u + voff;
            unsigned ts = 0;
            auto need = [&](int word, int bit) INL {
                while (!((lds_ld(ctl + word) >> bit) & 1u)) __builtin_amdgcn_s_sleep(1);
                asm volatile("" ::: "memory");
            };
            need(4 + (wid >> 2), (wid & 3) * 8);
            {
                unsigned long long t = __builtin_amdgcn_s_memrealtime();
                const unsigned long long tp = t64 + slots<TAU_ST>(12u);
                if (t < tp + SLACK) t = tp;
                ts = (unsigned)t - slots<TAU_IN>(12u);
                if (lane == 0) __hip_atomic_store((unsigned*)(myws + LNK_OFF) + wid * 32 + sb, (unsigned)t + slots<TAU_IN>(164u), __ATOMIC_RELAXED, __HIP_MEMORY_SCOPE_AGENT);
            }
            unsigned next = ts;
            u4 d[2] = {};
            unsigned lo[2] = {OOB, OOB};
            auto progress = [&](unsigned v) INL { if (lane == 0) __hip_atomic_store(ctl + 24 + wid, v, __ATOMIC_RELAXED, __HIP_MEMORY_SCOPE_WORKGROUP); };
            auto slot = [&](int i, unsigned n, unsigned ro, unsigned l) INL {
                pace_soft(ts + slots<TAU_IN>(n), next, GAP_IN);
                lo[i & 1] = l;
                d[i & 1] = ld16(rs, ro);
                asm volatile("" ::: "memory");
                st16(d[(i & 1) ^ 1], rws, lo[(i & 1) ^ 1]);
                asm volatile("" ::: "memory");
            };
#pragma unroll
            for (int i = 12; i < 20; ++i) {
                slot(i, i, wro + (i - 12) * 1024, wlo + (i - 12) * 1024);
                if (i == 12) { WAIT_V(2); lds_add(ctl + 0, xe ? 257u : 1u, lane); }
            }
            for (int c = 1; c < 8; ++c) {
                need(4 + (wid >> 2), (wid & 3) * 8 + c);
#pragma unroll
                for (int i = 0; i < 20; ++i) {
                    const int ks = i / 10, r = i % 10;
                    if (r < 6) { const unsigned o = xo + c * KT_A + xg[r] * 65536u + ks * 1024; slot(i, c * 20 + i, o, o); }
                    else slot(i, c * 20 + i, wro + c * KT_B + (ks * 4 + r - 6) * 1024, wlo + c * KT_B + (ks * 4 + r - 6) * 1024);
                    if (i == 1) { WAIT_V(2); progress(c); }
                    if (i == 11 && c == 7) { WAIT_V(2); progress(8); }
                }
            }
            need(6, wid);
#pragma unroll
            for (int i = 0; i < 16; ++i) {
                const unsigned o = xo + (unsigned)(i >> 1) * KT_A + (unsigned)rank * 65536u + (i & 1) * 1024;
                slot(i, 160 + i, o, o);
                if (i == 0) { WAIT_V(0); progress(9); }
            }
            st16(d[1], rws, lo[1]);
            WAIT_V(0);
            progress(10);
        }
        BAR();
    }

    const int wm = wid >> 1, wn = wid & 1;
    const int lvoff = wid * 1024 + voff;
    const int aoff = (lane & 15) * 64 + (((lane >> 4) ^ ((lane >> 2) & 2)) << 4);
    int fw0, fw1, tsrc;
    unsigned taba, tabb;
    {
        const int src = lane >> 3, c = lane & 7;
        fw0 = fw1 = orel == 0 && src != rank ? 64 + src : lane;
        tsrc = src * 17;
        taba = stg + (unsigned)lane * KT_A + (unsigned)(owner * 2 + hh) * STAGE_A;
        tabb = stg + B_REL + (unsigned)lane * KT_B + (unsigned)nt * STAGE_B;
        if (orel != 0 && src == owner) {
            fw0 = fw1 = rank * 8 + c;
            tsrc = rank * 17;
            taba = stg + (unsigned)(rank * 8 + c) * KT_A + (unsigned)(rank * 2 + hh) * STAGE_A;
            tabb = stg + W_REL + (unsigned)lane * KT_B + (unsigned)nt * STAGE_B;
        } else if (orel != 0 && c == 7 && src != rank) {
            const int t = (src - rank) & 7, k = t < orel ? t - 1 : t - 2, ka = k & ~1, kb = k | 1, h = k & 1;
            const int sa = (rank + (ka + 1 < orel ? ka + 1 : ka + 2)) & 7, sb2 = (rank + (kb + 1 < orel ? kb + 1 : kb + 2)) & 7;
            fw0 = h ? sa * 8 + 7 : 72 + sa; fw1 = h ? sb2 * 8 + 7 : 72 + sb2;
            tsrc = sa | sb2 << 4;
            taba += (unsigned)((((wid & 1) ? sb2 : sa) - src) * 8) * KT_A + (unsigned)((h - (wid & 1)) * 1024);
            tabb += (unsigned)((((wid >> 2) ? sb2 : sa) - src) * 8) * KT_B + (unsigned)((h - (wid >> 2)) * 4096);
        }
    }
    const unsigned* const myflags = (const unsigned*)(myws + FLG_OFF + xcd * FLG_REP);
    f4 acc[4][4], acc2[4][4];
#pragma unroll
    for (int m = 0; m < 4; ++m)
#pragma unroll
        for (int n = 0; n < 4; ++n) { acc[m][n] = (f4){0.f, 0.f, 0.f, 0.f}; acc2[m][n] = (f4){0.f, 0.f, 0.f, 0.f}; }

    auto stage1 = [&](unsigned ao, unsigned bo, int buf, int i) INL {
        LAS char* const l = (LAS char*)smem + buf * STAGE + wid * 1024 + i * 8192;
        __builtin_amdgcn_raw_ptr_buffer_load_lds(rws, l, 16, lvoff, i < 4 ? ao + i * 8192 : bo + (i - 4) * 8192, 0, AUX_LD);
    };
    auto stage = [&](int kt, int buf) INL {
        const unsigned ao = __builtin_amdgcn_readlane(taba, kt), bo = __builtin_amdgcn_readlane(tabb, kt);
#pragma unroll
        for (int i = 0; i < 6; ++i) stage1(ao, bo, buf, i);
    };
    auto frag = [&](h8 (&af)[4], h8 (&bf)[4], int buf, int ks, int g) INL {
        const LAS char* const l = (const LAS char*)smem + buf * STAGE;
        if (g < 4) bf[g] = *(const LAS h8*)(l + STAGE_A + wn * 8192 + ks * 4096 + g * 1024 + voff);
        else af[g - 4] = *(const LAS h8*)(l + wm * 8192 + ks * 1024 + (g - 4) * 2048 + aoff);
    };
    auto kstep = [&](f4 (&c)[4][4], const h8 (&af)[4], const h8 (&bf)[4], h8 (&an)[4], h8 (&bn)[4], int rbuf, int rks,
                     bool dma, unsigned ao, unsigned bo, int sbuf) INL {
#pragma unroll
        for (int g = 0; g < 8; ++g) {
            const int m = g >> 1, n0 = (g & 1) * 2;
            c[m][n0] = __builtin_amdgcn_mfma_f32_16x16x32_f16(af[m], bf[n0], c[m][n0], 0, 0, 0);
            if (g < 2) { frag(an, bn, rbuf, rks, 2 * g); frag(an, bn, rbuf, rks, 2 * g + 1); }
            else if (g < 6) frag(an, bn, rbuf, rks, g + 2);
            c[m][n0 + 1] = __builtin_amdgcn_mfma_f32_16x16x32_f16(af[m], bf[n0 + 1], c[m][n0 + 1], 0, 0, 0);
            if (dma && g >= 2) stage1(ao, bo, sbuf, g - 2);
            __builtin_amdgcn_sched_barrier(0);
        }
    };
    int buf = 0;
    unsigned long long done = 0, seen = 0;
    const unsigned* const pf0 = myflags + fw0;
    const unsigned* const pf1 = myflags + fw1;
    auto run = [&](f4 (&c)[4][4], unsigned long long kind) INL {
        unsigned long long avail = seen & kind & ~done;
        done |= avail;
        stage(__builtin_ctzll(avail), buf); avail &= avail - 1;
        int sbuf = buf + 1 == NSTAGE ? 0 : buf + 1, ahead = 0, par = 0;
        if (avail) { stage(__builtin_ctzll(avail), sbuf); avail &= avail - 1; sbuf = sbuf + 1 == NSTAGE ? 0 : sbuf + 1; ahead = 1; WAIT_V(6); }
        else {
            if (wid == 0) {
                __builtin_amdgcn_s_sleep(12);
                const unsigned v0 = __hip_atomic_load(pf0, __ATOMIC_RELAXED, __HIP_MEMORY_SCOPE_AGENT);
                const unsigned v1 = __hip_atomic_load(pf1, __ATOMIC_RELAXED, __HIP_MEMORY_SCOPE_AGENT);
                const unsigned long long m = __ballot(v0 == call2 && v1 == call2);
                if (lane < 2) ctl[14 + lane] = (unsigned)(m >> (lane * 32));
            }
            WAIT_V(0);
        }
        BAR();
        h8 a0[4], b0[4], a1[4], b1f[4];
#pragma unroll
        for (int g = 0; g < 8; ++g) frag(a0, b0, buf, 0, g);
        if (ahead == 0) {
            avail = (((unsigned long long)lds_ld(ctl + 15) << 32) | lds_ld(ctl + 14)) & kind & ~done;
            done |= avail;
        }
        bool more;
        do {
            unsigned v0 = 0, v1 = 0;
            if (wid == 0) asm volatile("global_load_dword %0, %2, off sc1\n\tglobal_load_dword %1, %3, off sc1" : "=&v"(v0), "=&v"(v1) : "v"(pf0), "v"(pf1) : "memory");
            if (ahead == 0 && avail) { stage(__builtin_ctzll(avail), sbuf); avail &= avail - 1; sbuf = sbuf + 1 == NSTAGE ? 0 : sbuf + 1; ahead = 1; }
            more = ahead != 0;
            const int kt = __builtin_ctzll(avail | (1ull << 63));
            const unsigned ao = avail ? __builtin_amdgcn_readlane(taba, kt) : OOB, bo = avail ? __builtin_amdgcn_readlane(tabb, kt) : OOB;
            const int b1 = buf + 1 == NSTAGE ? 0 : buf + 1;
            kstep(c, a0, b0, a1, b1f, buf, 1, true, ao, bo, sbuf);
            if (avail) { avail &= avail - 1; sbuf = sbuf + 1 == NSTAGE ? 0 : sbuf + 1; } else ahead = 0;
            WAIT_V(6);
            if (wid == 0) {
                asm volatile("" : "+v"(v0), "+v"(v1));
                const unsigned long long m = __ballot(v0 == call2 && v1 == call2);
                if (lane < 2) ctl[8 + par + lane] = (unsigned)(m >> (lane * 32));
            }
            BAR();
            const unsigned mlo = ctl[8 + par], mhi = ctl[9 + par];
            kstep(c, a1, b1f, a0, b0, b1, 0, false, 0, 0, 0);
            seen = ((unsigned long long)__builtin_amdgcn_readfirstlane(mhi) << 32) | (unsigned)__builtin_amdgcn_readfirstlane(mlo);
            if (more) { avail |= seen & kind & ~done; done |= avail; }
            par ^= 2;
            buf = b1;
        } while (more);
    };

    auto run_scaled = [&](f4 (&c)[4][4], unsigned long long todo, int jx, int jw) INL {
        done |= todo;
        stage(__builtin_ctzll(todo), buf);
        while (todo) {
            const int kt = __builtin_ctzll(todo);
            todo &= todo - 1;
            const int src = (__builtin_amdgcn_readlane(tsrc, kt) >> ((lane & 8) >> 1)) & 7;
            const int idx = (lane & 4) == 0 ? ((src * 16 + (kt & 7)) * 32 + hh * 16 + wm * 4 + (lane & 3)) * 8 + jx
                                            : ((src * 16 + 8 + (kt & 7)) * 32 + nt * 2 + wn) * 8 + jw;
            const unsigned long long ev = __hip_atomic_load((const unsigned long long*)(myws + expo) + idx, __ATOMIC_RELAXED, __HIP_MEMORY_SCOPE_SYSTEM);
            int e = (unsigned)(ev >> 32) == call ? (int)(unsigned)ev : 0;
            asm volatile("" : "+v"(e));
            WAIT_V(0);
            BAR();
            const int b1 = buf + 1 == NSTAGE ? 0 : buf + 1;
            if (todo) stage(__builtin_ctzll(todo), b1);
#pragma unroll
            for (int ks = 0; ks < 2; ++ks) {
                h8 af[4], bf[4];
#pragma unroll
                for (int g = 0; g < 8; ++g) frag(af, bf, buf, ks, g);
#pragma unroll
                for (int m = 0; m < 4; ++m) {
                    const int em = __builtin_amdgcn_readlane(e, ks * 8 + m) + __builtin_amdgcn_readlane(e, ks * 8 + 4);
#pragma unroll
                    for (int n = 0; n < 4; ++n) {
                        const f4 t = __builtin_amdgcn_mfma_f32_16x16x32_f16(af[m], bf[n], (f4){0.f, 0.f, 0.f, 0.f}, 0, 0, 0);
#pragma unroll
                        for (int r = 0; r < 4; ++r) c[m][n][r] += ldexpf(t[r], em);
                    }
                }
            }
            buf = b1;
        }
        BAR();
    };

    const unsigned long long locm = orel == 0 ? 0ull : 0xffull << (owner * 8);
    bool bad = false;
    while (done != ~0ull) {
        unsigned long long scaled = 0;
        if (!(seen & ~done)) {
            if (wid == 0) {
                const unsigned long long t0 = __builtin_amdgcn_s_memrealtime();
                unsigned long long m, ms;
                for (unsigned it = 1;; ++it) {
                    const unsigned v0 = __hip_atomic_load(pf0, __ATOMIC_RELAXED, __HIP_MEMORY_SCOPE_AGENT);
                    const unsigned v1 = __hip_atomic_load(pf1, __ATOMIC_RELAXED, __HIP_MEMORY_SCOPE_AGENT);
                    m = __ballot(v0 == call2 && v1 == call2);
                    ms = __ballot((v0 | 1u) == call2 + 1u && (v1 | 1u) == call2 + 1u);
                    if ((ms & ~done) || ((it & 63u) == 0 && give_up(tmo, call, t0, lane))) break;
                    __builtin_amdgcn_s_sleep(4);
                }
                if (lane == 0) { ctl[16] = (unsigned)m; ctl[17] = (unsigned)(m >> 32); ctl[20] = (unsigned)ms; ctl[21] = (unsigned)(ms >> 32); }
            }
            BAR();
            seen = ((unsigned long long)lds_ld(ctl + 17) << 32) | lds_ld(ctl + 16);
            scaled = (((unsigned long long)lds_ld(ctl + 21) << 32) | lds_ld(ctl + 20)) & ~done;
            if (!scaled) { bad = true; done = ~0ull; }
            scaled &= ~seen;
        }
        if (seen & ~done & locm) run(acc2, locm);
        if (seen & ~done & ~locm) run(acc, ~locm);
        if (__builtin_expect((scaled & locm) != 0, 0)) run_scaled(acc2, scaled & locm, rank, owner);
        if (__builtin_expect((scaled & ~locm) != 0, 0)) run_scaled(acc, scaled & ~locm, owner, rank);
    }

    const float qn = __builtin_nanf("");
    const __amdgpu_buffer_rsrc_t rout = mk_rsrc(p.out, OUT_BYTES);
    const unsigned oo = ((unsigned)(hh * 256 + wm * 64 + (lane >> 4) * 4) * NG + (unsigned)(owner * NS + nt * 128 + wn * 64 + (lane & 15) * 4)) * 4u;
    auto put = [&](int m) INL {
#pragma unroll
        for (int r = 0; r < 4; ++r) {
            f4 v = {acc[m][0][r] + acc2[m][0][r], acc[m][1][r] + acc2[m][1][r], acc[m][2][r] + acc2[m][2][r], acc[m][3][r] + acc2[m][3][r]};
            if (bad) v = (f4){qn, qn, qn, qn};
            st16(__builtin_bit_cast(u4, v), rout, oo + (unsigned)(m * 16 + r) * (NG * 4u));
        }
    };
    if (orel != 0) {
        const unsigned tile = (unsigned)(hh * 16 + nt);
        {
            if (wid > 0) pace<TAU_LD>(now32(), wid * 16 - LEAD);
            if (bad) {
#pragma unroll
                for (int q = 0; q < 16; ++q) acc[q >> 2][q & 3] = (f4){qn, qn, qn, qn};
            }
            const unsigned po = stg + P_REL + (((unsigned)owner * 32u + tile) * 8u + (unsigned)wid) * 16384u + voff;
#pragma unroll
            for (int q = 0; q < 16; ++q) st16(__builtin_bit_cast(u4, acc[q >> 2][q & 3]), rws, po + q * 1024);
            WAIT_V(0);
            if (lane == 0) __hip_atomic_store((unsigned*)(p.ws[owner] + PFL_OFF + ((unsigned)rank * 32u + tile) * 128u) + wid, call, __ATOMIC_RELAXED, __HIP_MEMORY_SCOPE_SYSTEM);
        }
        {
            const unsigned* f = (const unsigned*)(myws + PFL_OFF + ((unsigned)owner * 32u + tile) * 128u) + wid;
            const unsigned long long t0 = __builtin_amdgcn_s_memrealtime();
            unsigned tv;
            for (unsigned it = 1;; ++it) {
                const unsigned v = __hip_atomic_load(f, __ATOMIC_RELAXED, __HIP_MEMORY_SCOPE_SYSTEM);
                const unsigned tm = __hip_atomic_load(tmo, __ATOMIC_RELAXED, __HIP_MEMORY_SCOPE_AGENT);
                tv = __hip_atomic_load((unsigned*)(myws + LNK_OFF) + owner * 32 + (lane & 31), __ATOMIC_RELAXED, __HIP_MEMORY_SCOPE_AGENT);
                if (__builtin_amdgcn_readfirstlane(tm) == call) { bad = true; break; }
                if ((int)(__builtin_amdgcn_readfirstlane(v) - call) >= 0) break;
                if ((it & 15u) == 0 && give_up(tmo, call, t0, lane)) { bad = true; break; }
                __builtin_amdgcn_s_sleep(1);
            }
            const unsigned tf = now32();
            if (tv - tf + 20000u > 40000u) tv = tf;
            const unsigned tl = __builtin_amdgcn_readlane(tv, tile);
            while (lds_ld(ctl + 12) < (unsigned)wid) __builtin_amdgcn_s_sleep(1);
            unsigned te = wid > 0 ? lds_ld(ctl + 13) : tl;
            if ((int)(tf - te - slots<TAU_LD>(wid * 16)) > 0) te = tf - slots<TAU_LD>(wid * 16);
            unsigned tlast = te + 30000u;
            if (wid == 7) {
                int dm = (int)(tv - tf);
#pragma unroll
                for (int o = 16; o > 0; o >>= 1) dm = min(dm, __shfl_xor(dm, o));
                tlast = tf + (unsigned)__builtin_amdgcn_readfirstlane(dm) + slots<TAU_LD>(127u);
            }
            unsigned next = te;
            const __amdgpu_buffer_rsrc_t ro = mk_rsrc(p.ws[owner], WS_NEED);
            const unsigned pi = stg + P_REL + (((unsigned)rank * 32u + tile) * 8u + (unsigned)wid) * 16384u + voff;
#pragma unroll
            for (int q = 0; q < 16; ++q) {
                const unsigned due = te + slots<TAU_LD>(wid * 16 + q);
                if ((int)(due - tlast) > 0) wait_until(tlast); else pace_soft(due, next, GAP_OUT);
                acc[q >> 2][q & 3] = __builtin_bit_cast(f4, ld16(ro, pi + q * 1024));
                if ((q & 3) == 1 && q > 4) WAIT_V(5); else WAIT_V(1);
                if ((q & 3) == 0 && q > 0) put((q >> 2) - 1);
            }
            if (lane == 0) {
                __hip_atomic_store(ctl + 13, te, __ATOMIC_RELAXED, __HIP_MEMORY_SCOPE_WORKGROUP);
                __hip_atomic_store(ctl + 12, (unsigned)wid + 1u, __ATOMIC_RELAXED, __HIP_MEMORY_SCOPE_WORKGROUP);
            }
            put(3);
        }
    } else {
        if (!bad) bad = __builtin_amdgcn_readfirstlane(__hip_atomic_load(tmo, __ATOMIC_RELAXED, __HIP_MEMORY_SCOPE_AGENT)) == call;
#pragma unroll
        for (int m = 0; m < 4; ++m) put(m);
    }
}

}

extern "C" void kernel_launch(int rank, int nranks, ncclComm_t comm,
                              hipStream_t stream,
                              void* const* d_in, const int* in_sizes, int n_in,
                              void* d_out, int out_size,
                              const int* in_dtypes, int out_dtype,
                              void* const* d_out_all,
                              void* const* d_ws_all, size_t ws_bytes) {
    (void)comm; (void)in_dtypes; (void)out_dtype; (void)d_out_all;
    if (nranks != NR || (unsigned)rank >= (unsigned)NR || n_in < 2 || in_sizes[0] != MG * KS || in_sizes[1] != KS * NG
        || out_size != MS * NG || ws_bytes < WS_NEED) {
        fprintf(stderr, "kernel_launch: unexpected problem shape\n");
        return;
    }
    Params p;
    p.x = (const float*)d_in[0]; p.w = (const float*)d_in[1]; p.out = (float*)d_out; p.rank = rank; p.pad = 0; p.myws = (char*)d_ws_all[rank];
    for (int i = 0; i < NR; ++i) p.ws[i] = (char*)d_ws_all[i];
    hipLaunchKernelGGL(gemm_rs_kernel, dim3(NWG), dim3(512), 0, stream, p);
    HIP_CHECK(hipGetLastError());
}
```
